```python
import numpy as np

kernel_name = "gpu_a2a_gemm_m4096_k4096_n16384_e4m3e4m3_dqrelu"


ABSMAX_RTOL = 0.03

COMPUTE_DTYPE = "float32"
EPILOGUE = "dequant_relu"
IN_QUANT = None
OUT_QUANT = None
IN_FP8 = "e4m3e4m3"
IN_DTYPES = {"x": "e4m3", "w_mat": "e4m3",
             "scale_x": "float32", "scale_w": "float32"}
WORLD_SIZE = 8

def _snap_fp8(x, mant_bits, max_abs, min_normal):
    a = np.abs(x.astype(np.float64))
    _m, _e = np.frexp(a)
    q = np.where(a >= min_normal, np.ldexp(1.0, _e - 1 - mant_bits),
                 min_normal * 2.0 ** -mant_bits)
    snapped = np.minimum(np.round(a / q) * q, max_abs)
    return (np.sign(x) * snapped).astype(np.float32)

def setup_inputs():

    x = _snap_fp8(np.random.randn(4096, 4096).astype(np.float32), 3, 448.0, 2.0 ** -6)
    w_mat = _snap_fp8(np.random.randn(4096, 16384).astype(np.float32), 3, 448.0, 2.0 ** -6)
    scale_x = (0.02 * (1.0 + np.random.rand(1))).astype(np.float32)
    scale_w = (0.02 * (1.0 + np.random.rand(1))).astype(np.float32)
    return {"x": x, "w_mat": w_mat, "scale_x": scale_x, "scale_w": scale_w}

def reference(x, w_mat, scale_x, scale_w):
    acc = (x @ w_mat).astype(np.float32)
    y = (acc * (scale_x * scale_w)).astype(np.float32)
    return np.maximum(y, 0.0).astype(np.float32)
```

```cpp
#include <hip/hip_runtime.h>
#include <rccl/rccl.h>
#include <cstdio>
#include <cstdint>

#define HIP_CHECK(expr) do { hipError_t _e = (expr); if (_e != hipSuccess) { \
    fprintf(stderr, "HIP error %s at %s:%d\n", hipGetErrorString(_e), __FILE__, __LINE__); \
    return; } } while (0)

typedef int v8i __attribute__((ext_vector_type(8)));
typedef int v4i __attribute__((ext_vector_type(4)));
typedef int v2i __attribute__((ext_vector_type(2)));
typedef float f4 __attribute__((ext_vector_type(4)));
typedef unsigned u32x4 __attribute__((ext_vector_type(4)));
#define LAS __attribute__((address_space(3)))

constexpr int NR = 8;
constexpr int MLOC = 512;
constexpr int KTOT = 4096, NTOT = 16384;
constexpr int KSH = 512;
constexpr size_t WS_SEQ = 0;
constexpr size_t WS_TMO = 4096;
constexpr size_t WS_CNT = 8192;
constexpr size_t WS_A = 65536;
constexpr size_t A_BYTES = (size_t)MLOC * KTOT;
constexpr size_t WS_NEED = WS_A + 2 * A_BYTES;
constexpr unsigned long long DEADLINE = 40000000ull;

struct Params {
  const unsigned char* x;
  const unsigned char* w;
  const float* sx;
  const float* sw;
  float* out;
  unsigned char* ws[NR];
  int rank;
  int pad;
};

__global__ __launch_bounds__(256, 1) void fused(Params p) {
  extern __shared__ __attribute__((aligned(16))) unsigned char smem[];
  const int tid = threadIdx.x, lane = tid & 63;
  const int wave = __builtin_amdgcn_readfirstlane(tid >> 6);
  const int b = blockIdx.x;
  const int rank = p.rank;
  unsigned char* myws = p.ws[rank];
  unsigned* seq = (unsigned*)(myws + WS_SEQ) + b;
  const unsigned call = __hip_atomic_load(seq, __ATOMIC_RELAXED, __HIP_MEMORY_SCOPE_AGENT) + 1u;
  unsigned* tmo = (unsigned*)(myws + WS_TMO);
  const unsigned par = call & 1u;
  {
    const int j = b >> 5, rb = b & 31;
    if (wave < 2 && j < NR - 1) {
      const unsigned dest = (unsigned)(rank + 1 + j) & 7u;
      const int h = wave;
      const unsigned char* src = p.x + (size_t)(dest * MLOC + rb * 16 + (lane & 15)) * KSH + h * 64 + (lane >> 4) * 16;
      u32x4 v[4];
#pragma unroll
      for (int t = 0; t < 4; ++t) v[t] = *(const u32x4*)(src + t * 128);
      unsigned char* dst = p.ws[dest] + WS_A + par * A_BYTES + ((size_t)(rank * 4) * 32 + rb) * 2048 + h * 1024 + lane * 16;
#pragma unroll
      for (int t = 0; t < 4; ++t) {
        unsigned char* d = dst + (size_t)t * 65536;
        asm volatile("global_store_dwordx4 %0, %1, off sc0 sc1\n\ts_nop 1" ::"v"(d), "v"(v[t]) : "memory");
      }
      asm volatile("s_waitcnt vmcnt(0)" ::: "memory");
      if (lane == 0) __hip_atomic_fetch_add((unsigned*)(p.ws[dest] + WS_CNT + rank * 256), 1u, __ATOMIC_RELAXED, __HIP_MEMORY_SCOPE_SYSTEM);
    }
  }
  if (wave == 0) {
    if (lane < NR && lane != rank) {
      unsigned* c = (unsigned*)(myws + WS_CNT + lane * 256);
      unsigned long long t0 = __builtin_amdgcn_s_memrealtime();
      while (__hip_atomic_load(c, __ATOMIC_RELAXED, __HIP_MEMORY_SCOPE_SYSTEM) < call * 64u) {
        if (__hip_atomic_load(tmo, __ATOMIC_RELAXED, __HIP_MEMORY_SCOPE_AGENT) == call ||
            __builtin_amdgcn_s_memrealtime() - t0 > DEADLINE) {
          __hip_atomic_store(tmo, call, __ATOMIC_RELAXED, __HIP_MEMORY_SCOPE_AGENT);
          break;
        }
        __builtin_amdgcn_s_sleep(2);
      }
    }
    __builtin_amdgcn_fence(__ATOMIC_ACQUIRE, "");
    asm volatile("s_waitcnt vmcnt(0)" ::: "memory");
  }
  __syncthreads();
  const int xcd = b & 7, idx = b >> 3, nt = xcd * 16 + (idx >> 1), mt = idx & 1;
  const int r16 = lane & 15, g = lane >> 4;
  f4 acc[4][8];
#pragma unroll
  for (int i = 0; i < 4; ++i)
#pragma unroll
    for (int j = 0; j < 8; ++j) acc[i][j] = (f4){0, 0, 0, 0};
  const int srow = lane >> 3, sc = lane & 7;
  const int q = (lane & 15) >> 1, pp = lane & 1;
  const int f = ((q >> 1) & 3) | ((g & 1) << 2);
  const unsigned char* Abuf = myws + WS_A + par * A_BYTES;
  for (int ks = 0; ks < 32; ++ks) {
    unsigned char* buf = smem + (ks & 1) * 16384;
#pragma unroll
    for (int i = 0; i < 4; ++i) {
      const int row = wave * 32 + i * 8 + srow;
      const int fr = ((row >> 1) & 3) | (((row >> 4) & 1) << 2);
      const unsigned char* gp = p.w + (size_t)(ks * 128 + row) * NTOT + nt * 128 + ((sc ^ fr) * 16);
      __builtin_amdgcn_global_load_lds((const __attribute__((address_space(1))) void*)gp, (LAS void*)(buf + wave * 4096 + i * 1024), 16, 0, 0);
    }
    v4i a[4][2];
    const int src = ks >> 2;
#pragma unroll
    for (int i = 0; i < 4; ++i)
#pragma unroll
      for (int h = 0; h < 2; ++h) {
        const int rb = mt * 16 + wave * 4 + i;
        if (src == rank)
          a[i][h] = *(const v4i*)(p.x + (size_t)(rank * MLOC + rb * 16 + r16) * KSH + (ks & 3) * 128 + h * 64 + g * 16);
        else
          a[i][h] = *(const v4i*)(Abuf + ((size_t)ks * 32 + rb) * 2048 + h * 1024 + lane * 16);
      }
    asm volatile("s_waitcnt vmcnt(0)" ::: "memory");
    __syncthreads();
#pragma unroll
    for (int j = 0; j < 8; ++j) {
      v8i bf;
#pragma unroll
      for (int r = 0; r < 4; ++r) {
        const int row = 64 * (r >> 1) + 8 * (r & 1) + 16 * g + q;
        v2i t2 = __builtin_amdgcn_ds_read_tr8_b64_v2i32((LAS v2i*)(buf + row * 128 + ((j ^ f) * 16) + 8 * pp));
        bf[2 * r] = t2[0]; bf[2 * r + 1] = t2[1];
      }
#pragma unroll
      for (int i = 0; i < 4; ++i) {
        v8i af;
#pragma unroll
        for (int e = 0; e < 4; ++e) { af[e] = a[i][0][e]; af[4 + e] = a[i][1][e]; }
        acc[i][j] = __builtin_amdgcn_mfma_scale_f32_16x16x128_f8f6f4(bf, af, acc[i][j], 0, 0, 0, 0, 0, 0);
      }
    }
  }
  const float scale = p.sx[0] * p.sw[0];
  const bool bad = __hip_atomic_load(tmo, __ATOMIC_RELAXED, __HIP_MEMORY_SCOPE_AGENT) == call;
  const float qn = __builtin_nanf("");
#pragma unroll
  for (int i = 0; i < 4; ++i)
#pragma unroll
    for (int j = 0; j < 8; ++j) {
      f4 o;
#pragma unroll
      for (int e = 0; e < 4; ++e) { float y = acc[i][j][e] * scale; o[e] = bad ? qn : fmaxf(y, 0.0f); }
      float* op = p.out + (size_t)(mt * 256 + wave * 64 + i * 16 + r16) * NTOT + nt * 128 + j * 16 + g * 4;
      *(f4*)op = o;
    }
  if (tid == 0) __hip_atomic_store(seq, call, __ATOMIC_RELAXED, __HIP_MEMORY_SCOPE_AGENT);
}

extern "C" void kernel_launch(int rank, int nranks, ncclComm_t comm, hipStream_t stream,
                              void* const* d_in, const int* in_sizes, int n_in,
                              void* d_out, int out_size, const int* in_dtypes, int out_dtype,
                              void* const* d_out_all, void* const* d_ws_all, size_t ws_bytes) {
  (void)comm; (void)in_sizes; (void)n_in; (void)in_dtypes; (void)out_dtype; (void)d_out_all;
  if (nranks != NR || (unsigned)rank >= (unsigned)NR || ws_bytes < WS_NEED || out_size != MLOC * NTOT) {
    fprintf(stderr, "unexpected configuration\n");
    return;
  }
  Params p;
  p.x = (const unsigned char*)d_in[0];
  p.w = (const unsigned char*)d_in[1];
  p.sx = (const float*)d_in[2];
  p.sw = (const float*)d_in[3];
  p.out = (float*)d_out;
  for (int i = 0; i < NR; ++i) p.ws[i] = (unsigned char*)d_ws_all[i];
  p.rank = rank;
  p.pad = 0;
  hipLaunchKernelGGL(fused, dim3(256), dim3(256), 32768, stream, p);
  HIP_CHECK(hipGetLastError());
}
```

```cpp
#include <hip/hip_runtime.h>
#include <rccl/rccl.h>
#include <cstdio>
#include <cstdint>

#define HIP_CHECK(expr) do { hipError_t _e = (expr); if (_e != hipSuccess) { \
    fprintf(stderr, "HIP error %s at %s:%d\n", hipGetErrorString(_e), __FILE__, __LINE__); \
    return; } } while (0)

typedef int v4i __attribute__((ext_vector_type(4)));
typedef unsigned u32x4 __attribute__((ext_vector_type(4)));
#define LAS __attribute__((address_space(3)))
#define GAS __attribute__((address_space(1)))

constexpr int NR = 8;
constexpr int MLOC = 512, KTOT = 4096, NTOT = 16384, KSH = 512;
constexpr size_t WS_SEQ = 0;
constexpr size_t WS_TMO = 4096;
constexpr size_t WS_CNT = 8192;
constexpr size_t WS_HELLO = 16384;
constexpr size_t WS_A = 65536;
constexpr size_t A_BYTES = (size_t)MLOC * KTOT;
constexpr size_t WS_NEED = WS_A + 2 * A_BYTES;
constexpr unsigned long long DEADLINE = 30000000ull;
constexpr int C_FULL = 0, C_DONE = 4, C_FINAL = 8, C_ELIG = 9, C_ABORT = 10, C_PDONE = 11, C_BLK = 12, C_AOFF = 16, C_ORD = 64, C_PH = 100;
constexpr int LDS_RING = 1024, LDS_PUSH = LDS_RING + 8 * 16384, LDS_OST2 = LDS_PUSH, LDS_DUMP = LDS_PUSH + 16384, LDS_BYTES = LDS_DUMP + 1024;

struct Params {
  const unsigned char* x;
  const unsigned char* w;
  const float* sx;
  const float* sw;
  float* out;
  unsigned char* ws[NR];
  int rank;
  int pad;
};

__device__ __forceinline__ unsigned rfl(unsigned v) { return __builtin_amdgcn_readfirstlane(v); }
__device__ __forceinline__ unsigned long long rt() { return __builtin_amdgcn_s_memrealtime(); }
__device__ __forceinline__ v4i make_rsrc(const void* p, unsigned bytes) {
  const unsigned long long a = (unsigned long long)p;
  v4i r; r[0] = rfl((unsigned)a); r[1] = rfl((unsigned)(a >> 32)) & 0xffff; r[2] = bytes; r[3] = 0x20000;
  return r;
}

__device__ __forceinline__ unsigned idx_ks(unsigned idx, int rank) {
  const unsigned t = idx / 7u, j = idx - t * 7u;
  return ((((unsigned)rank + 1u + j) & 7u) << 2) + t;
}

extern "C" __global__ __launch_bounds__(512, 2) void fused(Params p) {
  extern __shared__ __attribute__((aligned(16))) unsigned char smem[];
  const int tid = threadIdx.x, lane = tid & 63;
  const int wave = rfl(tid >> 6);
  const int b = blockIdx.x;
  const int rank = p.rank;
  const int xcd = b & 7, idx = b >> 3, nt = xcd * 16 + (idx >> 1), mt = idx & 1;
  volatile LAS unsigned* ctl = (volatile LAS unsigned*)(LAS unsigned char*)smem;
  auto storer = [&](int hw) {
    __builtin_amdgcn_s_setprio(3);
    const float scale = p.sx[0] * p.sw[0];
    float* orow = p.out + (size_t)(mt * 256 + hw * 64 + (lane >> 3)) * NTOT + nt * 128 + (lane & 7) * 4;
    {
      unsigned spins = 0;
      while (rfl(ctl[C_DONE + hw]) < 4u * 27u && rfl(ctl[C_ABORT]) == 0u && ++spins < 0x400000u) __builtin_amdgcn_s_sleep(16);
    }
    for (int ph = 0; ph < 3; ++ph) {
      unsigned spins = 0;
      for (;;) {
        const unsigned f = ctl[C_PH + ph * 4 + hw], ab = ctl[C_ABORT];
        if (rfl(ab) != 0u) return;
        if (rfl(f) != 0u) break;
        if (ph == 0) __builtin_amdgcn_s_sleep(1);
        if (++spins > 0x4000000u) return;
      }
      const unsigned base = ph < 2 ? LDS_RING + ph * 32768 + hw * 8192 : (hw < 2 ? LDS_RING + 65536 + hw * 8192 : LDS_OST2 + (hw - 2) * 8192);
      typedef float f4v __attribute__((ext_vector_type(4)));
      f4v v[8];
#pragma unroll
      for (int u = 0; u < 8; ++u) {
        const unsigned row = u * 8 + (lane >> 3);
        v[u] = *(volatile LAS f4v*)((LAS unsigned char*)smem + base + row * 128 + (((lane & 7) ^ (row & 7)) << 4));
      }
#pragma unroll
      for (int u = 0; u < 8; ++u) {
        f4v y;
#pragma unroll
        for (int e2 = 0; e2 < 4; ++e2) y[e2] = fmaxf(v[u][e2] * scale, 0.0f);
        const float* dst = orow + (size_t)u * 8 * NTOT + ph * 32;
        asm volatile("global_store_dwordx4 %0, %1, off sc0 sc1\n\ts_nop 1" :: "v"(dst), "v"(y) : "memory");
      }
    }
  };
  if (wave < 4) {
    const unsigned char* px = p.x + (size_t)rank * MLOC * KSH;
    const unsigned char* pa = p.ws[rank] + WS_A;
    const unsigned xoff = (unsigned)(mt * 256 + wave * 64) * KSH;
    const unsigned awoff = (unsigned)(mt * 16 + wave * 4) * 2048;
    const unsigned ooff = (unsigned)(mt * 256 + wave * 64) * (NTOT * 4) + nt * 512;
    const unsigned wave4 = wave * 4;
    const unsigned stg = LDS_RING + 4 * 16384 + wave * 8192;
    const unsigned ost0 = LDS_RING + wave * 8192;
    const unsigned ost2 = wave < 2 ? LDS_RING + 65536 + wave * 8192 : LDS_OST2 + (wave - 2) * 8192;
    const unsigned phw = (C_PH + wave) * 4;
    const unsigned ntl_c = nt & 15;
    asm volatile(
      "v_mbcnt_lo_u32_b32 v126, -1, 0\n"
      "v_mbcnt_hi_u32_b32 v126, -1, v126\n"
      "s_mov_b64 s[64:65], %[px]\n"
      "s_and_b32 s65, s65, 0xffff\n"
      "s_mov_b32 s66, 0x40000\n"
      "s_mov_b32 s67, 0x20000\n"
      "v_lshrrev_b32 v127, 4, v126\n"
      "v_and_b32 v122, 7, v126\n"
      "v_xor_b32 v122, v122, v127\n"
      "v_lshlrev_b32 v122, 4, v122\n"
      "v_lshrrev_b32 v123, 3, v126\n"
      "v_lshl_add_u32 v122, v123, 9, v122\n"
      "v_xor_b32 v123, 64, v122\n"
      "s_barrier\n"
      "s_add_u32 m0, %[stg], 0\n"
      "s_add_u32 s79, %[xoff], 0\n"
      "buffer_load_dwordx4 v122, s[64:67], s79 offen lds\n"
      "s_add_u32 m0, m0, 1024\n"
      "s_add_u32 s79, s79, 4096\n"
      "buffer_load_dwordx4 v123, s[64:67], s79 offen lds\n"
      "s_add_u32 m0, m0, 1024\n"
      "s_add_u32 s79, s79, 4096\n"
      "buffer_load_dwordx4 v122, s[64:67], s79 offen lds\n"
      "s_add_u32 m0, m0, 1024\n"
      "s_add_u32 s79, s79, 4096\n"
      "buffer_load_dwordx4 v123, s[64:67], s79 offen lds\n"
      "s_add_u32 m0, m0, 1024\n"
      "s_add_u32 s79, s79, 4096\n"
      "buffer_load_dwordx4 v122, s[64:67], s79 offen lds\n"
      "s_add_u32 m0, m0, 1024\n"
      "s_add_u32 s79, s79, 4096\n"
      "buffer_load_dwordx4 v123, s[64:67], s79 offen lds\n"
      "s_add_u32 m0, m0, 1024\n"
      "s_add_u32 s79, s79, 4096\n"
      "buffer_load_dwordx4 v122, s[64:67], s79 offen lds\n"
      "s_add_u32 m0, m0, 1024\n"
      "s_add_u32 s79, s79, 4096\n"
      "buffer_load_dwordx4 v123, s[64:67], s79 offen lds\n"
      "s_load_dword s80, %[psx], 0\n"
      "s_load_dword s81, %[psw], 0\n"
      "s_mov_b64 s[68:69], %[pa]\n"
      "s_and_b32 s69, s69, 0xffff\n"
      "s_mov_b32 s70, 0x400000\n"
      "s_mov_b32 s71, 0x20000\n"
      "s_mov_b64 s[72:73], %[po]\n"
      "s_and_b32 s73, s73, 0xffff\n"
      "s_mov_b32 s74, 0x2000000\n"
      "s_mov_b32 s75, 0x20000\n"
      "v_and_b32 v120, 15, v126\n"
      "v_lshrrev_b32 v121, 1, v120\n"
      "v_xor_b32 v121, v121, v127\n"
      "v_lshlrev_b32 v120, 7, v120\n"
      "v_lshl_add_u32 v120, v121, 4, v120\n"
      "v_add_u32 v120, %[stg], v120\n"
      "v_bfe_u32 v124, v126, 1, 3\n"
      "v_and_b32 v125, 1, v126\n"
      "v_lshl_add_u32 v121, v127, 4, v124\n"
      "v_lshlrev_b32 v121, 7, v121\n"
      "v_lshl_add_u32 v121, v125, 3, v121\n"
      "v_lshrrev_b32 v124, 1, v124\n"
      "v_and_b32 v127, 1, v127\n"
      "v_lshl_or_b32 v124, v127, 2, v124\n"
      "v_lshl_add_u32 v121, v124, 4, v121\n"
      "v_mov_b32 v124, %[wave4]\n"
      "v_mov_b32 v125, 0\n"
      "v_mov_b32 v0, 0\n"
      "v_mov_b32 v1, 0\n"
      "v_mov_b32 v2, 0\n"
      "v_mov_b32 v3, 0\n"
      "v_mov_b32 v4, 0\n"
      "v_mov_b32 v5, 0\n"
      "v_mov_b32 v6, 0\n"
      "v_mov_b32 v7, 0\n"
      "v_mov_b32 v8, 0\n"
      "v_mov_b32 v9, 0\n"
      "v_mov_b32 v10, 0\n"
      "v_mov_b32 v11, 0\n"
      "v_mov_b32 v12, 0\n"
      "v_mov_b32 v13, 0\n"
      "v_mov_b32 v14, 0\n"
      "v_mov_b32 v15, 0\n"
      "v_mov_b32 v16, 0\n"
      "v_mov_b32 v17, 0\n"
      "v_mov_b32 v18, 0\n"
      "v_mov_b32 v19, 0\n"
      "v_mov_b32 v20, 0\n"
      "v_mov_b32 v21, 0\n"
      "v_mov_b32 v22, 0\n"
      "v_mov_b32 v23, 0\n"
      "v_mov_b32 v24, 0\n"
      "v_mov_b32 v25, 0\n"
      "v_mov_b32 v26, 0\n"
      "v_mov_b32 v27, 0\n"
      "v_mov_b32 v28, 0\n"
      "v_mov_b32 v29, 0\n"
      "v_mov_b32 v30, 0\n"
      "v_mov_b32 v31, 0\n"
      "v_mov_b32 v32, 0\n"
      "v_mov_b32 v33, 0\n"
      "v_mov_b32 v34, 0\n"
      "v_mov_b32 v35, 0\n"
      "v_mov_b32 v36, 0\n"
      "v_mov_b32 v37, 0\n"
      "v_mov_b32 v38, 0\n"
      "v_mov_b32 v39, 0\n"
      "v_mov_b32 v40, 0\n"
      "v_mov_b32 v41, 0\n"
      "v_mov_b32 v42, 0\n"
      "v_mov_b32 v43, 0\n"
      "v_mov_b32 v44, 0\n"
      "v_mov_b32 v45, 0\n"
      "v_mov_b32 v46, 0\n"
      "v_mov_b32 v47, 0\n"
      "v_mov_b32 v48, 0\n"
      "v_mov_b32 v49, 0\n"
      "v_mov_b32 v50, 0\n"
      "v_mov_b32 v51, 0\n"
      "v_mov_b32 v52, 0\n"
      "v_mov_b32 v53, 0\n"
      "v_mov_b32 v54, 0\n"
      "v_mov_b32 v55, 0\n"
      "v_mov_b32 v56, 0\n"
      "v_mov_b32 v57, 0\n"
      "v_mov_b32 v58, 0\n"
      "v_mov_b32 v59, 0\n"
      "v_mov_b32 v60, 0\n"
      "v_mov_b32 v61, 0\n"
      "v_mov_b32 v62, 0\n"
      "v_mov_b32 v63, 0\n"
      "v_mov_b32 v64, 0\n"
      "v_mov_b32 v65, 0\n"
      "v_mov_b32 v66, 0\n"
      "v_mov_b32 v67, 0\n"
      "v_mov_b32 v68, 0\n"
      "v_mov_b32 v69, 0\n"
      "v_mov_b32 v70, 0\n"
      "v_mov_b32 v71, 0\n"
      "v_mov_b32 v72, 0\n"
      "v_mov_b32 v73, 0\n"
      "v_mov_b32 v74, 0\n"
      "v_mov_b32 v75, 0\n"
      "v_mov_b32 v76, 0\n"
      "v_mov_b32 v77, 0\n"
      "v_mov_b32 v78, 0\n"
      "v_mov_b32 v79, 0\n"
      "v_mov_b32 v80, 0\n"
      "v_mov_b32 v81, 0\n"
      "v_mov_b32 v82, 0\n"
      "v_mov_b32 v83, 0\n"
      "v_mov_b32 v84, 0\n"
      "v_mov_b32 v85, 0\n"
      "v_mov_b32 v86, 0\n"
      "v_mov_b32 v87, 0\n"
      "v_mov_b32 v88, 0\n"
      "v_mov_b32 v89, 0\n"
      "v_mov_b32 v90, 0\n"
      "v_mov_b32 v91, 0\n"
      "v_mov_b32 v92, 0\n"
      "v_mov_b32 v93, 0\n"
      "v_mov_b32 v94, 0\n"
      "v_mov_b32 v95, 0\n"
      "v_mov_b32 v96, 0\n"
      "v_mov_b32 v97, 0\n"
      "v_mov_b32 v98, 0\n"
      "v_mov_b32 v99, 0\n"
      "v_mov_b32 v100, 0\n"
      "v_mov_b32 v101, 0\n"
      "v_mov_b32 v102, 0\n"
      "v_mov_b32 v103, 0\n"
      "v_mov_b32 v104, 0\n"
      "v_mov_b32 v105, 0\n"
      "v_mov_b32 v106, 0\n"
      "v_mov_b32 v107, 0\n"
      "v_mov_b32 v108, 0\n"
      "v_mov_b32 v109, 0\n"
      "v_mov_b32 v110, 0\n"
      "v_mov_b32 v111, 0\n"
      "v_mov_b32 v112, 0\n"
      "v_mov_b32 v113, 0\n"
      "v_mov_b32 v114, 0\n"
      "v_mov_b32 v115, 0\n"
      "v_mov_b32 v116, 0\n"
      "v_mov_b32 v117, 0\n"
      "v_mov_b32 v118, 0\n"
      "v_mov_b32 v119, 0\n"
      "v_accvgpr_write_b32 a0, 0\n"
      "v_accvgpr_write_b32 a1, 0\n"
      "v_accvgpr_write_b32 a2, 0\n"
      "v_accvgpr_write_b32 a3, 0\n"
      "v_accvgpr_write_b32 a4, 0\n"
      "v_accvgpr_write_b32 a5, 0\n"
      "v_accvgpr_write_b32 a6, 0\n"
      "v_accvgpr_write_b32 a7, 0\n"
      "s_mov_b32 s87, 0\n"
      "s_barrier\n"
      "s_add_u32 m0, %[stg], 32768\n"
      "s_add_u32 s79, %[xoff], 128\n"
      "buffer_load_dwordx4 v122, s[64:67], s79 offen lds\n"
      "s_add_u32 m0, m0, 1024\n"
      "s_add_u32 s79, s79, 4096\n"
      "buffer_load_dwordx4 v123, s[64:67], s79 offen lds\n"
      "s_add_u32 m0, m0, 1024\n"
      "s_add_u32 s79, s79, 4096\n"
      "buffer_load_dwordx4 v122, s[64:67], s79 offen lds\n"
      "s_add_u32 m0, m0, 1024\n"
      "s_add_u32 s79, s79, 4096\n"
      "buffer_load_dwordx4 v123, s[64:67], s79 offen lds\n"
      "s_add_u32 m0, m0, 1024\n"
      "s_add_u32 s79, s79, 4096\n"
      "buffer_load_dwordx4 v122, s[64:67], s79 offen lds\n"
      "s_add_u32 m0, m0, 1024\n"
      "s_add_u32 s79, s79, 4096\n"
      "buffer_load_dwordx4 v123, s[64:67], s79 offen lds\n"
      "s_add_u32 m0, m0, 1024\n"
      "s_add_u32 s79, s79, 4096\n"
      "buffer_load_dwordx4 v122, s[64:67], s79 offen lds\n"
      "s_add_u32 m0, m0, 1024\n"
      "s_add_u32 s79, s79, 4096\n"
      "buffer_load_dwordx4 v123, s[64:67], s79 offen lds\n"
      "v_mov_b32 v127, v122\n"
      "s_waitcnt vmcnt(8)\n"
      "ds_read_b128 a[40:43], v120 offset:0\n"
      "ds_read_b128 a[48:51], v120 offset:2048\n"
      "ds_read_b128 a[56:59], v120 offset:4096\n"
      "ds_read_b128 a[64:67], v120 offset:6144\n"
      "v_xor_b32 v120, 64, v120\n"
      "ds_read_b128 a[44:47], v120 offset:0\n"
      "ds_read_b128 a[52:55], v120 offset:2048\n"
      "ds_read_b128 a[60:63], v120 offset:4096\n"
      "ds_read_b128 a[68:71], v120 offset:6144\n"
      "v_xor_b32 v120, 64, v120\n"
      "s_waitcnt lgkmcnt(0)\n"
      "s_mov_b32 s85, 1\n"
      "s_mov_b32 s86, 0\n"
      "L_w0_%=:\n"
      "ds_read_b32 v126, v124 offset:0\n"
      "s_waitcnt lgkmcnt(0)\n"
      "v_readfirstlane_b32 s84, v126\n"
      "s_cmp_ge_u32 s84, s85\n"
      "s_cbranch_scc1 L_w0d_%=\n"
      "s_sleep 1\n"
      "s_add_u32 s86, s86, 1\n"
      "s_cmp_lt_u32 s86, 0x800000\n"
      "s_cbranch_scc1 L_w0_%=\n"
      "s_mov_b32 s87, 1\n"
      "L_w0d_%=:\n"
      "s_mov_b32 s85, 2\n"
      "ds_read_b64_tr_b8 a[104:105], v121 offset:1024\n"
      "ds_read_b64_tr_b8 a[106:107], v121 offset:2048\n"
      "ds_read_b64_tr_b8 a[108:109], v121 offset:9216\n"
      "ds_read_b64_tr_b8 a[110:111], v121 offset:10240\n"
      "v_xor_b32 v123, 16, v121\n"
      "ds_read_b64_tr_b8 a[112:113], v123 offset:1024\n"
      "ds_read_b64_tr_b8 a[114:115], v123 offset:2048\n"
      "ds_read_b64_tr_b8 a[116:117], v123 offset:9216\n"
      "ds_read_b64_tr_b8 a[118:119], v123 offset:10240\n"
      "s_mov_b32 s62, 0\n"
      "s_waitcnt lgkmcnt(4)\n"
      "v_mfma_f32_16x16x128_f8f6f4 v[0:3], a[104:111], a[40:47], v[0:3]\n"
      "v_xor_b32 v122, 32, v121\n"
      "ds_read_b64_tr_b8 a[120:121], v122 offset:1024\n"
      "ds_read_b64_tr_b8 a[122:123], v122 offset:2048\n"
      "s_add_u32 m0, %[stg], 0\n"
      "s_add_u32 s79, %[xoff], 256\n"
      "buffer_load_dwordx4 v127, s[64:67], s79 offen lds\n"
      "v_xor_b32 v127, 64, v127\n"
      "v_mfma_f32_16x16x128_f8f6f4 v[32:35], a[104:111], a[48:55], v[32:35]\n"
      "ds_read_b64_tr_b8 a[124:125], v122 offset:9216\n"
      "ds_read_b64_tr_b8 a[126:127], v122 offset:10240\n"
      "s_add_u32 m0, m0, 1024\n"
      "s_add_u32 s79, s79, 4096\n"
      "buffer_load_dwordx4 v127, s[64:67], s79 offen lds\n"
      "v_mfma_f32_16x16x128_f8f6f4 v[64:67], a[104:111], a[56:63], v[64:67]\n"
      "v_xor_b32 v127, 64, v127\n"
      "s_add_u32 m0, m0, 1024\n"
      "s_add_u32 s79, s79, 4096\n"
      "v_mfma_f32_16x16x128_f8f6f4 v[96:99], a[104:111], a[64:71], v[96:99]\n"
      "buffer_load_dwordx4 v127, s[64:67], s79 offen lds\n"
      "v_xor_b32 v127, 64, v127\n"
      "s_add_u32 m0, m0, 1024\n"
      "s_waitcnt lgkmcnt(4)\n"
      "v_mfma_f32_16x16x128_f8f6f4 v[4:7], a[112:119], a[40:47], v[4:7]\n"
      "v_xor_b32 v123, 48, v121\n"
      "ds_read_b64_tr_b8 a[104:105], v123 offset:1024\n"
      "ds_read_b64_tr_b8 a[106:107], v123 offset:2048\n"
      "s_add_u32 s79, s79, 4096\n"
      "buffer_load_dwordx4 v127, s[64:67], s79 offen lds\n"
      "v_xor_b32 v127, 64, v127\n"
      "v_mfma_f32_16x16x128_f8f6f4 v[36:39], a[112:119], a[48:55], v[36:39]\n"
      "ds_read_b64_tr_b8 a[108:109], v123 offset:9216\n"
      "ds_read_b64_tr_b8 a[110:111], v123 offset:10240\n"
      "s_add_u32 m0, m0, 1024\n"
      "s_add_u32 s79, s79, 4096\n"
      "buffer_load_dwordx4 v127, s[64:67], s79 offen lds\n"
      "v_xor_b32 v127, 64, v127\n"
      "v_mfma_f32_16x16x128_f8f6f4 v[68:71], a[112:119], a[56:63], v[68:71]\n"
      "s_add_u32 m0, m0, 1024\n"
      "s_add_u32 s79, s79, 4096\n"
      "buffer_load_dwordx4 v127, s[64:67], s79 offen lds\n"
      "v_mfma_f32_16x16x128_f8f6f4 v[100:103], a[112:119], a[64:71], v[100:103]\n"
      "v_xor_b32 v127, 64, v127\n"
      "s_add_u32 m0, m0, 1024\n"
      "s_add_u32 s79, s79, 4096\n"
      "s_waitcnt lgkmcnt(4)\n"
      "v_mfma_f32_16x16x128_f8f6f4 v[8:11], a[120:127], a[40:47], v[8:11]\n"
      "v_xor_b32 v122, 64, v121\n"
      "ds_read_b64_tr_b8 a[112:113], v122 offset:1024\n"
      "ds_read_b64_tr_b8 a[114:115], v122 offset:2048\n"
      "buffer_load_dwordx4 v127, s[64:67], s79 offen lds\n"
      "v_xor_b32 v127, 64, v127\n"
      "s_add_u32 m0, m0, 1024\n"
      "v_mfma_f32_16x16x128_f8f6f4 v[40:43], a[120:127], a[48:55], v[40:43]\n"
      "ds_read_b64_tr_b8 a[116:117], v122 offset:9216\n"
      "ds_read_b64_tr_b8 a[118:119], v122 offset:10240\n"
      "s_add_u32 s79, s79, 4096\n"
      "buffer_load_dwordx4 v127, s[64:67], s79 offen lds\n"
      "v_xor_b32 v127, 64, v127\n"
      "v_mfma_f32_16x16x128_f8f6f4 v[72:75], a[120:127], a[56:63], v[72:75]\n"
      "v_mfma_f32_16x16x128_f8f6f4 v[104:107], a[120:127], a[64:71], v[104:107]\n"
      "s_waitcnt lgkmcnt(4)\n"
      "v_mfma_f32_16x16x128_f8f6f4 v[12:15], a[104:111], a[40:47], v[12:15]\n"
      "v_xor_b32 v123, 80, v121\n"
      "ds_read_b64_tr_b8 a[120:121], v123 offset:1024\n"
      "ds_read_b64_tr_b8 a[122:123], v123 offset:2048\n"
      "v_mfma_f32_16x16x128_f8f6f4 v[44:47], a[104:111], a[48:55], v[44:47]\n"
      "ds_read_b64_tr_b8 a[124:125], v123 offset:9216\n"
      "ds_read_b64_tr_b8 a[126:127], v123 offset:10240\n"
      "v_mfma_f32_16x16x128_f8f6f4 v[76:79], a[104:111], a[56:63], v[76:79]\n"
      "v_mfma_f32_16x16x128_f8f6f4 v[108:111], a[104:111], a[64:71], v[108:111]\n"
      "s_waitcnt vmcnt(8)\n"
      "s_waitcnt lgkmcnt(4)\n"
      "v_mfma_f32_16x16x128_f8f6f4 v[16:19], a[112:119], a[40:47], v[16:19]\n"
      "ds_read_b32 v126, v124 offset:0\n"
      "v_xor_b32 v122, 96, v121\n"
      "ds_read_b64_tr_b8 a[104:105], v122 offset:1024\n"
      "ds_read_b64_tr_b8 a[106:107], v122 offset:2048\n"
      "ds_read_b128 a[72:75], v120 offset:32768\n"
      "ds_read_b128 a[80:83], v120 offset:34816\n"
      "v_mfma_f32_16x16x128_f8f6f4 v[48:51], a[112:119], a[48:55], v[48:51]\n"
      "ds_read_b64_tr_b8 a[108:109], v122 offset:9216\n"
      "ds_read_b64_tr_b8 a[110:111], v122 offset:10240\n"
      "ds_read_b128 a[88:91], v120 offset:36864\n"
      "ds_read_b128 a[96:99], v120 offset:38912\n"
      "v_mfma_f32_16x16x128_f8f6f4 v[80:83], a[112:119], a[56:63], v[80:83]\n"
      "v_xor_b32 v120, 64, v120\n"
      "v_mfma_f32_16x16x128_f8f6f4 v[112:115], a[112:119], a[64:71], v[112:115]\n"
      "ds_read_b128 a[76:79], v120 offset:32768\n"
      "ds_read_b128 a[84:87], v120 offset:34816\n"
      "s_waitcnt lgkmcnt(11)\n"
      "v_mfma_f32_16x16x128_f8f6f4 v[20:23], a[120:127], a[40:47], v[20:23]\n"
      "v_xor_b32 v123, 112, v121\n"
      "ds_read_b64_tr_b8 a[112:113], v123 offset:1024\n"
      "ds_read_b64_tr_b8 a[114:115], v123 offset:2048\n"
      "ds_read_b128 a[92:95], v120 offset:36864\n"
      "ds_read_b128 a[100:103], v120 offset:38912\n"
      "v_mfma_f32_16x16x128_f8f6f4 v[52:55], a[120:127], a[48:55], v[52:55]\n"
      "ds_read_b64_tr_b8 a[116:117], v123 offset:9216\n"
      "ds_read_b64_tr_b8 a[118:119], v123 offset:10240\n"
      "v_xor_b32 v120, 64, v120\n"
      "v_mfma_f32_16x16x128_f8f6f4 v[84:87], a[120:127], a[56:63], v[84:87]\n"
      "v_add_u32 v125, 4, v125\n"
      "v_mfma_f32_16x16x128_f8f6f4 v[116:119], a[120:127], a[64:71], v[116:119]\n"
      "s_waitcnt lgkmcnt(2)\n"
      "v_readfirstlane_b32 s84, v126\n"
      "v_add_u32 v121, 0x4000, v121\n"
      "s_cmp_lt_u32 s84, s85\n"
      "s_cbranch_scc1 L_slowFp0_%=\n"
      "L_backFp0_%=:\n"
      "v_and_b32 v121, 0x1ffff, v121\n"
      "v_mfma_f32_16x16x128_f8f6f4 v[24:27], a[104:111], a[40:47], v[24:27]\n"
      "ds_read_b64_tr_b8 a[120:121], v121 offset:1024\n"
      "ds_read_b64_tr_b8 a[122:123], v121 offset:2048\n"
      "s_add_u32 m0, %[stg], 32768\n"
      "s_add_u32 s79, %[xoff], 384\n"
      "buffer_load_dwordx4 v127, s[64:67], s79 offen lds\n"
      "v_xor_b32 v127, 64, v127\n"
      "v_mfma_f32_16x16x128_f8f6f4 v[56:59], a[104:111], a[48:55], v[56:59]\n"
      "ds_read_b64_tr_b8 a[124:125], v121 offset:9216\n"
      "ds_read_b64_tr_b8 a[126:127], v121 offset:10240\n"
      "s_add_u32 m0, m0, 1024\n"
      "s_add_u32 s79, s79, 4096\n"
      "buffer_load_dwordx4 v127, s[64:67], s79 offen lds\n"
      "v_xor_b32 v127, 64, v127\n"
      "v_mfma_f32_16x16x128_f8f6f4 v[88:91], a[104:111], a[56:63], v[88:91]\n"
      "s_add_u32 m0, m0, 1024\n"
      "s_add_u32 s79, s79, 4096\n"
      "buffer_load_dwordx4 v127, s[64:67], s79 offen lds\n"
      "v_xor_b32 v127, 64, v127\n"
      "v_mfma_f32_16x16x128_f8f6f4 a[0:3], a[104:111], a[64:71], a[0:3]\n"
      "s_add_u32 m0, m0, 1024\n"
      "s_add_u32 s79, s79, 4096\n"
      "buffer_load_dwordx4 v127, s[64:67], s79 offen lds\n"
      "v_xor_b32 v127, 64, v127\n"
      "s_waitcnt lgkmcnt(4)\n"
      "v_mfma_f32_16x16x128_f8f6f4 v[28:31], a[112:119], a[40:47], v[28:31]\n"
      "v_xor_b32 v123, 16, v121\n"
      "ds_read_b64_tr_b8 a[104:105], v123 offset:1024\n"
      "ds_read_b64_tr_b8 a[106:107], v123 offset:2048\n"
      "s_add_u32 m0, m0, 1024\n"
      "s_add_u32 s79, s79, 4096\n"
      "buffer_load_dwordx4 v127, s[64:67], s79 offen lds\n"
      "v_xor_b32 v127, 64, v127\n"
      "v_mfma_f32_16x16x128_f8f6f4 v[60:63], a[112:119], a[48:55], v[60:63]\n"
      "ds_read_b64_tr_b8 a[108:109], v123 offset:9216\n"
      "ds_read_b64_tr_b8 a[110:111], v123 offset:10240\n"
      "s_add_u32 m0, m0, 1024\n"
      "s_add_u32 s79, s79, 4096\n"
      "buffer_load_dwordx4 v127, s[64:67], s79 offen lds\n"
      "v_xor_b32 v127, 64, v127\n"
      "v_mfma_f32_16x16x128_f8f6f4 v[92:95], a[112:119], a[56:63], v[92:95]\n"
      "s_add_u32 m0, m0, 1024\n"
      "s_add_u32 s79, s79, 4096\n"
      "buffer_load_dwordx4 v127, s[64:67], s79 offen lds\n"
      "v_xor_b32 v127, 64, v127\n"
      "v_mfma_f32_16x16x128_f8f6f4 a[4:7], a[112:119], a[64:71], a[4:7]\n"
      "s_add_u32 s85, s85, 1\n"
      "s_add_u32 m0, m0, 1024\n"
      "s_add_u32 s79, s79, 4096\n"
      "buffer_load_dwordx4 v127, s[64:67], s79 offen lds\n"
      "v_xor_b32 v127, 64, v127\n"
      "s_waitcnt vmcnt(8)\n"
      "s_waitcnt lgkmcnt(4)\n"
      "v_mfma_f32_16x16x128_f8f6f4 v[0:3], a[120:127], a[72:79], v[0:3]\n"
      "v_xor_b32 v122, 32, v121\n"
      "ds_read_b64_tr_b8 a[112:113], v122 offset:1024\n"
      "ds_read_b64_tr_b8 a[114:115], v122 offset:2048\n"
      "ds_read_b128 a[8:11], v120 offset:0\n"
      "ds_read_b128 a[16:19], v120 offset:2048\n"
      "v_mfma_f32_16x16x128_f8f6f4 v[32:35], a[120:127], a[80:87], v[32:35]\n"
      "ds_read_b64_tr_b8 a[116:117], v122 offset:9216\n"
      "ds_read_b64_tr_b8 a[118:119], v122 offset:10240\n"
      "ds_read_b128 a[24:27], v120 offset:4096\n"
      "v_mfma_f32_16x16x128_f8f6f4 v[64:67], a[120:127], a[88:95], v[64:67]\n"
      "ds_read_b128 a[32:35], v120 offset:6144\n"
      "v_xor_b32 v120, 64, v120\n"
      "v_mfma_f32_16x16x128_f8f6f4 v[96:99], a[120:127], a[96:103], v[96:99]\n"
      "ds_read_b128 a[12:15], v120 offset:0\n"
      "s_waitcnt lgkmcnt(9)\n"
      "v_mfma_f32_16x16x128_f8f6f4 v[4:7], a[104:111], a[72:79], v[4:7]\n"
      "v_xor_b32 v123, 48, v121\n"
      "ds_read_b64_tr_b8 a[120:121], v123 offset:1024\n"
      "ds_read_b64_tr_b8 a[122:123], v123 offset:2048\n"
      "ds_read_b128 a[20:23], v120 offset:2048\n"
      "ds_read_b128 a[28:31], v120 offset:4096\n"
      "v_mfma_f32_16x16x128_f8f6f4 v[36:39], a[104:111], a[80:87], v[36:39]\n"
      "ds_read_b64_tr_b8 a[124:125], v123 offset:9216\n"
      "ds_read_b64_tr_b8 a[126:127], v123 offset:10240\n"
      "ds_read_b128 a[36:39], v120 offset:6144\n"
      "v_mfma_f32_16x16x128_f8f6f4 v[68:71], a[104:111], a[88:95], v[68:71]\n"
      "v_xor_b32 v120, 64, v120\n"
      "v_mfma_f32_16x16x128_f8f6f4 v[100:103], a[104:111], a[96:103], v[100:103]\n"
      "s_waitcnt lgkmcnt(10)\n"
      "v_mfma_f32_16x16x128_f8f6f4 v[8:11], a[112:119], a[72:79], v[8:11]\n"
      "v_xor_b32 v122, 64, v121\n"
      "ds_read_b64_tr_b8 a[104:105], v122 offset:1024\n"
      "ds_read_b64_tr_b8 a[106:107], v122 offset:2048\n"
      "v_mfma_f32_16x16x128_f8f6f4 v[40:43], a[112:119], a[80:87], v[40:43]\n"
      "ds_read_b64_tr_b8 a[108:109], v122 offset:9216\n"
      "ds_read_b64_tr_b8 a[110:111], v122 offset:10240\n"
      "v_mfma_f32_16x16x128_f8f6f4 v[72:75], a[112:119], a[88:95], v[72:75]\n"
      "v_mfma_f32_16x16x128_f8f6f4 v[104:107], a[112:119], a[96:103], v[104:107]\n"
      "ds_write_b32 v124, v125 offset:16\n"
      "s_waitcnt lgkmcnt(6)\n"
      "v_mfma_f32_16x16x128_f8f6f4 v[12:15], a[120:127], a[72:79], v[12:15]\n"
      "v_xor_b32 v123, 80, v121\n"
      "ds_read_b64_tr_b8 a[112:113], v123 offset:1024\n"
      "ds_read_b64_tr_b8 a[114:115], v123 offset:2048\n"
      "v_mfma_f32_16x16x128_f8f6f4 v[44:47], a[120:127], a[80:87], v[44:47]\n"
      "ds_read_b64_tr_b8 a[116:117], v123 offset:9216\n"
      "ds_read_b64_tr_b8 a[118:119], v123 offset:10240\n"
      "v_mfma_f32_16x16x128_f8f6f4 v[76:79], a[120:127], a[88:95], v[76:79]\n"
      "v_mfma_f32_16x16x128_f8f6f4 v[108:111], a[120:127], a[96:103], v[108:111]\n"
      "s_waitcnt vmcnt(0)\n"
      "s_waitcnt lgkmcnt(5)\n"
      "v_mfma_f32_16x16x128_f8f6f4 v[16:19], a[104:111], a[72:79], v[16:19]\n"
      "ds_read_b32 v126, v124 offset:0\n"
      "ds_read_b32 v127, v125 offset:76\n"
      "v_xor_b32 v122, 96, v121\n"
      "ds_read_b64_tr_b8 a[120:121], v122 offset:1024\n"
      "ds_read_b64_tr_b8 a[122:123], v122 offset:2048\n"
      "ds_read_b128 a[40:43], v120 offset:32768\n"
      "ds_read_b128 a[48:51], v120 offset:34816\n"
      "v_mfma_f32_16x16x128_f8f6f4 v[48:51], a[104:111], a[80:87], v[48:51]\n"
      "ds_read_b64_tr_b8 a[124:125], v122 offset:9216\n"
      "ds_read_b64_tr_b8 a[126:127], v122 offset:10240\n"
      "ds_read_b128 a[56:59], v120 offset:36864\n"
      "ds_read_b128 a[64:67], v120 offset:38912\n"
      "v_mfma_f32_16x16x128_f8f6f4 v[80:83], a[104:111], a[88:95], v[80:83]\n"
      "v_xor_b32 v120, 64, v120\n"
      "v_mfma_f32_16x16x128_f8f6f4 v[112:115], a[104:111], a[96:103], v[112:115]\n"
      "ds_read_b128 a[44:47], v120 offset:32768\n"
      "ds_read_b128 a[52:55], v120 offset:34816\n"
      "s_waitcnt lgkmcnt(12)\n"
      "v_mfma_f32_16x16x128_f8f6f4 v[20:23], a[112:119], a[72:79], v[20:23]\n"
      "v_xor_b32 v123, 112, v121\n"
      "ds_read_b64_tr_b8 a[104:105], v123 offset:1024\n"
      "ds_read_b64_tr_b8 a[106:107], v123 offset:2048\n"
      "ds_read_b128 a[60:63], v120 offset:36864\n"
      "ds_read_b128 a[68:71], v120 offset:38912\n"
      "v_mfma_f32_16x16x128_f8f6f4 v[52:55], a[112:119], a[80:87], v[52:55]\n"
      "ds_read_b64_tr_b8 a[108:109], v123 offset:9216\n"
      "ds_read_b64_tr_b8 a[110:111], v123 offset:10240\n"
      "v_xor_b32 v120, 64, v120\n"
      "v_mfma_f32_16x16x128_f8f6f4 v[84:87], a[112:119], a[88:95], v[84:87]\n"
      "v_add_u32 v125, 4, v125\n"
      "v_mfma_f32_16x16x128_f8f6f4 v[116:119], a[112:119], a[96:103], v[116:119]\n"
      "s_waitcnt lgkmcnt(10)\n"
      "v_readfirstlane_b32 s84, v126\n"
      "v_add_u32 v121, 0x4000, v121\n"
      "s_cmp_lt_u32 s84, s85\n"
      "s_cbranch_scc1 L_slowFp1_%=\n"
      "L_backFp1_%=:\n"
      "v_and_b32 v121, 0x1ffff, v121\n"
      "v_mfma_f32_16x16x128_f8f6f4 v[24:27], a[120:127], a[72:79], v[24:27]\n"
      "ds_read_b32 v126, v124 offset:48\n"
      "ds_read_b64_tr_b8 a[112:113], v121 offset:1024\n"
      "ds_read_b64_tr_b8 a[114:115], v121 offset:2048\n"
      "v_mfma_f32_16x16x128_f8f6f4 v[56:59], a[120:127], a[80:87], v[56:59]\n"
      "ds_read_b64_tr_b8 a[116:117], v121 offset:9216\n"
      "ds_read_b64_tr_b8 a[118:119], v121 offset:10240\n"
      "v_mbcnt_lo_u32_b32 v120, -1, 0\n"
      "v_mfma_f32_16x16x128_f8f6f4 v[88:91], a[120:127], a[88:95], v[88:91]\n"
      "v_mbcnt_hi_u32_b32 v120, -1, v120\n"
      "v_mfma_f32_16x16x128_f8f6f4 a[0:3], a[120:127], a[96:103], a[0:3]\n"
      "ds_write_b32 v124, v125 offset:16\n"
      "v_lshlrev_b32 v120, 4, v120\n"
      "s_waitcnt lgkmcnt(6)\n"
      "v_mfma_f32_16x16x128_f8f6f4 v[28:31], a[104:111], a[72:79], v[28:31]\n"
      "v_xor_b32 v123, 16, v121\n"
      "ds_read_b64_tr_b8 a[120:121], v123 offset:1024\n"
      "ds_read_b64_tr_b8 a[122:123], v123 offset:2048\n"
      "v_mfma_f32_16x16x128_f8f6f4 v[60:63], a[104:111], a[80:87], v[60:63]\n"
      "ds_read_b64_tr_b8 a[124:125], v123 offset:9216\n"
      "ds_read_b64_tr_b8 a[126:127], v123 offset:10240\n"
      "s_waitcnt lgkmcnt(9)\n"
      "v_readfirstlane_b32 s50, v127\n"
      "v_readfirstlane_b32 s58, v126\n"
      "v_add_u32 v120, %[awoff], v120\n"
      "v_mfma_f32_16x16x128_f8f6f4 v[92:95], a[104:111], a[88:95], v[92:95]\n"
      "s_cmp_eq_u32 s50, 0\n"
      "s_cselect_b32 s53, 1, s58\n"
      "s_add_u32 s51, s50, -1\n"
      "s_add_u32 s52, s51, 4096\n"
      "v_mfma_f32_16x16x128_f8f6f4 a[4:7], a[104:111], a[96:103], a[4:7]\n"
      "s_add_u32 s85, s85, 1\n"
      "L_D0_%=:\n"
      "s_cmp_lg_u32 s62, 0\n"
      "s_cbranch_scc1 L_ON0_%=\n"
      "s_waitcnt vmcnt(8)\n"
      "s_cmp_lg_u32 s53, 0\n"
      "s_cbranch_scc1 L_SK0_%=\n"
      "s_waitcnt lgkmcnt(4)\n"
      "v_mfma_f32_16x16x128_f8f6f4 v[0:3], a[112:119], a[8:15], v[0:3]\n"
      "v_xor_b32 v122, 32, v121\n"
      "ds_read_b64_tr_b8 a[104:105], v122 offset:1024\n"
      "ds_read_b64_tr_b8 a[106:107], v122 offset:2048\n"
      "buffer_load_dwordx4 a[72:75], v120, s[68:71], s51 offen offset:0 sc0 sc1\n"
      "v_mfma_f32_16x16x128_f8f6f4 v[32:35], a[112:119], a[16:23], v[32:35]\n"
      "ds_read_b64_tr_b8 a[108:109], v122 offset:9216\n"
      "ds_read_b64_tr_b8 a[110:111], v122 offset:10240\n"
      "buffer_load_dwordx4 a[76:79], v120, s[68:71], s51 offen offset:1024 sc0 sc1\n"
      "v_mfma_f32_16x16x128_f8f6f4 v[64:67], a[112:119], a[24:31], v[64:67]\n"
      "buffer_load_dwordx4 a[80:83], v120, s[68:71], s51 offen offset:2048 sc0 sc1\n"
      "v_mfma_f32_16x16x128_f8f6f4 v[96:99], a[112:119], a[32:39], v[96:99]\n"
      "buffer_load_dwordx4 a[84:87], v120, s[68:71], s51 offen offset:3072 sc0 sc1\n"
      "s_waitcnt lgkmcnt(4)\n"
      "v_mfma_f32_16x16x128_f8f6f4 v[4:7], a[120:127], a[8:15], v[4:7]\n"
      "v_xor_b32 v123, 48, v121\n"
      "ds_read_b64_tr_b8 a[112:113], v123 offset:1024\n"
      "ds_read_b64_tr_b8 a[114:115], v123 offset:2048\n"
      "buffer_load_dwordx4 a[88:91], v120, s[68:71], s52 offen offset:0 sc0 sc1\n"
      "v_mfma_f32_16x16x128_f8f6f4 v[36:39], a[120:127], a[16:23], v[36:39]\n"
      "ds_read_b64_tr_b8 a[116:117], v123 offset:9216\n"
      "ds_read_b64_tr_b8 a[118:119], v123 offset:10240\n"
      "buffer_load_dwordx4 a[92:95], v120, s[68:71], s52 offen offset:1024 sc0 sc1\n"
      "v_mfma_f32_16x16x128_f8f6f4 v[68:71], a[120:127], a[24:31], v[68:71]\n"
      "buffer_load_dwordx4 a[96:99], v120, s[68:71], s52 offen offset:2048 sc0 sc1\n"
      "v_mfma_f32_16x16x128_f8f6f4 v[100:103], a[120:127], a[32:39], v[100:103]\n"
      "buffer_load_dwordx4 a[100:103], v120, s[68:71], s52 offen offset:3072 sc0 sc1\n"
      "s_waitcnt lgkmcnt(4)\n"
      "v_mfma_f32_16x16x128_f8f6f4 v[8:11], a[104:111], a[8:15], v[8:11]\n"
      "v_xor_b32 v122, 64, v121\n"
      "ds_read_b64_tr_b8 a[120:121], v122 offset:1024\n"
      "ds_read_b64_tr_b8 a[122:123], v122 offset:2048\n"
      "v_mfma_f32_16x16x128_f8f6f4 v[40:43], a[104:111], a[16:23], v[40:43]\n"
      "ds_read_b64_tr_b8 a[124:125], v122 offset:9216\n"
      "ds_read_b64_tr_b8 a[126:127], v122 offset:10240\n"
      "v_mfma_f32_16x16x128_f8f6f4 v[72:75], a[104:111], a[24:31], v[72:75]\n"
      "v_mfma_f32_16x16x128_f8f6f4 v[104:107], a[104:111], a[32:39], v[104:107]\n"
      "ds_write_b32 v124, v125 offset:16\n"
      "s_waitcnt lgkmcnt(5)\n"
      "v_mfma_f32_16x16x128_f8f6f4 v[12:15], a[112:119], a[8:15], v[12:15]\n"
      "v_xor_b32 v123, 80, v121\n"
      "ds_read_b64_tr_b8 a[104:105], v123 offset:1024\n"
      "ds_read_b64_tr_b8 a[106:107], v123 offset:2048\n"
      "v_mfma_f32_16x16x128_f8f6f4 v[44:47], a[112:119], a[16:23], v[44:47]\n"
      "ds_read_b64_tr_b8 a[108:109], v123 offset:9216\n"
      "ds_read_b64_tr_b8 a[110:111], v123 offset:10240\n"
      "v_mfma_f32_16x16x128_f8f6f4 v[76:79], a[112:119], a[24:31], v[76:79]\n"
      "v_mfma_f32_16x16x128_f8f6f4 v[108:111], a[112:119], a[32:39], v[108:111]\n"
      "s_waitcnt lgkmcnt(5)\n"
      "v_mfma_f32_16x16x128_f8f6f4 v[16:19], a[120:127], a[8:15], v[16:19]\n"
      "ds_read_b32 v126, v124 offset:0\n"
      "ds_read_b32 v127, v125 offset:76\n"
      "v_xor_b32 v122, 96, v121\n"
      "ds_read_b64_tr_b8 a[112:113], v122 offset:1024\n"
      "ds_read_b64_tr_b8 a[114:115], v122 offset:2048\n"
      "v_mfma_f32_16x16x128_f8f6f4 v[48:51], a[120:127], a[16:23], v[48:51]\n"
      "ds_read_b64_tr_b8 a[116:117], v122 offset:9216\n"
      "ds_read_b64_tr_b8 a[118:119], v122 offset:10240\n"
      "v_mfma_f32_16x16x128_f8f6f4 v[80:83], a[120:127], a[24:31], v[80:83]\n"
      "v_mfma_f32_16x16x128_f8f6f4 v[112:115], a[120:127], a[32:39], v[112:115]\n"
      "s_waitcnt lgkmcnt(6)\n"
      "v_mfma_f32_16x16x128_f8f6f4 v[20:23], a[104:111], a[8:15], v[20:23]\n"
      "v_xor_b32 v123, 112, v121\n"
      "ds_read_b64_tr_b8 a[120:121], v123 offset:1024\n"
      "ds_read_b64_tr_b8 a[122:123], v123 offset:2048\n"
      "v_mfma_f32_16x16x128_f8f6f4 v[52:55], a[104:111], a[16:23], v[52:55]\n"
      "ds_read_b64_tr_b8 a[124:125], v123 offset:9216\n"
      "ds_read_b64_tr_b8 a[126:127], v123 offset:10240\n"
      "v_mfma_f32_16x16x128_f8f6f4 v[84:87], a[104:111], a[24:31], v[84:87]\n"
      "v_add_u32 v125, 4, v125\n"
      "v_mfma_f32_16x16x128_f8f6f4 v[116:119], a[104:111], a[32:39], v[116:119]\n"
      "s_waitcnt lgkmcnt(4)\n"
      "v_readfirstlane_b32 s84, v126\n"
      "v_add_u32 v121, 0x4000, v121\n"
      "s_cmp_lt_u32 s84, s85\n"
      "s_cbranch_scc1 L_slowFN0_%=\n"
      "L_backFN0_%=:\n"
      "v_and_b32 v121, 0x1ffff, v121\n"
      "v_mfma_f32_16x16x128_f8f6f4 v[24:27], a[112:119], a[8:15], v[24:27]\n"
      "ds_read_b32 v126, v124 offset:48\n"
      "ds_read_b64_tr_b8 a[104:105], v121 offset:1024\n"
      "ds_read_b64_tr_b8 a[106:107], v121 offset:2048\n"
      "v_mfma_f32_16x16x128_f8f6f4 v[56:59], a[112:119], a[16:23], v[56:59]\n"
      "ds_read_b64_tr_b8 a[108:109], v121 offset:9216\n"
      "ds_read_b64_tr_b8 a[110:111], v121 offset:10240\n"
      "v_mfma_f32_16x16x128_f8f6f4 v[88:91], a[112:119], a[24:31], v[88:91]\n"
      "v_mfma_f32_16x16x128_f8f6f4 a[0:3], a[112:119], a[32:39], a[0:3]\n"
      "s_waitcnt lgkmcnt(5)\n"
      "v_mfma_f32_16x16x128_f8f6f4 v[28:31], a[120:127], a[8:15], v[28:31]\n"
      "v_xor_b32 v123, 16, v121\n"
      "ds_read_b64_tr_b8 a[112:113], v123 offset:1024\n"
      "ds_read_b64_tr_b8 a[114:115], v123 offset:2048\n"
      "v_mfma_f32_16x16x128_f8f6f4 v[60:63], a[120:127], a[16:23], v[60:63]\n"
      "ds_read_b64_tr_b8 a[116:117], v123 offset:9216\n"
      "ds_read_b64_tr_b8 a[118:119], v123 offset:10240\n"
      "s_waitcnt lgkmcnt(8)\n"
      "v_readfirstlane_b32 s50, v127\n"
      "v_readfirstlane_b32 s58, v126\n"
      "v_mfma_f32_16x16x128_f8f6f4 v[92:95], a[120:127], a[24:31], v[92:95]\n"
      "s_cmp_eq_u32 s50, 0\n"
      "s_cselect_b32 s53, 1, s58\n"
      "s_add_u32 s51, s50, -1\n"
      "s_add_u32 s52, s51, 4096\n"
      "v_mfma_f32_16x16x128_f8f6f4 a[4:7], a[120:127], a[32:39], a[4:7]\n"
      "s_add_u32 s85, s85, 1\n"
      "L_D1_%=:\n"
      "s_cmp_lg_u32 s62, 0\n"
      "s_cbranch_scc1 L_ON1_%=\n"
      "s_waitcnt vmcnt(8)\n"
      "s_cmp_lg_u32 s53, 0\n"
      "s_cbranch_scc1 L_SK1_%=\n"
      "s_waitcnt lgkmcnt(4)\n"
      "v_mfma_f32_16x16x128_f8f6f4 v[0:3], a[104:111], a[40:47], v[0:3]\n"
      "v_xor_b32 v122, 32, v121\n"
      "ds_read_b64_tr_b8 a[120:121], v122 offset:1024\n"
      "ds_read_b64_tr_b8 a[122:123], v122 offset:2048\n"
      "buffer_load_dwordx4 a[8:11], v120, s[68:71], s51 offen offset:0 sc0 sc1\n"
      "v_mfma_f32_16x16x128_f8f6f4 v[32:35], a[104:111], a[48:55], v[32:35]\n"
      "ds_read_b64_tr_b8 a[124:125], v122 offset:9216\n"
      "ds_read_b64_tr_b8 a[126:127], v122 offset:10240\n"
      "buffer_load_dwordx4 a[12:15], v120, s[68:71], s51 offen offset:1024 sc0 sc1\n"
      "v_mfma_f32_16x16x128_f8f6f4 v[64:67], a[104:111], a[56:63], v[64:67]\n"
      "buffer_load_dwordx4 a[16:19], v120, s[68:71], s51 offen offset:2048 sc0 sc1\n"
      "v_mfma_f32_16x16x128_f8f6f4 v[96:99], a[104:111], a[64:71], v[96:99]\n"
      "buffer_load_dwordx4 a[20:23], v120, s[68:71], s51 offen offset:3072 sc0 sc1\n"
      "s_waitcnt lgkmcnt(4)\n"
      "v_mfma_f32_16x16x128_f8f6f4 v[4:7], a[112:119], a[40:47], v[4:7]\n"
      "v_xor_b32 v123, 48, v121\n"
      "ds_read_b64_tr_b8 a[104:105], v123 offset:1024\n"
      "ds_read_b64_tr_b8 a[106:107], v123 offset:2048\n"
      "buffer_load_dwordx4 a[24:27], v120, s[68:71], s52 offen offset:0 sc0 sc1\n"
      "v_mfma_f32_16x16x128_f8f6f4 v[36:39], a[112:119], a[48:55], v[36:39]\n"
      "ds_read_b64_tr_b8 a[108:109], v123 offset:9216\n"
      "ds_read_b64_tr_b8 a[110:111], v123 offset:10240\n"
      "buffer_load_dwordx4 a[28:31], v120, s[68:71], s52 offen offset:1024 sc0 sc1\n"
      "v_mfma_f32_16x16x128_f8f6f4 v[68:71], a[112:119], a[56:63], v[68:71]\n"
      "buffer_load_dwordx4 a[32:35], v120, s[68:71], s52 offen offset:2048 sc0 sc1\n"
      "v_mfma_f32_16x16x128_f8f6f4 v[100:103], a[112:119], a[64:71], v[100:103]\n"
      "buffer_load_dwordx4 a[36:39], v120, s[68:71], s52 offen offset:3072 sc0 sc1\n"
      "s_waitcnt lgkmcnt(4)\n"
      "v_mfma_f32_16x16x128_f8f6f4 v[8:11], a[120:127], a[40:47], v[8:11]\n"
      "v_xor_b32 v122, 64, v121\n"
      "ds_read_b64_tr_b8 a[112:113], v122 offset:1024\n"
      "ds_read_b64_tr_b8 a[114:115], v122 offset:2048\n"
      "v_mfma_f32_16x16x128_f8f6f4 v[40:43], a[120:127], a[48:55], v[40:43]\n"
      "ds_read_b64_tr_b8 a[116:117], v122 offset:9216\n"
      "ds_read_b64_tr_b8 a[118:119], v122 offset:10240\n"
      "v_mfma_f32_16x16x128_f8f6f4 v[72:75], a[120:127], a[56:63], v[72:75]\n"
      "v_mfma_f32_16x16x128_f8f6f4 v[104:107], a[120:127], a[64:71], v[104:107]\n"
      "ds_write_b32 v124, v125 offset:16\n"
      "s_waitcnt lgkmcnt(5)\n"
      "v_mfma_f32_16x16x128_f8f6f4 v[12:15], a[104:111], a[40:47], v[12:15]\n"
      "v_xor_b32 v123, 80, v121\n"
      "ds_read_b64_tr_b8 a[120:121], v123 offset:1024\n"
      "ds_read_b64_tr_b8 a[122:123], v123 offset:2048\n"
      "v_mfma_f32_16x16x128_f8f6f4 v[44:47], a[104:111], a[48:55], v[44:47]\n"
      "ds_read_b64_tr_b8 a[124:125], v123 offset:9216\n"
      "ds_read_b64_tr_b8 a[126:127], v123 offset:10240\n"
      "v_mfma_f32_16x16x128_f8f6f4 v[76:79], a[104:111], a[56:63], v[76:79]\n"
      "v_mfma_f32_16x16x128_f8f6f4 v[108:111], a[104:111], a[64:71], v[108:111]\n"
      "s_waitcnt lgkmcnt(5)\n"
      "v_mfma_f32_16x16x128_f8f6f4 v[16:19], a[112:119], a[40:47], v[16:19]\n"
      "ds_read_b32 v126, v124 offset:0\n"
      "ds_read_b32 v127, v125 offset:76\n"
      "v_xor_b32 v122, 96, v121\n"
      "ds_read_b64_tr_b8 a[104:105], v122 offset:1024\n"
      "ds_read_b64_tr_b8 a[106:107], v122 offset:2048\n"
      "v_mfma_f32_16x16x128_f8f6f4 v[48:51], a[112:119], a[48:55], v[48:51]\n"
      "ds_read_b64_tr_b8 a[108:109], v122 offset:9216\n"
      "ds_read_b64_tr_b8 a[110:111], v122 offset:10240\n"
      "v_mfma_f32_16x16x128_f8f6f4 v[80:83], a[112:119], a[56:63], v[80:83]\n"
      "v_mfma_f32_16x16x128_f8f6f4 v[112:115], a[112:119], a[64:71], v[112:115]\n"
      "s_waitcnt lgkmcnt(6)\n"
      "v_mfma_f32_16x16x128_f8f6f4 v[20:23], a[120:127], a[40:47], v[20:23]\n"
      "v_xor_b32 v123, 112, v121\n"
      "ds_read_b64_tr_b8 a[112:113], v123 offset:1024\n"
      "ds_read_b64_tr_b8 a[114:115], v123 offset:2048\n"
      "v_mfma_f32_16x16x128_f8f6f4 v[52:55], a[120:127], a[48:55], v[52:55]\n"
      "ds_read_b64_tr_b8 a[116:117], v123 offset:9216\n"
      "ds_read_b64_tr_b8 a[118:119], v123 offset:10240\n"
      "v_mfma_f32_16x16x128_f8f6f4 v[84:87], a[120:127], a[56:63], v[84:87]\n"
      "v_add_u32 v125, 4, v125\n"
      "v_mfma_f32_16x16x128_f8f6f4 v[116:119], a[120:127], a[64:71], v[116:119]\n"
      "s_waitcnt lgkmcnt(4)\n"
      "v_readfirstlane_b32 s84, v126\n"
      "v_add_u32 v121, 0x4000, v121\n"
      "s_cmp_lt_u32 s84, s85\n"
      "s_cbranch_scc1 L_slowFN1_%=\n"
      "L_backFN1_%=:\n"
      "v_and_b32 v121, 0x1ffff, v121\n"
      "v_mfma_f32_16x16x128_f8f6f4 v[24:27], a[104:111], a[40:47], v[24:27]\n"
      "ds_read_b32 v126, v124 offset:48\n"
      "ds_read_b64_tr_b8 a[120:121], v121 offset:1024\n"
      "ds_read_b64_tr_b8 a[122:123], v121 offset:2048\n"
      "v_mfma_f32_16x16x128_f8f6f4 v[56:59], a[104:111], a[48:55], v[56:59]\n"
      "ds_read_b64_tr_b8 a[124:125], v121 offset:9216\n"
      "ds_read_b64_tr_b8 a[126:127], v121 offset:10240\n"
      "v_mfma_f32_16x16x128_f8f6f4 v[88:91], a[104:111], a[56:63], v[88:91]\n"
      "v_mfma_f32_16x16x128_f8f6f4 a[0:3], a[104:111], a[64:71], a[0:3]\n"
      "s_waitcnt lgkmcnt(5)\n"
      "v_mfma_f32_16x16x128_f8f6f4 v[28:31], a[112:119], a[40:47], v[28:31]\n"
      "v_xor_b32 v123, 16, v121\n"
      "ds_read_b64_tr_b8 a[104:105], v123 offset:1024\n"
      "ds_read_b64_tr_b8 a[106:107], v123 offset:2048\n"
      "v_mfma_f32_16x16x128_f8f6f4 v[60:63], a[112:119], a[48:55], v[60:63]\n"
      "ds_read_b64_tr_b8 a[108:109], v123 offset:9216\n"
      "ds_read_b64_tr_b8 a[110:111], v123 offset:10240\n"
      "s_waitcnt lgkmcnt(8)\n"
      "v_readfirstlane_b32 s50, v127\n"
      "v_readfirstlane_b32 s58, v126\n"
      "v_mfma_f32_16x16x128_f8f6f4 v[92:95], a[112:119], a[56:63], v[92:95]\n"
      "s_cmp_eq_u32 s50, 0\n"
      "s_cselect_b32 s53, 1, s58\n"
      "s_add_u32 s51, s50, -1\n"
      "s_add_u32 s52, s51, 4096\n"
      "v_mfma_f32_16x16x128_f8f6f4 a[4:7], a[112:119], a[64:71], a[4:7]\n"
      "s_add_u32 s85, s85, 1\n"
      "L_D2_%=:\n"
      "s_cmp_lg_u32 s62, 0\n"
      "s_cbranch_scc1 L_ON2_%=\n"
      "s_waitcnt vmcnt(8)\n"
      "s_cmp_lg_u32 s53, 0\n"
      "s_cbranch_scc1 L_SK2_%=\n"
      "s_waitcnt lgkmcnt(4)\n"
      "v_mfma_f32_16x16x128_f8f6f4 v[0:3], a[120:127], a[72:79], v[0:3]\n"
      "v_xor_b32 v122, 32, v121\n"
      "ds_read_b64_tr_b8 a[112:113], v122 offset:1024\n"
      "ds_read_b64_tr_b8 a[114:115], v122 offset:2048\n"
      "buffer_load_dwordx4 a[40:43], v120, s[68:71], s51 offen offset:0 sc0 sc1\n"
      "v_mfma_f32_16x16x128_f8f6f4 v[32:35], a[120:127], a[80:87], v[32:35]\n"
      "ds_read_b64_tr_b8 a[116:117], v122 offset:9216\n"
      "ds_read_b64_tr_b8 a[118:119], v122 offset:10240\n"
      "buffer_load_dwordx4 a[44:47], v120, s[68:71], s51 offen offset:1024 sc0 sc1\n"
      "v_mfma_f32_16x16x128_f8f6f4 v[64:67], a[120:127], a[88:95], v[64:67]\n"
      "buffer_load_dwordx4 a[48:51], v120, s[68:71], s51 offen offset:2048 sc0 sc1\n"
      "v_mfma_f32_16x16x128_f8f6f4 v[96:99], a[120:127], a[96:103], v[96:99]\n"
      "buffer_load_dwordx4 a[52:55], v120, s[68:71], s51 offen offset:3072 sc0 sc1\n"
      "s_waitcnt lgkmcnt(4)\n"
      "v_mfma_f32_16x16x128_f8f6f4 v[4:7], a[104:111], a[72:79], v[4:7]\n"
      "v_xor_b32 v123, 48, v121\n"
      "ds_read_b64_tr_b8 a[120:121], v123 offset:1024\n"
      "ds_read_b64_tr_b8 a[122:123], v123 offset:2048\n"
      "buffer_load_dwordx4 a[56:59], v120, s[68:71], s52 offen offset:0 sc0 sc1\n"
      "v_mfma_f32_16x16x128_f8f6f4 v[36:39], a[104:111], a[80:87], v[36:39]\n"
      "ds_read_b64_tr_b8 a[124:125], v123 offset:9216\n"
      "ds_read_b64_tr_b8 a[126:127], v123 offset:10240\n"
      "buffer_load_dwordx4 a[60:63], v120, s[68:71], s52 offen offset:1024 sc0 sc1\n"
      "v_mfma_f32_16x16x128_f8f6f4 v[68:71], a[104:111], a[88:95], v[68:71]\n"
      "buffer_load_dwordx4 a[64:67], v120, s[68:71], s52 offen offset:2048 sc0 sc1\n"
      "v_mfma_f32_16x16x128_f8f6f4 v[100:103], a[104:111], a[96:103], v[100:103]\n"
      "buffer_load_dwordx4 a[68:71], v120, s[68:71], s52 offen offset:3072 sc0 sc1\n"
      "s_waitcnt lgkmcnt(4)\n"
      "v_mfma_f32_16x16x128_f8f6f4 v[8:11], a[112:119], a[72:79], v[8:11]\n"
      "v_xor_b32 v122, 64, v121\n"
      "ds_read_b64_tr_b8 a[104:105], v122 offset:1024\n"
      "ds_read_b64_tr_b8 a[106:107], v122 offset:2048\n"
      "v_mfma_f32_16x16x128_f8f6f4 v[40:43], a[112:119], a[80:87], v[40:43]\n"
      "ds_read_b64_tr_b8 a[108:109], v122 offset:9216\n"
      "ds_read_b64_tr_b8 a[110:111], v122 offset:10240\n"
      "v_mfma_f32_16x16x128_f8f6f4 v[72:75], a[112:119], a[88:95], v[72:75]\n"
      "v_mfma_f32_16x16x128_f8f6f4 v[104:107], a[112:119], a[96:103], v[104:107]\n"
      "ds_write_b32 v124, v125 offset:16\n"
      "s_waitcnt lgkmcnt(5)\n"
      "v_mfma_f32_16x16x128_f8f6f4 v[12:15], a[120:127], a[72:79], v[12:15]\n"
      "v_xor_b32 v123, 80, v121\n"
      "ds_read_b64_tr_b8 a[112:113], v123 offset:1024\n"
      "ds_read_b64_tr_b8 a[114:115], v123 offset:2048\n"
      "v_mfma_f32_16x16x128_f8f6f4 v[44:47], a[120:127], a[80:87], v[44:47]\n"
      "ds_read_b64_tr_b8 a[116:117], v123 offset:9216\n"
      "ds_read_b64_tr_b8 a[118:119], v123 offset:10240\n"
      "v_mfma_f32_16x16x128_f8f6f4 v[76:79], a[120:127], a[88:95], v[76:79]\n"
      "v_mfma_f32_16x16x128_f8f6f4 v[108:111], a[120:127], a[96:103], v[108:111]\n"
      "s_waitcnt lgkmcnt(5)\n"
      "v_mfma_f32_16x16x128_f8f6f4 v[16:19], a[104:111], a[72:79], v[16:19]\n"
      "ds_read_b32 v126, v124 offset:0\n"
      "ds_read_b32 v127, v125 offset:76\n"
      "v_xor_b32 v122, 96, v121\n"
      "ds_read_b64_tr_b8 a[120:121], v122 offset:1024\n"
      "ds_read_b64_tr_b8 a[122:123], v122 offset:2048\n"
      "v_mfma_f32_16x16x128_f8f6f4 v[48:51], a[104:111], a[80:87], v[48:51]\n"
      "ds_read_b64_tr_b8 a[124:125], v122 offset:9216\n"
      "ds_read_b64_tr_b8 a[126:127], v122 offset:10240\n"
      "v_mfma_f32_16x16x128_f8f6f4 v[80:83], a[104:111], a[88:95], v[80:83]\n"
      "v_mfma_f32_16x16x128_f8f6f4 v[112:115], a[104:111], a[96:103], v[112:115]\n"
      "s_waitcnt lgkmcnt(6)\n"
      "v_mfma_f32_16x16x128_f8f6f4 v[20:23], a[112:119], a[72:79], v[20:23]\n"
      "v_xor_b32 v123, 112, v121\n"
      "ds_read_b64_tr_b8 a[104:105], v123 offset:1024\n"
      "ds_read_b64_tr_b8 a[106:107], v123 offset:2048\n"
      "v_mfma_f32_16x16x128_f8f6f4 v[52:55], a[112:119], a[80:87], v[52:55]\n"
      "ds_read_b64_tr_b8 a[108:109], v123 offset:9216\n"
      "ds_read_b64_tr_b8 a[110:111], v123 offset:10240\n"
      "v_mfma_f32_16x16x128_f8f6f4 v[84:87], a[112:119], a[88:95], v[84:87]\n"
      "v_add_u32 v125, 4, v125\n"
      "v_mfma_f32_16x16x128_f8f6f4 v[116:119], a[112:119], a[96:103], v[116:119]\n"
      "s_waitcnt lgkmcnt(4)\n"
      "v_readfirstlane_b32 s84, v126\n"
      "v_add_u32 v121, 0x4000, v121\n"
      "s_cmp_lt_u32 s84, s85\n"
      "s_cbranch_scc1 L_slowFN2_%=\n"
      "L_backFN2_%=:\n"
      "v_and_b32 v121, 0x1ffff, v121\n"
      "v_mfma_f32_16x16x128_f8f6f4 v[24:27], a[120:127], a[72:79], v[24:27]\n"
      "ds_read_b32 v126, v124 offset:48\n"
      "ds_read_b64_tr_b8 a[112:113], v121 offset:1024\n"
      "ds_read_b64_tr_b8 a[114:115], v121 offset:2048\n"
      "v_mfma_f32_16x16x128_f8f6f4 v[56:59], a[120:127], a[80:87], v[56:59]\n"
      "ds_read_b64_tr_b8 a[116:117], v121 offset:9216\n"
      "ds_read_b64_tr_b8 a[118:119], v121 offset:10240\n"
      "v_mfma_f32_16x16x128_f8f6f4 v[88:91], a[120:127], a[88:95], v[88:91]\n"
      "v_mfma_f32_16x16x128_f8f6f4 a[0:3], a[120:127], a[96:103], a[0:3]\n"
      "s_waitcnt lgkmcnt(5)\n"
      "v_mfma_f32_16x16x128_f8f6f4 v[28:31], a[104:111], a[72:79], v[28:31]\n"
      "v_xor_b32 v123, 16, v121\n"
      "ds_read_b64_tr_b8 a[120:121], v123 offset:1024\n"
      "ds_read_b64_tr_b8 a[122:123], v123 offset:2048\n"
      "v_mfma_f32_16x16x128_f8f6f4 v[60:63], a[104:111], a[80:87], v[60:63]\n"
      "ds_read_b64_tr_b8 a[124:125], v123 offset:9216\n"
      "ds_read_b64_tr_b8 a[126:127], v123 offset:10240\n"
      "s_waitcnt lgkmcnt(8)\n"
      "v_readfirstlane_b32 s50, v127\n"
      "v_readfirstlane_b32 s58, v126\n"
      "v_mfma_f32_16x16x128_f8f6f4 v[92:95], a[104:111], a[88:95], v[92:95]\n"
      "s_cmp_eq_u32 s50, 0\n"
      "s_cselect_b32 s53, 1, s58\n"
      "s_add_u32 s51, s50, -1\n"
      "s_add_u32 s52, s51, 4096\n"
      "v_mfma_f32_16x16x128_f8f6f4 a[4:7], a[104:111], a[96:103], a[4:7]\n"
      "s_add_u32 s85, s85, 1\n"
      "s_cmp_lt_u32 s85, 31\n"
      "s_cbranch_scc1 L_D0_%=\n"
      "s_branch L_EPI_%=\n"
      "L_EPI_%=:\n"
      "v_readfirstlane_b32 s79, v127\n"
      "s_cmp_eq_u32 s62, 0\n"
      "s_cbranch_scc1 L_tno_%=\n"
      "s_cmp_eq_u32 s63, 0\n"
      "s_cbranch_scc1 L_slowOt_%=\n"
      "L_backOt_%=:\n"
      "s_add_u32 s63, s63, -1\n"
      "s_add_u32 s59, s63, 4096\n"
      "buffer_load_dwordx4 a[40:43], v120, s[68:71], s63 offen offset:0 sc0 sc1\n"
      "buffer_load_dwordx4 a[44:47], v120, s[68:71], s63 offen offset:1024 sc0 sc1\n"
      "buffer_load_dwordx4 a[48:51], v120, s[68:71], s63 offen offset:2048 sc0 sc1\n"
      "buffer_load_dwordx4 a[52:55], v120, s[68:71], s63 offen offset:3072 sc0 sc1\n"
      "buffer_load_dwordx4 a[56:59], v120, s[68:71], s59 offen offset:0 sc0 sc1\n"
      "buffer_load_dwordx4 a[60:63], v120, s[68:71], s59 offen offset:1024 sc0 sc1\n"
      "buffer_load_dwordx4 a[64:67], v120, s[68:71], s59 offen offset:2048 sc0 sc1\n"
      "buffer_load_dwordx4 a[68:71], v120, s[68:71], s59 offen offset:3072 sc0 sc1\n"
      "L_tno_%=:\n"
      "s_cmp_eq_u32 s79, 0\n"
      "s_cbranch_scc1 L_slowAt_%=\n"
      "L_backAt_%=:\n"
      "s_add_u32 s79, s79, -1\n"
      "s_add_u32 s78, s79, 4096\n"
      "buffer_load_dwordx4 a[72:75], v120, s[68:71], s79 offen offset:0 sc0 sc1\n"
      "buffer_load_dwordx4 a[76:79], v120, s[68:71], s79 offen offset:1024 sc0 sc1\n"
      "buffer_load_dwordx4 a[80:83], v120, s[68:71], s79 offen offset:2048 sc0 sc1\n"
      "buffer_load_dwordx4 a[84:87], v120, s[68:71], s79 offen offset:3072 sc0 sc1\n"
      "buffer_load_dwordx4 a[88:91], v120, s[68:71], s78 offen offset:0 sc0 sc1\n"
      "buffer_load_dwordx4 a[92:95], v120, s[68:71], s78 offen offset:1024 sc0 sc1\n"
      "buffer_load_dwordx4 a[96:99], v120, s[68:71], s78 offen offset:2048 sc0 sc1\n"
      "buffer_load_dwordx4 a[100:103], v120, s[68:71], s78 offen offset:3072 sc0 sc1\n"
      "ds_write_b32 v124, v125 offset:16\n"
      "s_mov_b32 s85, 33\n"
      "s_mov_b32 s86, 0\n"
      "L_tw_%=:\n"
      "ds_read_b32 v126, v124 offset:0\n"
      "s_waitcnt lgkmcnt(0)\n"
      "v_readfirstlane_b32 s84, v126\n"
      "s_cmp_ge_u32 s84, s85\n"
      "s_cbranch_scc1 L_twd_%=\n"
      "s_sleep 1\n"
      "s_add_u32 s86, s86, 1\n"
      "s_cmp_lt_u32 s86, 0x800000\n"
      "s_cbranch_scc1 L_tw_%=\n"
      "s_mov_b32 s87, 1\n"
      "L_twd_%=:\n"
      "v_mov_b32 v122, 0\n"
      "s_mov_b32 s86, 0\n"
      "L_fin_%=:\n"
      "ds_read_b32 v123, v122 offset:32\n"
      "s_waitcnt lgkmcnt(0)\n"
      "v_readfirstlane_b32 s84, v123\n"
      "s_cmp_lg_u32 s84, 0\n"
      "s_cbranch_scc1 L_find_%=\n"
      "s_sleep 1\n"
      "s_add_u32 s86, s86, 1\n"
      "s_cmp_lt_u32 s86, 0x800000\n"
      "s_cbranch_scc1 L_fin_%=\n"
      "s_mov_b32 s87, 1\n"
      "L_find_%=:\n"
      "s_cmp_eq_u32 s84, 2\n"
      "s_cselect_b32 s87, 1, s87\n"
      "s_cmp_lg_u32 s87, 0\n"
      "s_cbranch_scc1 L_nanpre_%=\n"
      "s_mov_b32 s76, %[ooff]\n"
      "s_add_u32 s77, s76, 0x100000\n"
      "s_add_u32 s78, s77, 0x100000\n"
      "s_add_u32 s79, s78, 0x100000\n"
      "v_mov_b32 v122, 0\n"
      "s_mov_b32 s86, 0\n"
      "L_td_%=:\n"
      "ds_read_b128 v[124:127], v122 offset:16\n"
      "s_waitcnt lgkmcnt(0)\n"
      "v_min_u32 v124, v124, v125\n"
      "v_min_u32 v126, v126, v127\n"
      "v_min_u32 v124, v124, v126\n"
      "s_nop 0\n"
      "v_readfirstlane_b32 s84, v124\n"
      "s_cmp_ge_u32 s84, 116\n"
      "s_cbranch_scc1 L_tdd_%=\n"
      "s_add_u32 s86, s86, 1\n"
      "s_cmp_lt_u32 s86, 0x1000000\n"
      "s_cbranch_scc1 L_td_%=\n"
      "L_tdd_%=:\n"
      "v_add_u32 v124, 0x4000, v121\n"
      "v_and_b32 v124, 0x1ffff, v124\n"
      "v_add_u32 v125, 0x4000, v124\n"
      "v_and_b32 v125, 0x1ffff, v125\n"
      "v_mbcnt_lo_u32_b32 v126, -1, 0\n"
      "v_mbcnt_hi_u32_b32 v126, -1, v126\n"
      "v_lshrrev_b32 v123, 4, v126\n"
      "v_and_b32 v126, 15, v126\n"
      "v_and_b32 v127, 7, v126\n"
      "v_xor_b32 v127, v127, v123\n"
      "v_lshlrev_b32 v126, 7, v126\n"
      "v_lshl_add_u32 v126, v127, 4, v126\n"
      "v_add_u32 v126, %[ost0], v126\n"
      "v_xor_b32 v127, 64, v126\n"
      "s_sub_u32 s60, %[ost2], %[ost0]\n"
      "s_sub_u32 s60, s60, 32768\n"
      "v_mov_b32 v120, %[phw]\n"
      "s_waitcnt vmcnt(16)\n"
      "v_mfma_f32_16x16x128_f8f6f4 v[0:3], a[112:119], a[8:15], v[0:3]\n"
      "ds_read_b64_tr_b8 a[104:105], v124 offset:1024\n"
      "ds_read_b64_tr_b8 a[106:107], v124 offset:2048\n"
      "v_mfma_f32_16x16x128_f8f6f4 v[32:35], a[112:119], a[16:23], v[32:35]\n"
      "ds_read_b64_tr_b8 a[108:109], v124 offset:9216\n"
      "ds_read_b64_tr_b8 a[110:111], v124 offset:10240\n"
      "v_mfma_f32_16x16x128_f8f6f4 v[64:67], a[112:119], a[24:31], v[64:67]\n"
      "v_mfma_f32_16x16x128_f8f6f4 v[96:99], a[112:119], a[32:39], v[96:99]\n"
      "v_mfma_f32_16x16x128_f8f6f4 v[4:7], a[120:127], a[8:15], v[4:7]\n"
      "v_xor_b32 v123, 16, v124\n"
      "ds_read_b64_tr_b8 a[112:113], v123 offset:1024\n"
      "ds_read_b64_tr_b8 a[114:115], v123 offset:2048\n"
      "v_mfma_f32_16x16x128_f8f6f4 v[36:39], a[120:127], a[16:23], v[36:39]\n"
      "ds_read_b64_tr_b8 a[116:117], v123 offset:9216\n"
      "ds_read_b64_tr_b8 a[118:119], v123 offset:10240\n"
      "v_mfma_f32_16x16x128_f8f6f4 v[68:71], a[120:127], a[24:31], v[68:71]\n"
      "v_mfma_f32_16x16x128_f8f6f4 v[100:103], a[120:127], a[32:39], v[100:103]\n"
      "s_waitcnt vmcnt(8)\n"
      "s_waitcnt lgkmcnt(4)\n"
      "v_mfma_f32_16x16x128_f8f6f4 v[0:3], a[104:111], a[40:47], v[0:3]\n"
      "ds_read_b64_tr_b8 a[120:121], v125 offset:1024\n"
      "ds_read_b64_tr_b8 a[122:123], v125 offset:2048\n"
      "v_mfma_f32_16x16x128_f8f6f4 v[32:35], a[104:111], a[48:55], v[32:35]\n"
      "ds_read_b64_tr_b8 a[124:125], v125 offset:9216\n"
      "ds_read_b64_tr_b8 a[126:127], v125 offset:10240\n"
      "v_mfma_f32_16x16x128_f8f6f4 v[64:67], a[104:111], a[56:63], v[64:67]\n"
      "v_mfma_f32_16x16x128_f8f6f4 v[96:99], a[104:111], a[64:71], v[96:99]\n"
      "s_waitcnt lgkmcnt(4)\n"
      "v_mfma_f32_16x16x128_f8f6f4 v[4:7], a[112:119], a[40:47], v[4:7]\n"
      "v_xor_b32 v123, 16, v125\n"
      "ds_read_b64_tr_b8 a[104:105], v123 offset:1024\n"
      "ds_read_b64_tr_b8 a[106:107], v123 offset:2048\n"
      "v_mfma_f32_16x16x128_f8f6f4 v[36:39], a[112:119], a[48:55], v[36:39]\n"
      "ds_read_b64_tr_b8 a[108:109], v123 offset:9216\n"
      "ds_read_b64_tr_b8 a[110:111], v123 offset:10240\n"
      "v_mfma_f32_16x16x128_f8f6f4 v[68:71], a[112:119], a[56:63], v[68:71]\n"
      "v_mfma_f32_16x16x128_f8f6f4 v[100:103], a[112:119], a[64:71], v[100:103]\n"
      "s_waitcnt vmcnt(0)\n"
      "s_waitcnt lgkmcnt(4)\n"
      "v_mfma_f32_16x16x128_f8f6f4 v[0:3], a[120:127], a[72:79], v[0:3]\n"
      "v_xor_b32 v122, 32, v121\n"
      "ds_read_b64_tr_b8 a[112:113], v122 offset:1024\n"
      "ds_read_b64_tr_b8 a[114:115], v122 offset:2048\n"
      "v_mfma_f32_16x16x128_f8f6f4 v[32:35], a[120:127], a[80:87], v[32:35]\n"
      "ds_read_b64_tr_b8 a[116:117], v122 offset:9216\n"
      "ds_read_b64_tr_b8 a[118:119], v122 offset:10240\n"
      "v_mfma_f32_16x16x128_f8f6f4 v[64:67], a[120:127], a[88:95], v[64:67]\n"
      "v_mfma_f32_16x16x128_f8f6f4 v[96:99], a[120:127], a[96:103], v[96:99]\n"
      "s_waitcnt lgkmcnt(4)\n"
      "v_mfma_f32_16x16x128_f8f6f4 v[4:7], a[104:111], a[72:79], v[4:7]\n"
      "v_xor_b32 v123, 48, v121\n"
      "ds_read_b64_tr_b8 a[120:121], v123 offset:1024\n"
      "ds_read_b64_tr_b8 a[122:123], v123 offset:2048\n"
      "v_mfma_f32_16x16x128_f8f6f4 v[36:39], a[104:111], a[80:87], v[36:39]\n"
      "ds_read_b64_tr_b8 a[124:125], v123 offset:9216\n"
      "ds_read_b64_tr_b8 a[126:127], v123 offset:10240\n"
      "v_mfma_f32_16x16x128_f8f6f4 v[68:71], a[104:111], a[88:95], v[68:71]\n"
      "v_mfma_f32_16x16x128_f8f6f4 v[100:103], a[104:111], a[96:103], v[100:103]\n"
      "s_waitcnt lgkmcnt(4)\n"
      "v_mfma_f32_16x16x128_f8f6f4 v[8:11], a[112:119], a[8:15], v[8:11]\n"
      "v_xor_b32 v122, 32, v124\n"
      "ds_read_b64_tr_b8 a[104:105], v122 offset:1024\n"
      "ds_read_b64_tr_b8 a[106:107], v122 offset:2048\n"
      "v_mfma_f32_16x16x128_f8f6f4 v[40:43], a[112:119], a[16:23], v[40:43]\n"
      "ds_read_b64_tr_b8 a[108:109], v122 offset:9216\n"
      "ds_read_b64_tr_b8 a[110:111], v122 offset:10240\n"
      "v_mfma_f32_16x16x128_f8f6f4 v[72:75], a[112:119], a[24:31], v[72:75]\n"
      "ds_write_b128 v126, v[0:3] offset:0\n"
      "v_mfma_f32_16x16x128_f8f6f4 v[104:107], a[112:119], a[32:39], v[104:107]\n"
      "ds_write_b128 v126, v[32:35] offset:2048\n"
      "s_waitcnt lgkmcnt(6)\n"
      "v_mfma_f32_16x16x128_f8f6f4 v[12:15], a[120:127], a[8:15], v[12:15]\n"
      "v_xor_b32 v123, 48, v124\n"
      "ds_read_b64_tr_b8 a[112:113], v123 offset:1024\n"
      "ds_read_b64_tr_b8 a[114:115], v123 offset:2048\n"
      "v_mfma_f32_16x16x128_f8f6f4 v[44:47], a[120:127], a[16:23], v[44:47]\n"
      "ds_read_b64_tr_b8 a[116:117], v123 offset:9216\n"
      "ds_read_b64_tr_b8 a[118:119], v123 offset:10240\n"
      "v_mfma_f32_16x16x128_f8f6f4 v[76:79], a[120:127], a[24:31], v[76:79]\n"
      "ds_write_b128 v126, v[64:67] offset:4096\n"
      "v_mfma_f32_16x16x128_f8f6f4 v[108:111], a[120:127], a[32:39], v[108:111]\n"
      "ds_write_b128 v126, v[96:99] offset:6144\n"
      "s_waitcnt lgkmcnt(8)\n"
      "v_mfma_f32_16x16x128_f8f6f4 v[8:11], a[104:111], a[40:47], v[8:11]\n"
      "v_xor_b32 v122, 32, v125\n"
      "ds_read_b64_tr_b8 a[120:121], v122 offset:1024\n"
      "ds_read_b64_tr_b8 a[122:123], v122 offset:2048\n"
      "v_mfma_f32_16x16x128_f8f6f4 v[40:43], a[104:111], a[48:55], v[40:43]\n"
      "ds_read_b64_tr_b8 a[124:125], v122 offset:9216\n"
      "ds_read_b64_tr_b8 a[126:127], v122 offset:10240\n"
      "v_mfma_f32_16x16x128_f8f6f4 v[72:75], a[104:111], a[56:63], v[72:75]\n"
      "ds_write_b128 v127, v[4:7] offset:0\n"
      "v_mfma_f32_16x16x128_f8f6f4 v[104:107], a[104:111], a[64:71], v[104:107]\n"
      "ds_write_b128 v127, v[36:39] offset:2048\n"
      "s_waitcnt lgkmcnt(8)\n"
      "v_mfma_f32_16x16x128_f8f6f4 v[12:15], a[112:119], a[40:47], v[12:15]\n"
      "v_xor_b32 v123, 48, v125\n"
      "ds_read_b64_tr_b8 a[104:105], v123 offset:1024\n"
      "ds_read_b64_tr_b8 a[106:107], v123 offset:2048\n"
      "v_mfma_f32_16x16x128_f8f6f4 v[44:47], a[112:119], a[48:55], v[44:47]\n"
      "ds_read_b64_tr_b8 a[108:109], v123 offset:9216\n"
      "ds_read_b64_tr_b8 a[110:111], v123 offset:10240\n"
      "v_mfma_f32_16x16x128_f8f6f4 v[76:79], a[112:119], a[56:63], v[76:79]\n"
      "ds_write_b128 v127, v[68:71] offset:4096\n"
      "v_mfma_f32_16x16x128_f8f6f4 v[108:111], a[112:119], a[64:71], v[108:111]\n"
      "ds_write_b128 v127, v[100:103] offset:6144\n"
      "s_waitcnt lgkmcnt(8)\n"
      "v_mfma_f32_16x16x128_f8f6f4 v[8:11], a[120:127], a[72:79], v[8:11]\n"
      "v_xor_b32 v122, 64, v121\n"
      "ds_read_b64_tr_b8 a[112:113], v122 offset:1024\n"
      "ds_read_b64_tr_b8 a[114:115], v122 offset:2048\n"
      "v_mfma_f32_16x16x128_f8f6f4 v[40:43], a[120:127], a[80:87], v[40:43]\n"
      "ds_read_b64_tr_b8 a[116:117], v122 offset:9216\n"
      "ds_read_b64_tr_b8 a[118:119], v122 offset:10240\n"
      "v_mfma_f32_16x16x128_f8f6f4 v[72:75], a[120:127], a[88:95], v[72:75]\n"
      "ds_write_b32 v120, v121 offset:0\n"
      "v_mfma_f32_16x16x128_f8f6f4 v[104:107], a[120:127], a[96:103], v[104:107]\n"
      "s_waitcnt lgkmcnt(7)\n"
      "v_mfma_f32_16x16x128_f8f6f4 v[12:15], a[104:111], a[72:79], v[12:15]\n"
      "v_xor_b32 v123, 80, v121\n"
      "ds_read_b64_tr_b8 a[120:121], v123 offset:1024\n"
      "ds_read_b64_tr_b8 a[122:123], v123 offset:2048\n"
      "v_mfma_f32_16x16x128_f8f6f4 v[44:47], a[104:111], a[80:87], v[44:47]\n"
      "ds_read_b64_tr_b8 a[124:125], v123 offset:9216\n"
      "ds_read_b64_tr_b8 a[126:127], v123 offset:10240\n"
      "v_mfma_f32_16x16x128_f8f6f4 v[76:79], a[104:111], a[88:95], v[76:79]\n"
      "v_mfma_f32_16x16x128_f8f6f4 v[108:111], a[104:111], a[96:103], v[108:111]\n"
      "s_waitcnt lgkmcnt(5)\n"
      "v_mfma_f32_16x16x128_f8f6f4 v[16:19], a[112:119], a[8:15], v[16:19]\n"
      "v_xor_b32 v122, 64, v124\n"
      "ds_read_b64_tr_b8 a[104:105], v122 offset:1024\n"
      "ds_read_b64_tr_b8 a[106:107], v122 offset:2048\n"
      "v_mfma_f32_16x16x128_f8f6f4 v[48:51], a[112:119], a[16:23], v[48:51]\n"
      "ds_read_b64_tr_b8 a[108:109], v122 offset:9216\n"
      "ds_read_b64_tr_b8 a[110:111], v122 offset:10240\n"
      "v_mfma_f32_16x16x128_f8f6f4 v[80:83], a[112:119], a[24:31], v[80:83]\n"
      "v_add_u32 v126, 0x8000, v126\n"
      "v_mfma_f32_16x16x128_f8f6f4 v[112:115], a[112:119], a[32:39], v[112:115]\n"
      "v_add_u32 v127, 0x8000, v127\n"
      "s_waitcnt lgkmcnt(4)\n"
      "v_mfma_f32_16x16x128_f8f6f4 v[20:23], a[120:127], a[8:15], v[20:23]\n"
      "v_xor_b32 v123, 80, v124\n"
      "ds_read_b64_tr_b8 a[112:113], v123 offset:1024\n"
      "ds_read_b64_tr_b8 a[114:115], v123 offset:2048\n"
      "v_mfma_f32_16x16x128_f8f6f4 v[52:55], a[120:127], a[16:23], v[52:55]\n"
      "ds_read_b64_tr_b8 a[116:117], v123 offset:9216\n"
      "ds_read_b64_tr_b8 a[118:119], v123 offset:10240\n"
      "v_mfma_f32_16x16x128_f8f6f4 v[84:87], a[120:127], a[24:31], v[84:87]\n"
      "ds_write_b128 v126, v[8:11] offset:0\n"
      "v_mfma_f32_16x16x128_f8f6f4 v[116:119], a[120:127], a[32:39], v[116:119]\n"
      "ds_write_b128 v126, v[40:43] offset:2048\n"
      "s_waitcnt lgkmcnt(6)\n"
      "v_mfma_f32_16x16x128_f8f6f4 v[16:19], a[104:111], a[40:47], v[16:19]\n"
      "v_xor_b32 v122, 64, v125\n"
      "ds_read_b64_tr_b8 a[120:121], v122 offset:1024\n"
      "ds_read_b64_tr_b8 a[122:123], v122 offset:2048\n"
      "v_mfma_f32_16x16x128_f8f6f4 v[48:51], a[104:111], a[48:55], v[48:51]\n"
      "ds_read_b64_tr_b8 a[124:125], v122 offset:9216\n"
      "ds_read_b64_tr_b8 a[126:127], v122 offset:10240\n"
      "v_mfma_f32_16x16x128_f8f6f4 v[80:83], a[104:111], a[56:63], v[80:83]\n"
      "ds_write_b128 v126, v[72:75] offset:4096\n"
      "v_mfma_f32_16x16x128_f8f6f4 v[112:115], a[104:111], a[64:71], v[112:115]\n"
      "ds_write_b128 v126, v[104:107] offset:6144\n"
      "s_waitcnt lgkmcnt(8)\n"
      "v_mfma_f32_16x16x128_f8f6f4 v[20:23], a[112:119], a[40:47], v[20:23]\n"
      "v_xor_b32 v123, 80, v125\n"
      "ds_read_b64_tr_b8 a[104:105], v123 offset:1024\n"
      "ds_read_b64_tr_b8 a[106:107], v123 offset:2048\n"
      "v_mfma_f32_16x16x128_f8f6f4 v[52:55], a[112:119], a[48:55], v[52:55]\n"
      "ds_read_b64_tr_b8 a[108:109], v123 offset:9216\n"
      "ds_read_b64_tr_b8 a[110:111], v123 offset:10240\n"
      "v_mfma_f32_16x16x128_f8f6f4 v[84:87], a[112:119], a[56:63], v[84:87]\n"
      "ds_write_b128 v127, v[12:15] offset:0\n"
      "v_mfma_f32_16x16x128_f8f6f4 v[116:119], a[112:119], a[64:71], v[116:119]\n"
      "ds_write_b128 v127, v[44:47] offset:2048\n"
      "s_waitcnt lgkmcnt(8)\n"
      "v_mfma_f32_16x16x128_f8f6f4 v[16:19], a[120:127], a[72:79], v[16:19]\n"
      "v_xor_b32 v122, 96, v121\n"
      "ds_read_b64_tr_b8 a[112:113], v122 offset:1024\n"
      "ds_read_b64_tr_b8 a[114:115], v122 offset:2048\n"
      "v_mfma_f32_16x16x128_f8f6f4 v[48:51], a[120:127], a[80:87], v[48:51]\n"
      "ds_read_b64_tr_b8 a[116:117], v122 offset:9216\n"
      "ds_read_b64_tr_b8 a[118:119], v122 offset:10240\n"
      "v_mfma_f32_16x16x128_f8f6f4 v[80:83], a[120:127], a[88:95], v[80:83]\n"
      "ds_write_b128 v127, v[76:79] offset:4096\n"
      "v_mfma_f32_16x16x128_f8f6f4 v[112:115], a[120:127], a[96:103], v[112:115]\n"
      "ds_write_b128 v127, v[108:111] offset:6144\n"
      "s_waitcnt lgkmcnt(8)\n"
      "v_mfma_f32_16x16x128_f8f6f4 v[20:23], a[104:111], a[72:79], v[20:23]\n"
      "v_xor_b32 v123, 112, v121\n"
      "ds_read_b64_tr_b8 a[120:121], v123 offset:1024\n"
      "ds_read_b64_tr_b8 a[122:123], v123 offset:2048\n"
      "v_mfma_f32_16x16x128_f8f6f4 v[52:55], a[104:111], a[80:87], v[52:55]\n"
      "ds_read_b64_tr_b8 a[124:125], v123 offset:9216\n"
      "ds_read_b64_tr_b8 a[126:127], v123 offset:10240\n"
      "v_mfma_f32_16x16x128_f8f6f4 v[84:87], a[104:111], a[88:95], v[84:87]\n"
      "ds_write_b32 v120, v121 offset:16\n"
      "v_mfma_f32_16x16x128_f8f6f4 v[116:119], a[104:111], a[96:103], v[116:119]\n"
      "s_waitcnt lgkmcnt(7)\n"
      "v_mfma_f32_16x16x128_f8f6f4 v[24:27], a[112:119], a[8:15], v[24:27]\n"
      "v_xor_b32 v122, 96, v124\n"
      "ds_read_b64_tr_b8 a[104:105], v122 offset:1024\n"
      "ds_read_b64_tr_b8 a[106:107], v122 offset:2048\n"
      "v_mfma_f32_16x16x128_f8f6f4 v[56:59], a[112:119], a[16:23], v[56:59]\n"
      "ds_read_b64_tr_b8 a[108:109], v122 offset:9216\n"
      "ds_read_b64_tr_b8 a[110:111], v122 offset:10240\n"
      "v_mfma_f32_16x16x128_f8f6f4 v[88:91], a[112:119], a[24:31], v[88:91]\n"
      "v_add_u32 v126, s60, v126\n"
      "v_mfma_f32_16x16x128_f8f6f4 a[0:3], a[112:119], a[32:39], a[0:3]\n"
      "v_add_u32 v127, s60, v127\n"
      "s_waitcnt lgkmcnt(5)\n"
      "v_mfma_f32_16x16x128_f8f6f4 v[28:31], a[120:127], a[8:15], v[28:31]\n"
      "v_xor_b32 v123, 112, v124\n"
      "ds_read_b64_tr_b8 a[112:113], v123 offset:1024\n"
      "ds_read_b64_tr_b8 a[114:115], v123 offset:2048\n"
      "v_mfma_f32_16x16x128_f8f6f4 v[60:63], a[120:127], a[16:23], v[60:63]\n"
      "ds_read_b64_tr_b8 a[116:117], v123 offset:9216\n"
      "ds_read_b64_tr_b8 a[118:119], v123 offset:10240\n"
      "v_mfma_f32_16x16x128_f8f6f4 v[92:95], a[120:127], a[24:31], v[92:95]\n"
      "ds_write_b128 v126, v[16:19] offset:0\n"
      "v_mfma_f32_16x16x128_f8f6f4 a[4:7], a[120:127], a[32:39], a[4:7]\n"
      "ds_write_b128 v126, v[48:51] offset:2048\n"
      "s_waitcnt lgkmcnt(6)\n"
      "v_mfma_f32_16x16x128_f8f6f4 v[24:27], a[104:111], a[40:47], v[24:27]\n"
      "v_xor_b32 v122, 96, v125\n"
      "ds_read_b64_tr_b8 a[120:121], v122 offset:1024\n"
      "ds_read_b64_tr_b8 a[122:123], v122 offset:2048\n"
      "v_mfma_f32_16x16x128_f8f6f4 v[56:59], a[104:111], a[48:55], v[56:59]\n"
      "ds_read_b64_tr_b8 a[124:125], v122 offset:9216\n"
      "ds_read_b64_tr_b8 a[126:127], v122 offset:10240\n"
      "v_mfma_f32_16x16x128_f8f6f4 v[88:91], a[104:111], a[56:63], v[88:91]\n"
      "ds_write_b128 v126, v[80:83] offset:4096\n"
      "v_mfma_f32_16x16x128_f8f6f4 a[0:3], a[104:111], a[64:71], a[0:3]\n"
      "ds_write_b128 v126, v[112:115] offset:6144\n"
      "s_waitcnt lgkmcnt(8)\n"
      "v_mfma_f32_16x16x128_f8f6f4 v[28:31], a[112:119], a[40:47], v[28:31]\n"
      "v_xor_b32 v123, 112, v125\n"
      "ds_read_b64_tr_b8 a[104:105], v123 offset:1024\n"
      "ds_read_b64_tr_b8 a[106:107], v123 offset:2048\n"
      "v_mfma_f32_16x16x128_f8f6f4 v[60:63], a[112:119], a[48:55], v[60:63]\n"
      "ds_read_b64_tr_b8 a[108:109], v123 offset:9216\n"
      "ds_read_b64_tr_b8 a[110:111], v123 offset:10240\n"
      "v_mfma_f32_16x16x128_f8f6f4 v[92:95], a[112:119], a[56:63], v[92:95]\n"
      "ds_write_b128 v127, v[20:23] offset:0\n"
      "v_mfma_f32_16x16x128_f8f6f4 a[4:7], a[112:119], a[64:71], a[4:7]\n"
      "ds_write_b128 v127, v[52:55] offset:2048\n"
      "s_waitcnt lgkmcnt(8)\n"
      "v_mfma_f32_16x16x128_f8f6f4 v[24:27], a[120:127], a[72:79], v[24:27]\n"
      "v_mfma_f32_16x16x128_f8f6f4 v[56:59], a[120:127], a[80:87], v[56:59]\n"
      "v_mfma_f32_16x16x128_f8f6f4 v[88:91], a[120:127], a[88:95], v[88:91]\n"
      "ds_write_b128 v127, v[84:87] offset:4096\n"
      "v_mfma_f32_16x16x128_f8f6f4 a[0:3], a[120:127], a[96:103], a[0:3]\n"
      "ds_write_b128 v127, v[116:119] offset:6144\n"
      "s_waitcnt lgkmcnt(4)\n"
      "v_mfma_f32_16x16x128_f8f6f4 v[28:31], a[104:111], a[72:79], v[28:31]\n"
      "v_mfma_f32_16x16x128_f8f6f4 v[60:63], a[104:111], a[80:87], v[60:63]\n"
      "v_mfma_f32_16x16x128_f8f6f4 v[92:95], a[104:111], a[88:95], v[92:95]\n"
      "ds_write_b32 v120, v121 offset:32\n"
      "v_mfma_f32_16x16x128_f8f6f4 a[4:7], a[104:111], a[96:103], a[4:7]\n"
      "v_mov_b32 v122, s80\n"
      "v_mul_f32 v122, s81, v122\n"
      "v_mbcnt_lo_u32_b32 v123, -1, 0\n"
      "v_mbcnt_hi_u32_b32 v123, -1, v123\n"
      "v_readfirstlane_b32 s82, v122\n"
      "v_and_b32 v120, 15, v123\n"
      "s_mov_b32 s83, s82\n"
      "v_lshlrev_b32 v120, 16, v120\n"
      "v_lshrrev_b32 v123, 4, v123\n"
      "v_lshl_add_u32 v120, v123, 4, v120\n"
      "s_nop 7\n"
      "s_nop 7\n"
      "v_pk_mul_f32 v[24:25], v[24:25], s[82:83]\n"
      "v_pk_mul_f32 v[26:27], v[26:27], s[82:83]\n"
      "v_max_f32 v24, 0, v24\n"
      "v_max_f32 v25, 0, v25\n"
      "v_max_f32 v26, 0, v26\n"
      "v_max_f32 v27, 0, v27\n"
      "buffer_store_dwordx4 v[24:27], v120, s[72:75], s76 offen offset:384 sc0 sc1\n"
      "v_pk_mul_f32 v[56:57], v[56:57], s[82:83]\n"
      "v_pk_mul_f32 v[58:59], v[58:59], s[82:83]\n"
      "v_max_f32 v56, 0, v56\n"
      "v_max_f32 v57, 0, v57\n"
      "v_max_f32 v58, 0, v58\n"
      "v_max_f32 v59, 0, v59\n"
      "buffer_store_dwordx4 v[56:59], v120, s[72:75], s77 offen offset:384 sc0 sc1\n"
      "v_pk_mul_f32 v[88:89], v[88:89], s[82:83]\n"
      "v_pk_mul_f32 v[90:91], v[90:91], s[82:83]\n"
      "v_max_f32 v88, 0, v88\n"
      "v_max_f32 v89, 0, v89\n"
      "v_max_f32 v90, 0, v90\n"
      "v_max_f32 v91, 0, v91\n"
      "buffer_store_dwordx4 v[88:91], v120, s[72:75], s78 offen offset:384 sc0 sc1\n"
      "v_accvgpr_read_b32 v0, a0\n"
      "v_accvgpr_read_b32 v1, a1\n"
      "v_accvgpr_read_b32 v2, a2\n"
      "v_accvgpr_read_b32 v3, a3\n"
      "v_pk_mul_f32 v[0:1], v[0:1], s[82:83]\n"
      "v_pk_mul_f32 v[2:3], v[2:3], s[82:83]\n"
      "v_max_f32 v0, 0, v0\n"
      "v_max_f32 v1, 0, v1\n"
      "v_max_f32 v2, 0, v2\n"
      "v_max_f32 v3, 0, v3\n"
      "buffer_store_dwordx4 v[0:3], v120, s[72:75], s79 offen offset:384 sc0 sc1\n"
      "v_pk_mul_f32 v[28:29], v[28:29], s[82:83]\n"
      "v_pk_mul_f32 v[30:31], v[30:31], s[82:83]\n"
      "v_max_f32 v28, 0, v28\n"
      "v_max_f32 v29, 0, v29\n"
      "v_max_f32 v30, 0, v30\n"
      "v_max_f32 v31, 0, v31\n"
      "buffer_store_dwordx4 v[28:31], v120, s[72:75], s76 offen offset:448 sc0 sc1\n"
      "v_pk_mul_f32 v[60:61], v[60:61], s[82:83]\n"
      "v_pk_mul_f32 v[62:63], v[62:63], s[82:83]\n"
      "v_max_f32 v60, 0, v60\n"
      "v_max_f32 v61, 0, v61\n"
      "v_max_f32 v62, 0, v62\n"
      "v_max_f32 v63, 0, v63\n"
      "buffer_store_dwordx4 v[60:63], v120, s[72:75], s77 offen offset:448 sc0 sc1\n"
      "v_pk_mul_f32 v[92:93], v[92:93], s[82:83]\n"
      "v_pk_mul_f32 v[94:95], v[94:95], s[82:83]\n"
      "v_max_f32 v92, 0, v92\n"
      "v_max_f32 v93, 0, v93\n"
      "v_max_f32 v94, 0, v94\n"
      "v_max_f32 v95, 0, v95\n"
      "buffer_store_dwordx4 v[92:95], v120, s[72:75], s78 offen offset:448 sc0 sc1\n"
      "v_accvgpr_read_b32 v4, a4\n"
      "v_accvgpr_read_b32 v5, a5\n"
      "v_accvgpr_read_b32 v6, a6\n"
      "v_accvgpr_read_b32 v7, a7\n"
      "v_pk_mul_f32 v[4:5], v[4:5], s[82:83]\n"
      "v_pk_mul_f32 v[6:7], v[6:7], s[82:83]\n"
      "v_max_f32 v4, 0, v4\n"
      "v_max_f32 v5, 0, v5\n"
      "v_max_f32 v6, 0, v6\n"
      "v_max_f32 v7, 0, v7\n"
      "buffer_store_dwordx4 v[4:7], v120, s[72:75], s79 offen offset:448 sc0 sc1\n"
      "s_branch L_end_%=\n"
      "L_nanpre_%=:\n"
      "v_mov_b32 v122, s80\n"
      "v_mul_f32 v122, s81, v122\n"
      "v_mbcnt_lo_u32_b32 v123, -1, 0\n"
      "v_mbcnt_hi_u32_b32 v123, -1, v123\n"
      "v_readfirstlane_b32 s82, v122\n"
      "v_and_b32 v120, 15, v123\n"
      "s_mov_b32 s83, s82\n"
      "v_lshlrev_b32 v120, 16, v120\n"
      "v_lshrrev_b32 v123, 4, v123\n"
      "v_lshl_add_u32 v120, v123, 4, v120\n"
      "L_nan_%=:\n"
      "v_mov_b32 v122, 0\n"
      "v_mov_b32 v123, 1\n"
      "ds_write_b32 v122, v123 offset:40\n"
      "s_mov_b32 s79, %[ooff]\n"
      "s_nop 7\n"
      "s_nop 7\n"
      "v_mov_b32 v0, 0x7fc00000\n"
      "v_mov_b32 v1, 0x7fc00000\n"
      "v_mov_b32 v2, 0x7fc00000\n"
      "v_mov_b32 v3, 0x7fc00000\n"
      "s_mov_b32 s86, 0\n"
      "L_nanl_%=:\n"
      "buffer_store_dwordx4 v[0:3], v120, s[72:75], s79 offen offset:0\n"
      "buffer_store_dwordx4 v[0:3], v120, s[72:75], s79 offen offset:64\n"
      "buffer_store_dwordx4 v[0:3], v120, s[72:75], s79 offen offset:128\n"
      "buffer_store_dwordx4 v[0:3], v120, s[72:75], s79 offen offset:192\n"
      "buffer_store_dwordx4 v[0:3], v120, s[72:75], s79 offen offset:256\n"
      "buffer_store_dwordx4 v[0:3], v120, s[72:75], s79 offen offset:320\n"
      "buffer_store_dwordx4 v[0:3], v120, s[72:75], s79 offen offset:384\n"
      "buffer_store_dwordx4 v[0:3], v120, s[72:75], s79 offen offset:448\n"
      "s_add_u32 s79, s79, 0x100000\n"
      "s_add_u32 s86, s86, 1\n"
      "s_cmp_lt_u32 s86, 4\n"
      "s_cbranch_scc1 L_nanl_%=\n"
      "s_branch L_end_%=\n"
      "L_SK0_%=:\n"
      "s_mov_b32 s63, s50\n"
      "s_mov_b32 s62, 1\n"
      "s_waitcnt lgkmcnt(4)\n"
      "v_mfma_f32_16x16x128_f8f6f4 v[0:3], a[112:119], a[8:15], v[0:3]\n"
      "v_xor_b32 v122, 32, v121\n"
      "ds_read_b64_tr_b8 a[104:105], v122 offset:1024\n"
      "ds_read_b64_tr_b8 a[106:107], v122 offset:2048\n"
      "v_mfma_f32_16x16x128_f8f6f4 v[32:35], a[112:119], a[16:23], v[32:35]\n"
      "ds_read_b64_tr_b8 a[108:109], v122 offset:9216\n"
      "ds_read_b64_tr_b8 a[110:111], v122 offset:10240\n"
      "v_mfma_f32_16x16x128_f8f6f4 v[64:67], a[112:119], a[24:31], v[64:67]\n"
      "v_mfma_f32_16x16x128_f8f6f4 v[96:99], a[112:119], a[32:39], v[96:99]\n"
      "s_waitcnt lgkmcnt(4)\n"
      "v_mfma_f32_16x16x128_f8f6f4 v[4:7], a[120:127], a[8:15], v[4:7]\n"
      "v_xor_b32 v123, 48, v121\n"
      "ds_read_b64_tr_b8 a[112:113], v123 offset:1024\n"
      "ds_read_b64_tr_b8 a[114:115], v123 offset:2048\n"
      "v_mfma_f32_16x16x128_f8f6f4 v[36:39], a[120:127], a[16:23], v[36:39]\n"
      "ds_read_b64_tr_b8 a[116:117], v123 offset:9216\n"
      "ds_read_b64_tr_b8 a[118:119], v123 offset:10240\n"
      "v_mfma_f32_16x16x128_f8f6f4 v[68:71], a[120:127], a[24:31], v[68:71]\n"
      "v_mfma_f32_16x16x128_f8f6f4 v[100:103], a[120:127], a[32:39], v[100:103]\n"
      "s_waitcnt lgkmcnt(4)\n"
      "v_mfma_f32_16x16x128_f8f6f4 v[8:11], a[104:111], a[8:15], v[8:11]\n"
      "v_xor_b32 v122, 64, v121\n"
      "ds_read_b64_tr_b8 a[120:121], v122 offset:1024\n"
      "ds_read_b64_tr_b8 a[122:123], v122 offset:2048\n"
      "v_mfma_f32_16x16x128_f8f6f4 v[40:43], a[104:111], a[16:23], v[40:43]\n"
      "ds_read_b64_tr_b8 a[124:125], v122 offset:9216\n"
      "ds_read_b64_tr_b8 a[126:127], v122 offset:10240\n"
      "v_mfma_f32_16x16x128_f8f6f4 v[72:75], a[104:111], a[24:31], v[72:75]\n"
      "v_mfma_f32_16x16x128_f8f6f4 v[104:107], a[104:111], a[32:39], v[104:107]\n"
      "ds_write_b32 v124, v125 offset:16\n"
      "s_waitcnt lgkmcnt(5)\n"
      "v_mfma_f32_16x16x128_f8f6f4 v[12:15], a[112:119], a[8:15], v[12:15]\n"
      "v_xor_b32 v123, 80, v121\n"
      "ds_read_b64_tr_b8 a[104:105], v123 offset:1024\n"
      "ds_read_b64_tr_b8 a[106:107], v123 offset:2048\n"
      "v_mfma_f32_16x16x128_f8f6f4 v[44:47], a[112:119], a[16:23], v[44:47]\n"
      "ds_read_b64_tr_b8 a[108:109], v123 offset:9216\n"
      "ds_read_b64_tr_b8 a[110:111], v123 offset:10240\n"
      "v_mfma_f32_16x16x128_f8f6f4 v[76:79], a[112:119], a[24:31], v[76:79]\n"
      "v_mfma_f32_16x16x128_f8f6f4 v[108:111], a[112:119], a[32:39], v[108:111]\n"
      "s_waitcnt lgkmcnt(5)\n"
      "v_mfma_f32_16x16x128_f8f6f4 v[16:19], a[120:127], a[8:15], v[16:19]\n"
      "ds_read_b32 v126, v124 offset:0\n"
      "ds_read_b32 v127, v125 offset:76\n"
      "v_xor_b32 v122, 96, v121\n"
      "ds_read_b64_tr_b8 a[112:113], v122 offset:1024\n"
      "ds_read_b64_tr_b8 a[114:115], v122 offset:2048\n"
      "v_mfma_f32_16x16x128_f8f6f4 v[48:51], a[120:127], a[16:23], v[48:51]\n"
      "ds_read_b64_tr_b8 a[116:117], v122 offset:9216\n"
      "ds_read_b64_tr_b8 a[118:119], v122 offset:10240\n"
      "v_mfma_f32_16x16x128_f8f6f4 v[80:83], a[120:127], a[24:31], v[80:83]\n"
      "v_mfma_f32_16x16x128_f8f6f4 v[112:115], a[120:127], a[32:39], v[112:115]\n"
      "s_waitcnt lgkmcnt(6)\n"
      "v_mfma_f32_16x16x128_f8f6f4 v[20:23], a[104:111], a[8:15], v[20:23]\n"
      "v_xor_b32 v123, 112, v121\n"
      "ds_read_b64_tr_b8 a[120:121], v123 offset:1024\n"
      "ds_read_b64_tr_b8 a[122:123], v123 offset:2048\n"
      "v_mfma_f32_16x16x128_f8f6f4 v[52:55], a[104:111], a[16:23], v[52:55]\n"
      "ds_read_b64_tr_b8 a[124:125], v123 offset:9216\n"
      "ds_read_b64_tr_b8 a[126:127], v123 offset:10240\n"
      "v_mfma_f32_16x16x128_f8f6f4 v[84:87], a[104:111], a[24:31], v[84:87]\n"
      "v_add_u32 v125, 4, v125\n"
      "v_mfma_f32_16x16x128_f8f6f4 v[116:119], a[104:111], a[32:39], v[116:119]\n"
      "s_waitcnt lgkmcnt(4)\n"
      "v_readfirstlane_b32 s84, v126\n"
      "v_add_u32 v121, 0x4000, v121\n"
      "s_cmp_lt_u32 s84, s85\n"
      "s_cbranch_scc1 L_slowFK0_%=\n"
      "L_backFK0_%=:\n"
      "v_and_b32 v121, 0x1ffff, v121\n"
      "v_mfma_f32_16x16x128_f8f6f4 v[24:27], a[112:119], a[8:15], v[24:27]\n"
      "ds_read_b32 v126, v124 offset:48\n"
      "ds_read_b64_tr_b8 a[104:105], v121 offset:1024\n"
      "ds_read_b64_tr_b8 a[106:107], v121 offset:2048\n"
      "v_mfma_f32_16x16x128_f8f6f4 v[56:59], a[112:119], a[16:23], v[56:59]\n"
      "ds_read_b64_tr_b8 a[108:109], v121 offset:9216\n"
      "ds_read_b64_tr_b8 a[110:111], v121 offset:10240\n"
      "v_mfma_f32_16x16x128_f8f6f4 v[88:91], a[112:119], a[24:31], v[88:91]\n"
      "v_mfma_f32_16x16x128_f8f6f4 a[0:3], a[112:119], a[32:39], a[0:3]\n"
      "s_waitcnt lgkmcnt(5)\n"
      "v_mfma_f32_16x16x128_f8f6f4 v[28:31], a[120:127], a[8:15], v[28:31]\n"
      "v_xor_b32 v123, 16, v121\n"
      "ds_read_b64_tr_b8 a[112:113], v123 offset:1024\n"
      "ds_read_b64_tr_b8 a[114:115], v123 offset:2048\n"
      "v_mfma_f32_16x16x128_f8f6f4 v[60:63], a[120:127], a[16:23], v[60:63]\n"
      "ds_read_b64_tr_b8 a[116:117], v123 offset:9216\n"
      "ds_read_b64_tr_b8 a[118:119], v123 offset:10240\n"
      "s_waitcnt lgkmcnt(8)\n"
      "v_readfirstlane_b32 s50, v127\n"
      "v_readfirstlane_b32 s58, v126\n"
      "v_mfma_f32_16x16x128_f8f6f4 v[92:95], a[120:127], a[24:31], v[92:95]\n"
      "s_cmp_eq_u32 s50, 0\n"
      "s_cselect_b32 s53, 1, s58\n"
      "s_add_u32 s51, s50, -1\n"
      "s_add_u32 s52, s51, 4096\n"
      "v_mfma_f32_16x16x128_f8f6f4 a[4:7], a[120:127], a[32:39], a[4:7]\n"
      "s_add_u32 s85, s85, 1\n"
      "s_branch L_D1_%=\n"
      "L_ON0_%=:\n"
      "s_waitcnt vmcnt(0)\n"
      "s_mov_b32 s79, s50\n"
      "s_cmp_eq_u32 s63, 0\n"
      "s_cbranch_scc1 L_slowO0_%=\n"
      "L_backO0_%=:\n"
      "s_add_u32 s63, s63, -1\n"
      "s_add_u32 s59, s63, 4096\n"
      "s_cmp_eq_u32 s79, 0\n"
      "s_cbranch_scc1 L_OS0_%=\n"
      "L_backA0_%=:\n"
      "s_add_u32 s79, s79, -1\n"
      "s_add_u32 s78, s79, 4096\n"
      "s_mov_b32 s62, 0\n"
      "s_cmp_lg_u32 s58, 0\n"
      "s_cbranch_scc1 L_OL0_%=\n"
      "s_waitcnt lgkmcnt(4)\n"
      "v_mfma_f32_16x16x128_f8f6f4 v[0:3], a[112:119], a[8:15], v[0:3]\n"
      "v_xor_b32 v122, 32, v121\n"
      "ds_read_b64_tr_b8 a[104:105], v122 offset:1024\n"
      "ds_read_b64_tr_b8 a[106:107], v122 offset:2048\n"
      "buffer_load_dwordx4 a[40:43], v120, s[68:71], s63 offen offset:0 sc0 sc1\n"
      "v_mfma_f32_16x16x128_f8f6f4 v[32:35], a[112:119], a[16:23], v[32:35]\n"
      "ds_read_b64_tr_b8 a[108:109], v122 offset:9216\n"
      "ds_read_b64_tr_b8 a[110:111], v122 offset:10240\n"
      "buffer_load_dwordx4 a[44:47], v120, s[68:71], s63 offen offset:1024 sc0 sc1\n"
      "v_mfma_f32_16x16x128_f8f6f4 v[64:67], a[112:119], a[24:31], v[64:67]\n"
      "buffer_load_dwordx4 a[48:51], v120, s[68:71], s63 offen offset:2048 sc0 sc1\n"
      "v_mfma_f32_16x16x128_f8f6f4 v[96:99], a[112:119], a[32:39], v[96:99]\n"
      "buffer_load_dwordx4 a[52:55], v120, s[68:71], s63 offen offset:3072 sc0 sc1\n"
      "s_waitcnt lgkmcnt(4)\n"
      "v_mfma_f32_16x16x128_f8f6f4 v[4:7], a[120:127], a[8:15], v[4:7]\n"
      "v_xor_b32 v123, 48, v121\n"
      "ds_read_b64_tr_b8 a[112:113], v123 offset:1024\n"
      "ds_read_b64_tr_b8 a[114:115], v123 offset:2048\n"
      "buffer_load_dwordx4 a[56:59], v120, s[68:71], s59 offen offset:0 sc0 sc1\n"
      "v_mfma_f32_16x16x128_f8f6f4 v[36:39], a[120:127], a[16:23], v[36:39]\n"
      "ds_read_b64_tr_b8 a[116:117], v123 offset:9216\n"
      "ds_read_b64_tr_b8 a[118:119], v123 offset:10240\n"
      "buffer_load_dwordx4 a[60:63], v120, s[68:71], s59 offen offset:1024 sc0 sc1\n"
      "v_mfma_f32_16x16x128_f8f6f4 v[68:71], a[120:127], a[24:31], v[68:71]\n"
      "buffer_load_dwordx4 a[64:67], v120, s[68:71], s59 offen offset:2048 sc0 sc1\n"
      "v_mfma_f32_16x16x128_f8f6f4 v[100:103], a[120:127], a[32:39], v[100:103]\n"
      "buffer_load_dwordx4 a[68:71], v120, s[68:71], s59 offen offset:3072 sc0 sc1\n"
      "s_waitcnt lgkmcnt(4)\n"
      "v_mfma_f32_16x16x128_f8f6f4 v[8:11], a[104:111], a[8:15], v[8:11]\n"
      "v_xor_b32 v122, 64, v121\n"
      "ds_read_b64_tr_b8 a[120:121], v122 offset:1024\n"
      "ds_read_b64_tr_b8 a[122:123], v122 offset:2048\n"
      "v_mfma_f32_16x16x128_f8f6f4 v[40:43], a[104:111], a[16:23], v[40:43]\n"
      "ds_read_b64_tr_b8 a[124:125], v122 offset:9216\n"
      "ds_read_b64_tr_b8 a[126:127], v122 offset:10240\n"
      "v_mfma_f32_16x16x128_f8f6f4 v[72:75], a[104:111], a[24:31], v[72:75]\n"
      "v_mfma_f32_16x16x128_f8f6f4 v[104:107], a[104:111], a[32:39], v[104:107]\n"
      "ds_write_b32 v124, v125 offset:16\n"
      "s_waitcnt lgkmcnt(5)\n"
      "v_mfma_f32_16x16x128_f8f6f4 v[12:15], a[112:119], a[8:15], v[12:15]\n"
      "v_xor_b32 v123, 80, v121\n"
      "ds_read_b64_tr_b8 a[104:105], v123 offset:1024\n"
      "ds_read_b64_tr_b8 a[106:107], v123 offset:2048\n"
      "v_mfma_f32_16x16x128_f8f6f4 v[44:47], a[112:119], a[16:23], v[44:47]\n"
      "ds_read_b64_tr_b8 a[108:109], v123 offset:9216\n"
      "ds_read_b64_tr_b8 a[110:111], v123 offset:10240\n"
      "v_mfma_f32_16x16x128_f8f6f4 v[76:79], a[112:119], a[24:31], v[76:79]\n"
      "v_mfma_f32_16x16x128_f8f6f4 v[108:111], a[112:119], a[32:39], v[108:111]\n"
      "s_waitcnt lgkmcnt(5)\n"
      "v_mfma_f32_16x16x128_f8f6f4 v[16:19], a[120:127], a[8:15], v[16:19]\n"
      "ds_read_b32 v126, v124 offset:0\n"
      "ds_read_b32 v127, v125 offset:76\n"
      "v_xor_b32 v122, 96, v121\n"
      "ds_read_b64_tr_b8 a[112:113], v122 offset:1024\n"
      "ds_read_b64_tr_b8 a[114:115], v122 offset:2048\n"
      "buffer_load_dwordx4 a[72:75], v120, s[68:71], s79 offen offset:0 sc0 sc1\n"
      "v_mfma_f32_16x16x128_f8f6f4 v[48:51], a[120:127], a[16:23], v[48:51]\n"
      "ds_read_b64_tr_b8 a[116:117], v122 offset:9216\n"
      "ds_read_b64_tr_b8 a[118:119], v122 offset:10240\n"
      "buffer_load_dwordx4 a[76:79], v120, s[68:71], s79 offen offset:1024 sc0 sc1\n"
      "v_mfma_f32_16x16x128_f8f6f4 v[80:83], a[120:127], a[24:31], v[80:83]\n"
      "buffer_load_dwordx4 a[80:83], v120, s[68:71], s79 offen offset:2048 sc0 sc1\n"
      "v_mfma_f32_16x16x128_f8f6f4 v[112:115], a[120:127], a[32:39], v[112:115]\n"
      "buffer_load_dwordx4 a[84:87], v120, s[68:71], s79 offen offset:3072 sc0 sc1\n"
      "s_waitcnt lgkmcnt(6)\n"
      "v_mfma_f32_16x16x128_f8f6f4 v[20:23], a[104:111], a[8:15], v[20:23]\n"
      "v_xor_b32 v123, 112, v121\n"
      "ds_read_b64_tr_b8 a[120:121], v123 offset:1024\n"
      "ds_read_b64_tr_b8 a[122:123], v123 offset:2048\n"
      "buffer_load_dwordx4 a[88:91], v120, s[68:71], s78 offen offset:0 sc0 sc1\n"
      "v_mfma_f32_16x16x128_f8f6f4 v[52:55], a[104:111], a[16:23], v[52:55]\n"
      "ds_read_b64_tr_b8 a[124:125], v123 offset:9216\n"
      "ds_read_b64_tr_b8 a[126:127], v123 offset:10240\n"
      "buffer_load_dwordx4 a[92:95], v120, s[68:71], s78 offen offset:1024 sc0 sc1\n"
      "v_mfma_f32_16x16x128_f8f6f4 v[84:87], a[104:111], a[24:31], v[84:87]\n"
      "v_add_u32 v125, 4, v125\n"
      "buffer_load_dwordx4 a[96:99], v120, s[68:71], s78 offen offset:2048 sc0 sc1\n"
      "v_mfma_f32_16x16x128_f8f6f4 v[116:119], a[104:111], a[32:39], v[116:119]\n"
      "buffer_load_dwordx4 a[100:103], v120, s[68:71], s78 offen offset:3072 sc0 sc1\n"
      "s_waitcnt lgkmcnt(4)\n"
      "v_readfirstlane_b32 s84, v126\n"
      "v_add_u32 v121, 0x4000, v121\n"
      "s_cmp_lt_u32 s84, s85\n"
      "s_cbranch_scc1 L_slowFO0_%=\n"
      "L_backFO0_%=:\n"
      "v_and_b32 v121, 0x1ffff, v121\n"
      "v_mfma_f32_16x16x128_f8f6f4 v[24:27], a[112:119], a[8:15], v[24:27]\n"
      "ds_read_b32 v126, v124 offset:48\n"
      "ds_read_b64_tr_b8 a[104:105], v121 offset:1024\n"
      "ds_read_b64_tr_b8 a[106:107], v121 offset:2048\n"
      "v_mfma_f32_16x16x128_f8f6f4 v[56:59], a[112:119], a[16:23], v[56:59]\n"
      "ds_read_b64_tr_b8 a[108:109], v121 offset:9216\n"
      "ds_read_b64_tr_b8 a[110:111], v121 offset:10240\n"
      "v_mfma_f32_16x16x128_f8f6f4 v[88:91], a[112:119], a[24:31], v[88:91]\n"
      "v_mfma_f32_16x16x128_f8f6f4 a[0:3], a[112:119], a[32:39], a[0:3]\n"
      "s_waitcnt lgkmcnt(5)\n"
      "v_mfma_f32_16x16x128_f8f6f4 v[28:31], a[120:127], a[8:15], v[28:31]\n"
      "v_xor_b32 v123, 16, v121\n"
      "ds_read_b64_tr_b8 a[112:113], v123 offset:1024\n"
      "ds_read_b64_tr_b8 a[114:115], v123 offset:2048\n"
      "v_mfma_f32_16x16x128_f8f6f4 v[60:63], a[120:127], a[16:23], v[60:63]\n"
      "ds_read_b64_tr_b8 a[116:117], v123 offset:9216\n"
      "ds_read_b64_tr_b8 a[118:119], v123 offset:10240\n"
      "s_waitcnt lgkmcnt(8)\n"
      "v_readfirstlane_b32 s50, v127\n"
      "v_readfirstlane_b32 s58, v126\n"
      "v_mfma_f32_16x16x128_f8f6f4 v[92:95], a[120:127], a[24:31], v[92:95]\n"
      "s_cmp_eq_u32 s50, 0\n"
      "s_cselect_b32 s53, 1, s58\n"
      "s_add_u32 s51, s50, -1\n"
      "s_add_u32 s52, s51, 4096\n"
      "v_mfma_f32_16x16x128_f8f6f4 a[4:7], a[120:127], a[32:39], a[4:7]\n"
      "s_add_u32 s85, s85, 1\n"
      "s_branch L_D1_%=\n"
      "L_OL0_%=:\n"
      "s_waitcnt lgkmcnt(4)\n"
      "v_mfma_f32_16x16x128_f8f6f4 v[0:3], a[112:119], a[8:15], v[0:3]\n"
      "v_xor_b32 v122, 32, v121\n"
      "ds_read_b64_tr_b8 a[104:105], v122 offset:1024\n"
      "ds_read_b64_tr_b8 a[106:107], v122 offset:2048\n"
      "v_mfma_f32_16x16x128_f8f6f4 v[32:35], a[112:119], a[16:23], v[32:35]\n"
      "ds_read_b64_tr_b8 a[108:109], v122 offset:9216\n"
      "ds_read_b64_tr_b8 a[110:111], v122 offset:10240\n"
      "v_mfma_f32_16x16x128_f8f6f4 v[64:67], a[112:119], a[24:31], v[64:67]\n"
      "v_mfma_f32_16x16x128_f8f6f4 v[96:99], a[112:119], a[32:39], v[96:99]\n"
      "s_waitcnt lgkmcnt(4)\n"
      "v_mfma_f32_16x16x128_f8f6f4 v[4:7], a[120:127], a[8:15], v[4:7]\n"
      "v_xor_b32 v123, 48, v121\n"
      "ds_read_b64_tr_b8 a[112:113], v123 offset:1024\n"
      "ds_read_b64_tr_b8 a[114:115], v123 offset:2048\n"
      "v_mfma_f32_16x16x128_f8f6f4 v[36:39], a[120:127], a[16:23], v[36:39]\n"
      "ds_read_b64_tr_b8 a[116:117], v123 offset:9216\n"
      "ds_read_b64_tr_b8 a[118:119], v123 offset:10240\n"
      "v_mfma_f32_16x16x128_f8f6f4 v[68:71], a[120:127], a[24:31], v[68:71]\n"
      "v_mfma_f32_16x16x128_f8f6f4 v[100:103], a[120:127], a[32:39], v[100:103]\n"
      "s_waitcnt lgkmcnt(4)\n"
      "v_mfma_f32_16x16x128_f8f6f4 v[8:11], a[104:111], a[8:15], v[8:11]\n"
      "v_xor_b32 v122, 64, v121\n"
      "ds_read_b64_tr_b8 a[120:121], v122 offset:1024\n"
      "ds_read_b64_tr_b8 a[122:123], v122 offset:2048\n"
      "v_mfma_f32_16x16x128_f8f6f4 v[40:43], a[104:111], a[16:23], v[40:43]\n"
      "ds_read_b64_tr_b8 a[124:125], v122 offset:9216\n"
      "ds_read_b64_tr_b8 a[126:127], v122 offset:10240\n"
      "v_mfma_f32_16x16x128_f8f6f4 v[72:75], a[104:111], a[24:31], v[72:75]\n"
      "v_mfma_f32_16x16x128_f8f6f4 v[104:107], a[104:111], a[32:39], v[104:107]\n"
      "ds_write_b32 v124, v125 offset:16\n"
      "s_waitcnt lgkmcnt(5)\n"
      "v_mfma_f32_16x16x128_f8f6f4 v[12:15], a[112:119], a[8:15], v[12:15]\n"
      "v_xor_b32 v123, 80, v121\n"
      "ds_read_b64_tr_b8 a[104:105], v123 offset:1024\n"
      "ds_read_b64_tr_b8 a[106:107], v123 offset:2048\n"
      "v_mfma_f32_16x16x128_f8f6f4 v[44:47], a[112:119], a[16:23], v[44:47]\n"
      "ds_read_b64_tr_b8 a[108:109], v123 offset:9216\n"
      "ds_read_b64_tr_b8 a[110:111], v123 offset:10240\n"
      "v_mfma_f32_16x16x128_f8f6f4 v[76:79], a[112:119], a[24:31], v[76:79]\n"
      "v_mfma_f32_16x16x128_f8f6f4 v[108:111], a[112:119], a[32:39], v[108:111]\n"
      "s_waitcnt lgkmcnt(5)\n"
      "v_mfma_f32_16x16x128_f8f6f4 v[16:19], a[120:127], a[8:15], v[16:19]\n"
      "ds_read_b32 v126, v124 offset:0\n"
      "ds_read_b32 v127, v125 offset:76\n"
      "v_xor_b32 v122, 96, v121\n"
      "ds_read_b64_tr_b8 a[112:113], v122 offset:1024\n"
      "ds_read_b64_tr_b8 a[114:115], v122 offset:2048\n"
      "buffer_load_dwordx4 a[40:43], v120, s[68:71], s63 offen offset:0 sc0 sc1\n"
      "v_mfma_f32_16x16x128_f8f6f4 v[48:51], a[120:127], a[16:23], v[48:51]\n"
      "ds_read_b64_tr_b8 a[116:117], v122 offset:9216\n"
      "ds_read_b64_tr_b8 a[118:119], v122 offset:10240\n"
      "buffer_load_dwordx4 a[44:47], v120, s[68:71], s63 offen offset:1024 sc0 sc1\n"
      "v_mfma_f32_16x16x128_f8f6f4 v[80:83], a[120:127], a[24:31], v[80:83]\n"
      "buffer_load_dwordx4 a[48:51], v120, s[68:71], s63 offen offset:2048 sc0 sc1\n"
      "v_mfma_f32_16x16x128_f8f6f4 v[112:115], a[120:127], a[32:39], v[112:115]\n"
      "buffer_load_dwordx4 a[52:55], v120, s[68:71], s63 offen offset:3072 sc0 sc1\n"
      "s_waitcnt lgkmcnt(6)\n"
      "v_mfma_f32_16x16x128_f8f6f4 v[20:23], a[104:111], a[8:15], v[20:23]\n"
      "v_xor_b32 v123, 112, v121\n"
      "ds_read_b64_tr_b8 a[120:121], v123 offset:1024\n"
      "ds_read_b64_tr_b8 a[122:123], v123 offset:2048\n"
      "buffer_load_dwordx4 a[56:59], v120, s[68:71], s59 offen offset:0 sc0 sc1\n"
      "v_mfma_f32_16x16x128_f8f6f4 v[52:55], a[104:111], a[16:23], v[52:55]\n"
      "ds_read_b64_tr_b8 a[124:125], v123 offset:9216\n"
      "ds_read_b64_tr_b8 a[126:127], v123 offset:10240\n"
      "buffer_load_dwordx4 a[60:63], v120, s[68:71], s59 offen offset:1024 sc0 sc1\n"
      "v_mfma_f32_16x16x128_f8f6f4 v[84:87], a[104:111], a[24:31], v[84:87]\n"
      "v_add_u32 v125, 4, v125\n"
      "buffer_load_dwordx4 a[64:67], v120, s[68:71], s59 offen offset:2048 sc0 sc1\n"
      "v_mfma_f32_16x16x128_f8f6f4 v[116:119], a[104:111], a[32:39], v[116:119]\n"
      "buffer_load_dwordx4 a[68:71], v120, s[68:71], s59 offen offset:3072 sc0 sc1\n"
      "s_waitcnt lgkmcnt(4)\n"
      "v_readfirstlane_b32 s84, v126\n"
      "v_add_u32 v121, 0x4000, v121\n"
      "s_cmp_lt_u32 s84, s85\n"
      "s_cbranch_scc1 L_slowFL0_%=\n"
      "L_backFL0_%=:\n"
      "v_and_b32 v121, 0x1ffff, v121\n"
      "v_mfma_f32_16x16x128_f8f6f4 v[24:27], a[112:119], a[8:15], v[24:27]\n"
      "ds_read_b32 v126, v124 offset:48\n"
      "ds_read_b64_tr_b8 a[104:105], v121 offset:1024\n"
      "ds_read_b64_tr_b8 a[106:107], v121 offset:2048\n"
      "buffer_load_dwordx4 a[72:75], v120, s[68:71], s79 offen offset:0 sc0 sc1\n"
      "v_mfma_f32_16x16x128_f8f6f4 v[56:59], a[112:119], a[16:23], v[56:59]\n"
      "ds_read_b64_tr_b8 a[108:109], v121 offset:9216\n"
      "ds_read_b64_tr_b8 a[110:111], v121 offset:10240\n"
      "buffer_load_dwordx4 a[76:79], v120, s[68:71], s79 offen offset:1024 sc0 sc1\n"
      "v_mfma_f32_16x16x128_f8f6f4 v[88:91], a[112:119], a[24:31], v[88:91]\n"
      "buffer_load_dwordx4 a[80:83], v120, s[68:71], s79 offen offset:2048 sc0 sc1\n"
      "v_mfma_f32_16x16x128_f8f6f4 a[0:3], a[112:119], a[32:39], a[0:3]\n"
      "buffer_load_dwordx4 a[84:87], v120, s[68:71], s79 offen offset:3072 sc0 sc1\n"
      "s_waitcnt lgkmcnt(5)\n"
      "v_mfma_f32_16x16x128_f8f6f4 v[28:31], a[120:127], a[8:15], v[28:31]\n"
      "v_xor_b32 v123, 16, v121\n"
      "ds_read_b64_tr_b8 a[112:113], v123 offset:1024\n"
      "ds_read_b64_tr_b8 a[114:115], v123 offset:2048\n"
      "buffer_load_dwordx4 a[88:91], v120, s[68:71], s78 offen offset:0 sc0 sc1\n"
      "v_mfma_f32_16x16x128_f8f6f4 v[60:63], a[120:127], a[16:23], v[60:63]\n"
      "ds_read_b64_tr_b8 a[116:117], v123 offset:9216\n"
      "ds_read_b64_tr_b8 a[118:119], v123 offset:10240\n"
      "s_waitcnt lgkmcnt(8)\n"
      "v_readfirstlane_b32 s50, v127\n"
      "v_readfirstlane_b32 s58, v126\n"
      "buffer_load_dwordx4 a[92:95], v120, s[68:71], s78 offen offset:1024 sc0 sc1\n"
      "v_mfma_f32_16x16x128_f8f6f4 v[92:95], a[120:127], a[24:31], v[92:95]\n"
      "s_cmp_eq_u32 s50, 0\n"
      "s_cselect_b32 s53, 1, s58\n"
      "s_add_u32 s51, s50, -1\n"
      "s_add_u32 s52, s51, 4096\n"
      "buffer_load_dwordx4 a[96:99], v120, s[68:71], s78 offen offset:2048 sc0 sc1\n"
      "v_mfma_f32_16x16x128_f8f6f4 a[4:7], a[120:127], a[32:39], a[4:7]\n"
      "s_add_u32 s85, s85, 1\n"
      "buffer_load_dwordx4 a[100:103], v120, s[68:71], s78 offen offset:3072 sc0 sc1\n"
      "s_branch L_D1_%=\n"
      "L_OS0_%=:\n"
      "s_waitcnt lgkmcnt(4)\n"
      "v_mfma_f32_16x16x128_f8f6f4 v[0:3], a[112:119], a[8:15], v[0:3]\n"
      "v_xor_b32 v122, 32, v121\n"
      "ds_read_b64_tr_b8 a[104:105], v122 offset:1024\n"
      "ds_read_b64_tr_b8 a[106:107], v122 offset:2048\n"
      "buffer_load_dwordx4 a[40:43], v120, s[68:71], s63 offen offset:0 sc0 sc1\n"
      "v_mfma_f32_16x16x128_f8f6f4 v[32:35], a[112:119], a[16:23], v[32:35]\n"
      "ds_read_b64_tr_b8 a[108:109], v122 offset:9216\n"
      "ds_read_b64_tr_b8 a[110:111], v122 offset:10240\n"
      "buffer_load_dwordx4 a[44:47], v120, s[68:71], s63 offen offset:1024 sc0 sc1\n"
      "v_mfma_f32_16x16x128_f8f6f4 v[64:67], a[112:119], a[24:31], v[64:67]\n"
      "buffer_load_dwordx4 a[48:51], v120, s[68:71], s63 offen offset:2048 sc0 sc1\n"
      "v_mfma_f32_16x16x128_f8f6f4 v[96:99], a[112:119], a[32:39], v[96:99]\n"
      "buffer_load_dwordx4 a[52:55], v120, s[68:71], s63 offen offset:3072 sc0 sc1\n"
      "s_waitcnt lgkmcnt(4)\n"
      "v_mfma_f32_16x16x128_f8f6f4 v[4:7], a[120:127], a[8:15], v[4:7]\n"
      "v_xor_b32 v123, 48, v121\n"
      "ds_read_b64_tr_b8 a[112:113], v123 offset:1024\n"
      "ds_read_b64_tr_b8 a[114:115], v123 offset:2048\n"
      "buffer_load_dwordx4 a[56:59], v120, s[68:71], s59 offen offset:0 sc0 sc1\n"
      "v_mfma_f32_16x16x128_f8f6f4 v[36:39], a[120:127], a[16:23], v[36:39]\n"
      "ds_read_b64_tr_b8 a[116:117], v123 offset:9216\n"
      "ds_read_b64_tr_b8 a[118:119], v123 offset:10240\n"
      "buffer_load_dwordx4 a[60:63], v120, s[68:71], s59 offen offset:1024 sc0 sc1\n"
      "v_mfma_f32_16x16x128_f8f6f4 v[68:71], a[120:127], a[24:31], v[68:71]\n"
      "buffer_load_dwordx4 a[64:67], v120, s[68:71], s59 offen offset:2048 sc0 sc1\n"
      "v_mfma_f32_16x16x128_f8f6f4 v[100:103], a[120:127], a[32:39], v[100:103]\n"
      "buffer_load_dwordx4 a[68:71], v120, s[68:71], s59 offen offset:3072 sc0 sc1\n"
      "s_waitcnt lgkmcnt(4)\n"
      "v_mfma_f32_16x16x128_f8f6f4 v[8:11], a[104:111], a[8:15], v[8:11]\n"
      "v_xor_b32 v122, 64, v121\n"
      "ds_read_b64_tr_b8 a[120:121], v122 offset:1024\n"
      "ds_read_b64_tr_b8 a[122:123], v122 offset:2048\n"
      "v_mfma_f32_16x16x128_f8f6f4 v[40:43], a[104:111], a[16:23], v[40:43]\n"
      "ds_read_b64_tr_b8 a[124:125], v122 offset:9216\n"
      "ds_read_b64_tr_b8 a[126:127], v122 offset:10240\n"
      "v_mfma_f32_16x16x128_f8f6f4 v[72:75], a[104:111], a[24:31], v[72:75]\n"
      "v_mfma_f32_16x16x128_f8f6f4 v[104:107], a[104:111], a[32:39], v[104:107]\n"
      "ds_write_b32 v124, v125 offset:16\n"
      "s_waitcnt lgkmcnt(5)\n"
      "v_mfma_f32_16x16x128_f8f6f4 v[12:15], a[112:119], a[8:15], v[12:15]\n"
      "v_xor_b32 v123, 80, v121\n"
      "ds_read_b64_tr_b8 a[104:105], v123 offset:1024\n"
      "ds_read_b64_tr_b8 a[106:107], v123 offset:2048\n"
      "s_mov_b32 s63, 0\n"
      "v_mfma_f32_16x16x128_f8f6f4 v[44:47], a[112:119], a[16:23], v[44:47]\n"
      "ds_read_b64_tr_b8 a[108:109], v123 offset:9216\n"
      "ds_read_b64_tr_b8 a[110:111], v123 offset:10240\n"
      "v_mfma_f32_16x16x128_f8f6f4 v[76:79], a[112:119], a[24:31], v[76:79]\n"
      "v_mfma_f32_16x16x128_f8f6f4 v[108:111], a[112:119], a[32:39], v[108:111]\n"
      "s_waitcnt lgkmcnt(5)\n"
      "v_mfma_f32_16x16x128_f8f6f4 v[16:19], a[120:127], a[8:15], v[16:19]\n"
      "ds_read_b32 v126, v124 offset:0\n"
      "ds_read_b32 v127, v125 offset:76\n"
      "v_xor_b32 v122, 96, v121\n"
      "ds_read_b64_tr_b8 a[112:113], v122 offset:1024\n"
      "ds_read_b64_tr_b8 a[114:115], v122 offset:2048\n"
      "v_mfma_f32_16x16x128_f8f6f4 v[48:51], a[120:127], a[16:23], v[48:51]\n"
      "ds_read_b64_tr_b8 a[116:117], v122 offset:9216\n"
      "ds_read_b64_tr_b8 a[118:119], v122 offset:10240\n"
      "v_mfma_f32_16x16x128_f8f6f4 v[80:83], a[120:127], a[24:31], v[80:83]\n"
      "v_mfma_f32_16x16x128_f8f6f4 v[112:115], a[120:127], a[32:39], v[112:115]\n"
      "s_waitcnt lgkmcnt(6)\n"
      "v_mfma_f32_16x16x128_f8f6f4 v[20:23], a[104:111], a[8:15], v[20:23]\n"
      "v_xor_b32 v123, 112, v121\n"
      "ds_read_b64_tr_b8 a[120:121], v123 offset:1024\n"
      "ds_read_b64_tr_b8 a[122:123], v123 offset:2048\n"
      "v_mfma_f32_16x16x128_f8f6f4 v[52:55], a[104:111], a[16:23], v[52:55]\n"
      "ds_read_b64_tr_b8 a[124:125], v123 offset:9216\n"
      "ds_read_b64_tr_b8 a[126:127], v123 offset:10240\n"
      "v_mfma_f32_16x16x128_f8f6f4 v[84:87], a[104:111], a[24:31], v[84:87]\n"
      "v_add_u32 v125, 4, v125\n"
      "v_mfma_f32_16x16x128_f8f6f4 v[116:119], a[104:111], a[32:39], v[116:119]\n"
      "s_waitcnt lgkmcnt(4)\n"
      "v_readfirstlane_b32 s84, v126\n"
      "v_add_u32 v121, 0x4000, v121\n"
      "s_cmp_lt_u32 s84, s85\n"
      "s_cbranch_scc1 L_slowFS0_%=\n"
      "L_backFS0_%=:\n"
      "v_and_b32 v121, 0x1ffff, v121\n"
      "v_mfma_f32_16x16x128_f8f6f4 v[24:27], a[112:119], a[8:15], v[24:27]\n"
      "ds_read_b32 v126, v124 offset:48\n"
      "ds_read_b64_tr_b8 a[104:105], v121 offset:1024\n"
      "ds_read_b64_tr_b8 a[106:107], v121 offset:2048\n"
      "v_mfma_f32_16x16x128_f8f6f4 v[56:59], a[112:119], a[16:23], v[56:59]\n"
      "ds_read_b64_tr_b8 a[108:109], v121 offset:9216\n"
      "ds_read_b64_tr_b8 a[110:111], v121 offset:10240\n"
      "v_mfma_f32_16x16x128_f8f6f4 v[88:91], a[112:119], a[24:31], v[88:91]\n"
      "v_mfma_f32_16x16x128_f8f6f4 a[0:3], a[112:119], a[32:39], a[0:3]\n"
      "s_waitcnt lgkmcnt(5)\n"
      "v_mfma_f32_16x16x128_f8f6f4 v[28:31], a[120:127], a[8:15], v[28:31]\n"
      "v_xor_b32 v123, 16, v121\n"
      "ds_read_b64_tr_b8 a[112:113], v123 offset:1024\n"
      "ds_read_b64_tr_b8 a[114:115], v123 offset:2048\n"
      "v_mfma_f32_16x16x128_f8f6f4 v[60:63], a[120:127], a[16:23], v[60:63]\n"
      "ds_read_b64_tr_b8 a[116:117], v123 offset:9216\n"
      "ds_read_b64_tr_b8 a[118:119], v123 offset:10240\n"
      "s_waitcnt lgkmcnt(8)\n"
      "v_readfirstlane_b32 s50, v127\n"
      "v_readfirstlane_b32 s58, v126\n"
      "v_mfma_f32_16x16x128_f8f6f4 v[92:95], a[120:127], a[24:31], v[92:95]\n"
      "s_cmp_eq_u32 s50, 0\n"
      "s_cselect_b32 s53, 1, s58\n"
      "s_add_u32 s51, s50, -1\n"
      "s_add_u32 s52, s51, 4096\n"
      "v_mfma_f32_16x16x128_f8f6f4 a[4:7], a[120:127], a[32:39], a[4:7]\n"
      "s_add_u32 s85, s85, 1\n"
      "s_branch L_D1_%=\n"
      "L_SK1_%=:\n"
      "s_mov_b32 s63, s50\n"
      "s_mov_b32 s62, 1\n"
      "s_waitcnt lgkmcnt(4)\n"
      "v_mfma_f32_16x16x128_f8f6f4 v[0:3], a[104:111], a[40:47], v[0:3]\n"
      "v_xor_b32 v122, 32, v121\n"
      "ds_read_b64_tr_b8 a[120:121], v122 offset:1024\n"
      "ds_read_b64_tr_b8 a[122:123], v122 offset:2048\n"
      "v_mfma_f32_16x16x128_f8f6f4 v[32:35], a[104:111], a[48:55], v[32:35]\n"
      "ds_read_b64_tr_b8 a[124:125], v122 offset:9216\n"
      "ds_read_b64_tr_b8 a[126:127], v122 offset:10240\n"
      "v_mfma_f32_16x16x128_f8f6f4 v[64:67], a[104:111], a[56:63], v[64:67]\n"
      "v_mfma_f32_16x16x128_f8f6f4 v[96:99], a[104:111], a[64:71], v[96:99]\n"
      "s_waitcnt lgkmcnt(4)\n"
      "v_mfma_f32_16x16x128_f8f6f4 v[4:7], a[112:119], a[40:47], v[4:7]\n"
      "v_xor_b32 v123, 48, v121\n"
      "ds_read_b64_tr_b8 a[104:105], v123 offset:1024\n"
      "ds_read_b64_tr_b8 a[106:107], v123 offset:2048\n"
      "v_mfma_f32_16x16x128_f8f6f4 v[36:39], a[112:119], a[48:55], v[36:39]\n"
      "ds_read_b64_tr_b8 a[108:109], v123 offset:9216\n"
      "ds_read_b64_tr_b8 a[110:111], v123 offset:10240\n"
      "v_mfma_f32_16x16x128_f8f6f4 v[68:71], a[112:119], a[56:63], v[68:71]\n"
      "v_mfma_f32_16x16x128_f8f6f4 v[100:103], a[112:119], a[64:71], v[100:103]\n"
      "s_waitcnt lgkmcnt(4)\n"
      "v_mfma_f32_16x16x128_f8f6f4 v[8:11], a[120:127], a[40:47], v[8:11]\n"
      "v_xor_b32 v122, 64, v121\n"
      "ds_read_b64_tr_b8 a[112:113], v122 offset:1024\n"
      "ds_read_b64_tr_b8 a[114:115], v122 offset:2048\n"
      "v_mfma_f32_16x16x128_f8f6f4 v[40:43], a[120:127], a[48:55], v[40:43]\n"
      "ds_read_b64_tr_b8 a[116:117], v122 offset:9216\n"
      "ds_read_b64_tr_b8 a[118:119], v122 offset:10240\n"
      "v_mfma_f32_16x16x128_f8f6f4 v[72:75], a[120:127], a[56:63], v[72:75]\n"
      "v_mfma_f32_16x16x128_f8f6f4 v[104:107], a[120:127], a[64:71], v[104:107]\n"
      "ds_write_b32 v124, v125 offset:16\n"
      "s_waitcnt lgkmcnt(5)\n"
      "v_mfma_f32_16x16x128_f8f6f4 v[12:15], a[104:111], a[40:47], v[12:15]\n"
      "v_xor_b32 v123, 80, v121\n"
      "ds_read_b64_tr_b8 a[120:121], v123 offset:1024\n"
      "ds_read_b64_tr_b8 a[122:123], v123 offset:2048\n"
      "v_mfma_f32_16x16x128_f8f6f4 v[44:47], a[104:111], a[48:55], v[44:47]\n"
      "ds_read_b64_tr_b8 a[124:125], v123 offset:9216\n"
      "ds_read_b64_tr_b8 a[126:127], v123 offset:10240\n"
      "v_mfma_f32_16x16x128_f8f6f4 v[76:79], a[104:111], a[56:63], v[76:79]\n"
      "v_mfma_f32_16x16x128_f8f6f4 v[108:111], a[104:111], a[64:71], v[108:111]\n"
      "s_waitcnt lgkmcnt(5)\n"
      "v_mfma_f32_16x16x128_f8f6f4 v[16:19], a[112:119], a[40:47], v[16:19]\n"
      "ds_read_b32 v126, v124 offset:0\n"
      "ds_read_b32 v127, v125 offset:76\n"
      "v_xor_b32 v122, 96, v121\n"
      "ds_read_b64_tr_b8 a[104:105], v122 offset:1024\n"
      "ds_read_b64_tr_b8 a[106:107], v122 offset:2048\n"
      "v_mfma_f32_16x16x128_f8f6f4 v[48:51], a[112:119], a[48:55], v[48:51]\n"
      "ds_read_b64_tr_b8 a[108:109], v122 offset:9216\n"
      "ds_read_b64_tr_b8 a[110:111], v122 offset:10240\n"
      "v_mfma_f32_16x16x128_f8f6f4 v[80:83], a[112:119], a[56:63], v[80:83]\n"
      "v_mfma_f32_16x16x128_f8f6f4 v[112:115], a[112:119], a[64:71], v[112:115]\n"
      "s_waitcnt lgkmcnt(6)\n"
      "v_mfma_f32_16x16x128_f8f6f4 v[20:23], a[120:127], a[40:47], v[20:23]\n"
      "v_xor_b32 v123, 112, v121\n"
      "ds_read_b64_tr_b8 a[112:113], v123 offset:1024\n"
      "ds_read_b64_tr_b8 a[114:115], v123 offset:2048\n"
      "v_mfma_f32_16x16x128_f8f6f4 v[52:55], a[120:127], a[48:55], v[52:55]\n"
      "ds_read_b64_tr_b8 a[116:117], v123 offset:9216\n"
      "ds_read_b64_tr_b8 a[118:119], v123 offset:10240\n"
      "v_mfma_f32_16x16x128_f8f6f4 v[84:87], a[120:127], a[56:63], v[84:87]\n"
      "v_add_u32 v125, 4, v125\n"
      "v_mfma_f32_16x16x128_f8f6f4 v[116:119], a[120:127], a[64:71], v[116:119]\n"
      "s_waitcnt lgkmcnt(4)\n"
      "v_readfirstlane_b32 s84, v126\n"
      "v_add_u32 v121, 0x4000, v121\n"
      "s_cmp_lt_u32 s84, s85\n"
      "s_cbranch_scc1 L_slowFK1_%=\n"
      "L_backFK1_%=:\n"
      "v_and_b32 v121, 0x1ffff, v121\n"
      "v_mfma_f32_16x16x128_f8f6f4 v[24:27], a[104:111], a[40:47], v[24:27]\n"
      "ds_read_b32 v126, v124 offset:48\n"
      "ds_read_b64_tr_b8 a[120:121], v121 offset:1024\n"
      "ds_read_b64_tr_b8 a[122:123], v121 offset:2048\n"
      "v_mfma_f32_16x16x128_f8f6f4 v[56:59], a[104:111], a[48:55], v[56:59]\n"
      "ds_read_b64_tr_b8 a[124:125], v121 offset:9216\n"
      "ds_read_b64_tr_b8 a[126:127], v121 offset:10240\n"
      "v_mfma_f32_16x16x128_f8f6f4 v[88:91], a[104:111], a[56:63], v[88:91]\n"
      "v_mfma_f32_16x16x128_f8f6f4 a[0:3], a[104:111], a[64:71], a[0:3]\n"
      "s_waitcnt lgkmcnt(5)\n"
      "v_mfma_f32_16x16x128_f8f6f4 v[28:31], a[112:119], a[40:47], v[28:31]\n"
      "v_xor_b32 v123, 16, v121\n"
      "ds_read_b64_tr_b8 a[104:105], v123 offset:1024\n"
      "ds_read_b64_tr_b8 a[106:107], v123 offset:2048\n"
      "v_mfma_f32_16x16x128_f8f6f4 v[60:63], a[112:119], a[48:55], v[60:63]\n"
      "ds_read_b64_tr_b8 a[108:109], v123 offset:9216\n"
      "ds_read_b64_tr_b8 a[110:111], v123 offset:10240\n"
      "s_waitcnt lgkmcnt(8)\n"
      "v_readfirstlane_b32 s50, v127\n"
      "v_readfirstlane_b32 s58, v126\n"
      "v_mfma_f32_16x16x128_f8f6f4 v[92:95], a[112:119], a[56:63], v[92:95]\n"
      "s_cmp_eq_u32 s50, 0\n"
      "s_cselect_b32 s53, 1, s58\n"
      "s_add_u32 s51, s50, -1\n"
      "s_add_u32 s52, s51, 4096\n"
      "v_mfma_f32_16x16x128_f8f6f4 a[4:7], a[112:119], a[64:71], a[4:7]\n"
      "s_add_u32 s85, s85, 1\n"
      "s_branch L_D2_%=\n"
      "L_ON1_%=:\n"
      "s_waitcnt vmcnt(0)\n"
      "s_mov_b32 s79, s50\n"
      "s_cmp_eq_u32 s63, 0\n"
      "s_cbranch_scc1 L_slowO1_%=\n"
      "L_backO1_%=:\n"
      "s_add_u32 s63, s63, -1\n"
      "s_add_u32 s59, s63, 4096\n"
      "s_cmp_eq_u32 s79, 0\n"
      "s_cbranch_scc1 L_OS1_%=\n"
      "L_backA1_%=:\n"
      "s_add_u32 s79, s79, -1\n"
      "s_add_u32 s78, s79, 4096\n"
      "s_mov_b32 s62, 0\n"
      "s_cmp_lg_u32 s58, 0\n"
      "s_cbranch_scc1 L_OL1_%=\n"
      "s_waitcnt lgkmcnt(4)\n"
      "v_mfma_f32_16x16x128_f8f6f4 v[0:3], a[104:111], a[40:47], v[0:3]\n"
      "v_xor_b32 v122, 32, v121\n"
      "ds_read_b64_tr_b8 a[120:121], v122 offset:1024\n"
      "ds_read_b64_tr_b8 a[122:123], v122 offset:2048\n"
      "buffer_load_dwordx4 a[72:75], v120, s[68:71], s63 offen offset:0 sc0 sc1\n"
      "v_mfma_f32_16x16x128_f8f6f4 v[32:35], a[104:111], a[48:55], v[32:35]\n"
      "ds_read_b64_tr_b8 a[124:125], v122 offset:9216\n"
      "ds_read_b64_tr_b8 a[126:127], v122 offset:10240\n"
      "buffer_load_dwordx4 a[76:79], v120, s[68:71], s63 offen offset:1024 sc0 sc1\n"
      "v_mfma_f32_16x16x128_f8f6f4 v[64:67], a[104:111], a[56:63], v[64:67]\n"
      "buffer_load_dwordx4 a[80:83], v120, s[68:71], s63 offen offset:2048 sc0 sc1\n"
      "v_mfma_f32_16x16x128_f8f6f4 v[96:99], a[104:111], a[64:71], v[96:99]\n"
      "buffer_load_dwordx4 a[84:87], v120, s[68:71], s63 offen offset:3072 sc0 sc1\n"
      "s_waitcnt lgkmcnt(4)\n"
      "v_mfma_f32_16x16x128_f8f6f4 v[4:7], a[112:119], a[40:47], v[4:7]\n"
      "v_xor_b32 v123, 48, v121\n"
      "ds_read_b64_tr_b8 a[104:105], v123 offset:1024\n"
      "ds_read_b64_tr_b8 a[106:107], v123 offset:2048\n"
      "buffer_load_dwordx4 a[88:91], v120, s[68:71], s59 offen offset:0 sc0 sc1\n"
      "v_mfma_f32_16x16x128_f8f6f4 v[36:39], a[112:119], a[48:55], v[36:39]\n"
      "ds_read_b64_tr_b8 a[108:109], v123 offset:9216\n"
      "ds_read_b64_tr_b8 a[110:111], v123 offset:10240\n"
      "buffer_load_dwordx4 a[92:95], v120, s[68:71], s59 offen offset:1024 sc0 sc1\n"
      "v_mfma_f32_16x16x128_f8f6f4 v[68:71], a[112:119], a[56:63], v[68:71]\n"
      "buffer_load_dwordx4 a[96:99], v120, s[68:71], s59 offen offset:2048 sc0 sc1\n"
      "v_mfma_f32_16x16x128_f8f6f4 v[100:103], a[112:119], a[64:71], v[100:103]\n"
      "buffer_load_dwordx4 a[100:103], v120, s[68:71], s59 offen offset:3072 sc0 sc1\n"
      "s_waitcnt lgkmcnt(4)\n"
      "v_mfma_f32_16x16x128_f8f6f4 v[8:11], a[120:127], a[40:47], v[8:11]\n"
      "v_xor_b32 v122, 64, v121\n"
      "ds_read_b64_tr_b8 a[112:113], v122 offset:1024\n"
      "ds_read_b64_tr_b8 a[114:115], v122 offset:2048\n"
      "v_mfma_f32_16x16x128_f8f6f4 v[40:43], a[120:127], a[48:55], v[40:43]\n"
      "ds_read_b64_tr_b8 a[116:117], v122 offset:9216\n"
      "ds_read_b64_tr_b8 a[118:119], v122 offset:10240\n"
      "v_mfma_f32_16x16x128_f8f6f4 v[72:75], a[120:127], a[56:63], v[72:75]\n"
      "v_mfma_f32_16x16x128_f8f6f4 v[104:107], a[120:127], a[64:71], v[104:107]\n"
      "ds_write_b32 v124, v125 offset:16\n"
      "s_waitcnt lgkmcnt(5)\n"
      "v_mfma_f32_16x16x128_f8f6f4 v[12:15], a[104:111], a[40:47], v[12:15]\n"
      "v_xor_b32 v123, 80, v121\n"
      "ds_read_b64_tr_b8 a[120:121], v123 offset:1024\n"
      "ds_read_b64_tr_b8 a[122:123], v123 offset:2048\n"
      "v_mfma_f32_16x16x128_f8f6f4 v[44:47], a[104:111], a[48:55], v[44:47]\n"
      "ds_read_b64_tr_b8 a[124:125], v123 offset:9216\n"
      "ds_read_b64_tr_b8 a[126:127], v123 offset:10240\n"
      "v_mfma_f32_16x16x128_f8f6f4 v[76:79], a[104:111], a[56:63], v[76:79]\n"
      "v_mfma_f32_16x16x128_f8f6f4 v[108:111], a[104:111], a[64:71], v[108:111]\n"
      "s_waitcnt lgkmcnt(5)\n"
      "v_mfma_f32_16x16x128_f8f6f4 v[16:19], a[112:119], a[40:47], v[16:19]\n"
      "ds_read_b32 v126, v124 offset:0\n"
      "ds_read_b32 v127, v125 offset:76\n"
      "v_xor_b32 v122, 96, v121\n"
      "ds_read_b64_tr_b8 a[104:105], v122 offset:1024\n"
      "ds_read_b64_tr_b8 a[106:107], v122 offset:2048\n"
      "buffer_load_dwordx4 a[8:11], v120, s[68:71], s79 offen offset:0 sc0 sc1\n"
      "v_mfma_f32_16x16x128_f8f6f4 v[48:51], a[112:119], a[48:55], v[48:51]\n"
      "ds_read_b64_tr_b8 a[108:109], v122 offset:9216\n"
      "ds_read_b64_tr_b8 a[110:111], v122 offset:10240\n"
      "buffer_load_dwordx4 a[12:15], v120, s[68:71], s79 offen offset:1024 sc0 sc1\n"
      "v_mfma_f32_16x16x128_f8f6f4 v[80:83], a[112:119], a[56:63], v[80:83]\n"
      "buffer_load_dwordx4 a[16:19], v120, s[68:71], s79 offen offset:2048 sc0 sc1\n"
      "v_mfma_f32_16x16x128_f8f6f4 v[112:115], a[112:119], a[64:71], v[112:115]\n"
      "buffer_load_dwordx4 a[20:23], v120, s[68:71], s79 offen offset:3072 sc0 sc1\n"
      "s_waitcnt lgkmcnt(6)\n"
      "v_mfma_f32_16x16x128_f8f6f4 v[20:23], a[120:127], a[40:47], v[20:23]\n"
      "v_xor_b32 v123, 112, v121\n"
      "ds_read_b64_tr_b8 a[112:113], v123 offset:1024\n"
      "ds_read_b64_tr_b8 a[114:115], v123 offset:2048\n"
      "buffer_load_dwordx4 a[24:27], v120, s[68:71], s78 offen offset:0 sc0 sc1\n"
      "v_mfma_f32_16x16x128_f8f6f4 v[52:55], a[120:127], a[48:55], v[52:55]\n"
      "ds_read_b64_tr_b8 a[116:117], v123 offset:9216\n"
      "ds_read_b64_tr_b8 a[118:119], v123 offset:10240\n"
      "buffer_load_dwordx4 a[28:31], v120, s[68:71], s78 offen offset:1024 sc0 sc1\n"
      "v_mfma_f32_16x16x128_f8f6f4 v[84:87], a[120:127], a[56:63], v[84:87]\n"
      "v_add_u32 v125, 4, v125\n"
      "buffer_load_dwordx4 a[32:35], v120, s[68:71], s78 offen offset:2048 sc0 sc1\n"
      "v_mfma_f32_16x16x128_f8f6f4 v[116:119], a[120:127], a[64:71], v[116:119]\n"
      "buffer_load_dwordx4 a[36:39], v120, s[68:71], s78 offen offset:3072 sc0 sc1\n"
      "s_waitcnt lgkmcnt(4)\n"
      "v_readfirstlane_b32 s84, v126\n"
      "v_add_u32 v121, 0x4000, v121\n"
      "s_cmp_lt_u32 s84, s85\n"
      "s_cbranch_scc1 L_slowFO1_%=\n"
      "L_backFO1_%=:\n"
      "v_and_b32 v121, 0x1ffff, v121\n"
      "v_mfma_f32_16x16x128_f8f6f4 v[24:27], a[104:111], a[40:47], v[24:27]\n"
      "ds_read_b32 v126, v124 offset:48\n"
      "ds_read_b64_tr_b8 a[120:121], v121 offset:1024\n"
      "ds_read_b64_tr_b8 a[122:123], v121 offset:2048\n"
      "v_mfma_f32_16x16x128_f8f6f4 v[56:59], a[104:111], a[48:55], v[56:59]\n"
      "ds_read_b64_tr_b8 a[124:125], v121 offset:9216\n"
      "ds_read_b64_tr_b8 a[126:127], v121 offset:10240\n"
      "v_mfma_f32_16x16x128_f8f6f4 v[88:91], a[104:111], a[56:63], v[88:91]\n"
      "v_mfma_f32_16x16x128_f8f6f4 a[0:3], a[104:111], a[64:71], a[0:3]\n"
      "s_waitcnt lgkmcnt(5)\n"
      "v_mfma_f32_16x16x128_f8f6f4 v[28:31], a[112:119], a[40:47], v[28:31]\n"
      "v_xor_b32 v123, 16, v121\n"
      "ds_read_b64_tr_b8 a[104:105], v123 offset:1024\n"
      "ds_read_b64_tr_b8 a[106:107], v123 offset:2048\n"
      "v_mfma_f32_16x16x128_f8f6f4 v[60:63], a[112:119], a[48:55], v[60:63]\n"
      "ds_read_b64_tr_b8 a[108:109], v123 offset:9216\n"
      "ds_read_b64_tr_b8 a[110:111], v123 offset:10240\n"
      "s_waitcnt lgkmcnt(8)\n"
      "v_readfirstlane_b32 s50, v127\n"
      "v_readfirstlane_b32 s58, v126\n"
      "v_mfma_f32_16x16x128_f8f6f4 v[92:95], a[112:119], a[56:63], v[92:95]\n"
      "s_cmp_eq_u32 s50, 0\n"
      "s_cselect_b32 s53, 1, s58\n"
      "s_add_u32 s51, s50, -1\n"
      "s_add_u32 s52, s51, 4096\n"
      "v_mfma_f32_16x16x128_f8f6f4 a[4:7], a[112:119], a[64:71], a[4:7]\n"
      "s_add_u32 s85, s85, 1\n"
      "s_branch L_D2_%=\n"
      "L_OL1_%=:\n"
      "s_waitcnt lgkmcnt(4)\n"
      "v_mfma_f32_16x16x128_f8f6f4 v[0:3], a[104:111], a[40:47], v[0:3]\n"
      "v_xor_b32 v122, 32, v121\n"
      "ds_read_b64_tr_b8 a[120:121], v122 offset:1024\n"
      "ds_read_b64_tr_b8 a[122:123], v122 offset:2048\n"
      "v_mfma_f32_16x16x128_f8f6f4 v[32:35], a[104:111], a[48:55], v[32:35]\n"
      "ds_read_b64_tr_b8 a[124:125], v122 offset:9216\n"
      "ds_read_b64_tr_b8 a[126:127], v122 offset:10240\n"
      "v_mfma_f32_16x16x128_f8f6f4 v[64:67], a[104:111], a[56:63], v[64:67]\n"
      "v_mfma_f32_16x16x128_f8f6f4 v[96:99], a[104:111], a[64:71], v[96:99]\n"
      "s_waitcnt lgkmcnt(4)\n"
      "v_mfma_f32_16x16x128_f8f6f4 v[4:7], a[112:119], a[40:47], v[4:7]\n"
      "v_xor_b32 v123, 48, v121\n"
      "ds_read_b64_tr_b8 a[104:105], v123 offset:1024\n"
      "ds_read_b64_tr_b8 a[106:107], v123 offset:2048\n"
      "v_mfma_f32_16x16x128_f8f6f4 v[36:39], a[112:119], a[48:55], v[36:39]\n"
      "ds_read_b64_tr_b8 a[108:109], v123 offset:9216\n"
      "ds_read_b64_tr_b8 a[110:111], v123 offset:10240\n"
      "v_mfma_f32_16x16x128_f8f6f4 v[68:71], a[112:119], a[56:63], v[68:71]\n"
      "v_mfma_f32_16x16x128_f8f6f4 v[100:103], a[112:119], a[64:71], v[100:103]\n"
      "s_waitcnt lgkmcnt(4)\n"
      "v_mfma_f32_16x16x128_f8f6f4 v[8:11], a[120:127], a[40:47], v[8:11]\n"
      "v_xor_b32 v122, 64, v121\n"
      "ds_read_b64_tr_b8 a[112:113], v122 offset:1024\n"
      "ds_read_b64_tr_b8 a[114:115], v122 offset:2048\n"
      "v_mfma_f32_16x16x128_f8f6f4 v[40:43], a[120:127], a[48:55], v[40:43]\n"
      "ds_read_b64_tr_b8 a[116:117], v122 offset:9216\n"
      "ds_read_b64_tr_b8 a[118:119], v122 offset:10240\n"
      "v_mfma_f32_16x16x128_f8f6f4 v[72:75], a[120:127], a[56:63], v[72:75]\n"
      "v_mfma_f32_16x16x128_f8f6f4 v[104:107], a[120:127], a[64:71], v[104:107]\n"
      "ds_write_b32 v124, v125 offset:16\n"
      "s_waitcnt lgkmcnt(5)\n"
      "v_mfma_f32_16x16x128_f8f6f4 v[12:15], a[104:111], a[40:47], v[12:15]\n"
      "v_xor_b32 v123, 80, v121\n"
      "ds_read_b64_tr_b8 a[120:121], v123 offset:1024\n"
      "ds_read_b64_tr_b8 a[122:123], v123 offset:2048\n"
      "v_mfma_f32_16x16x128_f8f6f4 v[44:47], a[104:111], a[48:55], v[44:47]\n"
      "ds_read_b64_tr_b8 a[124:125], v123 offset:9216\n"
      "ds_read_b64_tr_b8 a[126:127], v123 offset:10240\n"
      "v_mfma_f32_16x16x128_f8f6f4 v[76:79], a[104:111], a[56:63], v[76:79]\n"
      "v_mfma_f32_16x16x128_f8f6f4 v[108:111], a[104:111], a[64:71], v[108:111]\n"
      "s_waitcnt lgkmcnt(5)\n"
      "v_mfma_f32_16x16x128_f8f6f4 v[16:19], a[112:119], a[40:47], v[16:19]\n"
      "ds_read_b32 v126, v124 offset:0\n"
      "ds_read_b32 v127, v125 offset:76\n"
      "v_xor_b32 v122, 96, v121\n"
      "ds_read_b64_tr_b8 a[104:105], v122 offset:1024\n"
      "ds_read_b64_tr_b8 a[106:107], v122 offset:2048\n"
      "buffer_load_dwordx4 a[72:75], v120, s[68:71], s63 offen offset:0 sc0 sc1\n"
      "v_mfma_f32_16x16x128_f8f6f4 v[48:51], a[112:119], a[48:55], v[48:51]\n"
      "ds_read_b64_tr_b8 a[108:109], v122 offset:9216\n"
      "ds_read_b64_tr_b8 a[110:111], v122 offset:10240\n"
      "buffer_load_dwordx4 a[76:79], v120, s[68:71], s63 offen offset:1024 sc0 sc1\n"
      "v_mfma_f32_16x16x128_f8f6f4 v[80:83], a[112:119], a[56:63], v[80:83]\n"
      "buffer_load_dwordx4 a[80:83], v120, s[68:71], s63 offen offset:2048 sc0 sc1\n"
      "v_mfma_f32_16x16x128_f8f6f4 v[112:115], a[112:119], a[64:71], v[112:115]\n"
      "buffer_load_dwordx4 a[84:87], v120, s[68:71], s63 offen offset:3072 sc0 sc1\n"
      "s_waitcnt lgkmcnt(6)\n"
      "v_mfma_f32_16x16x128_f8f6f4 v[20:23], a[120:127], a[40:47], v[20:23]\n"
      "v_xor_b32 v123, 112, v121\n"
      "ds_read_b64_tr_b8 a[112:113], v123 offset:1024\n"
      "ds_read_b64_tr_b8 a[114:115], v123 offset:2048\n"
      "buffer_load_dwordx4 a[88:91], v120, s[68:71], s59 offen offset:0 sc0 sc1\n"
      "v_mfma_f32_16x16x128_f8f6f4 v[52:55], a[120:127], a[48:55], v[52:55]\n"
      "ds_read_b64_tr_b8 a[116:117], v123 offset:9216\n"
      "ds_read_b64_tr_b8 a[118:119], v123 offset:10240\n"
      "buffer_load_dwordx4 a[92:95], v120, s[68:71], s59 offen offset:1024 sc0 sc1\n"
      "v_mfma_f32_16x16x128_f8f6f4 v[84:87], a[120:127], a[56:63], v[84:87]\n"
      "v_add_u32 v125, 4, v125\n"
      "buffer_load_dwordx4 a[96:99], v120, s[68:71], s59 offen offset:2048 sc0 sc1\n"
      "v_mfma_f32_16x16x128_f8f6f4 v[116:119], a[120:127], a[64:71], v[116:119]\n"
      "buffer_load_dwordx4 a[100:103], v120, s[68:71], s59 offen offset:3072 sc0 sc1\n"
      "s_waitcnt lgkmcnt(4)\n"
      "v_readfirstlane_b32 s84, v126\n"
      "v_add_u32 v121, 0x4000, v121\n"
      "s_cmp_lt_u32 s84, s85\n"
      "s_cbranch_scc1 L_slowFL1_%=\n"
      "L_backFL1_%=:\n"
      "v_and_b32 v121, 0x1ffff, v121\n"
      "v_mfma_f32_16x16x128_f8f6f4 v[24:27], a[104:111], a[40:47], v[24:27]\n"
      "ds_read_b32 v126, v124 offset:48\n"
      "ds_read_b64_tr_b8 a[120:121], v121 offset:1024\n"
      "ds_read_b64_tr_b8 a[122:123], v121 offset:2048\n"
      "buffer_load_dwordx4 a[8:11], v120, s[68:71], s79 offen offset:0 sc0 sc1\n"
      "v_mfma_f32_16x16x128_f8f6f4 v[56:59], a[104:111], a[48:55], v[56:59]\n"
      "ds_read_b64_tr_b8 a[124:125], v121 offset:9216\n"
      "ds_read_b64_tr_b8 a[126:127], v121 offset:10240\n"
      "buffer_load_dwordx4 a[12:15], v120, s[68:71], s79 offen offset:1024 sc0 sc1\n"
      "v_mfma_f32_16x16x128_f8f6f4 v[88:91], a[104:111], a[56:63], v[88:91]\n"
      "buffer_load_dwordx4 a[16:19], v120, s[68:71], s79 offen offset:2048 sc0 sc1\n"
      "v_mfma_f32_16x16x128_f8f6f4 a[0:3], a[104:111], a[64:71], a[0:3]\n"
      "buffer_load_dwordx4 a[20:23], v120, s[68:71], s79 offen offset:3072 sc0 sc1\n"
      "s_waitcnt lgkmcnt(5)\n"
      "v_mfma_f32_16x16x128_f8f6f4 v[28:31], a[112:119], a[40:47], v[28:31]\n"
      "v_xor_b32 v123, 16, v121\n"
      "ds_read_b64_tr_b8 a[104:105], v123 offset:1024\n"
      "ds_read_b64_tr_b8 a[106:107], v123 offset:2048\n"
      "buffer_load_dwordx4 a[24:27], v120, s[68:71], s78 offen offset:0 sc0 sc1\n"
      "v_mfma_f32_16x16x128_f8f6f4 v[60:63], a[112:119], a[48:55], v[60:63]\n"
      "ds_read_b64_tr_b8 a[108:109], v123 offset:9216\n"
      "ds_read_b64_tr_b8 a[110:111], v123 offset:10240\n"
      "s_waitcnt lgkmcnt(8)\n"
      "v_readfirstlane_b32 s50, v127\n"
      "v_readfirstlane_b32 s58, v126\n"
      "buffer_load_dwordx4 a[28:31], v120, s[68:71], s78 offen offset:1024 sc0 sc1\n"
      "v_mfma_f32_16x16x128_f8f6f4 v[92:95], a[112:119], a[56:63], v[92:95]\n"
      "s_cmp_eq_u32 s50, 0\n"
      "s_cselect_b32 s53, 1, s58\n"
      "s_add_u32 s51, s50, -1\n"
      "s_add_u32 s52, s51, 4096\n"
      "buffer_load_dwordx4 a[32:35], v120, s[68:71], s78 offen offset:2048 sc0 sc1\n"
      "v_mfma_f32_16x16x128_f8f6f4 a[4:7], a[112:119], a[64:71], a[4:7]\n"
      "s_add_u32 s85, s85, 1\n"
      "buffer_load_dwordx4 a[36:39], v120, s[68:71], s78 offen offset:3072 sc0 sc1\n"
      "s_branch L_D2_%=\n"
      "L_OS1_%=:\n"
      "s_waitcnt lgkmcnt(4)\n"
      "v_mfma_f32_16x16x128_f8f6f4 v[0:3], a[104:111], a[40:47], v[0:3]\n"
      "v_xor_b32 v122, 32, v121\n"
      "ds_read_b64_tr_b8 a[120:121], v122 offset:1024\n"
      "ds_read_b64_tr_b8 a[122:123], v122 offset:2048\n"
      "buffer_load_dwordx4 a[72:75], v120, s[68:71], s63 offen offset:0 sc0 sc1\n"
      "v_mfma_f32_16x16x128_f8f6f4 v[32:35], a[104:111], a[48:55], v[32:35]\n"
      "ds_read_b64_tr_b8 a[124:125], v122 offset:9216\n"
      "ds_read_b64_tr_b8 a[126:127], v122 offset:10240\n"
      "buffer_load_dwordx4 a[76:79], v120, s[68:71], s63 offen offset:1024 sc0 sc1\n"
      "v_mfma_f32_16x16x128_f8f6f4 v[64:67], a[104:111], a[56:63], v[64:67]\n"
      "buffer_load_dwordx4 a[80:83], v120, s[68:71], s63 offen offset:2048 sc0 sc1\n"
      "v_mfma_f32_16x16x128_f8f6f4 v[96:99], a[104:111], a[64:71], v[96:99]\n"
      "buffer_load_dwordx4 a[84:87], v120, s[68:71], s63 offen offset:3072 sc0 sc1\n"
      "s_waitcnt lgkmcnt(4)\n"
      "v_mfma_f32_16x16x128_f8f6f4 v[4:7], a[112:119], a[40:47], v[4:7]\n"
      "v_xor_b32 v123, 48, v121\n"
      "ds_read_b64_tr_b8 a[104:105], v123 offset:1024\n"
      "ds_read_b64_tr_b8 a[106:107], v123 offset:2048\n"
      "buffer_load_dwordx4 a[88:91], v120, s[68:71], s59 offen offset:0 sc0 sc1\n"
      "v_mfma_f32_16x16x128_f8f6f4 v[36:39], a[112:119], a[48:55], v[36:39]\n"
      "ds_read_b64_tr_b8 a[108:109], v123 offset:9216\n"
      "ds_read_b64_tr_b8 a[110:111], v123 offset:10240\n"
      "buffer_load_dwordx4 a[92:95], v120, s[68:71], s59 offen offset:1024 sc0 sc1\n"
      "v_mfma_f32_16x16x128_f8f6f4 v[68:71], a[112:119], a[56:63], v[68:71]\n"
      "buffer_load_dwordx4 a[96:99], v120, s[68:71], s59 offen offset:2048 sc0 sc1\n"
      "v_mfma_f32_16x16x128_f8f6f4 v[100:103], a[112:119], a[64:71], v[100:103]\n"
      "buffer_load_dwordx4 a[100:103], v120, s[68:71], s59 offen offset:3072 sc0 sc1\n"
      "s_waitcnt lgkmcnt(4)\n"
      "v_mfma_f32_16x16x128_f8f6f4 v[8:11], a[120:127], a[40:47], v[8:11]\n"
      "v_xor_b32 v122, 64, v121\n"
      "ds_read_b64_tr_b8 a[112:113], v122 offset:1024\n"
      "ds_read_b64_tr_b8 a[114:115], v122 offset:2048\n"
      "v_mfma_f32_16x16x128_f8f6f4 v[40:43], a[120:127], a[48:55], v[40:43]\n"
      "ds_read_b64_tr_b8 a[116:117], v122 offset:9216\n"
      "ds_read_b64_tr_b8 a[118:119], v122 offset:10240\n"
      "v_mfma_f32_16x16x128_f8f6f4 v[72:75], a[120:127], a[56:63], v[72:75]\n"
      "v_mfma_f32_16x16x128_f8f6f4 v[104:107], a[120:127], a[64:71], v[104:107]\n"
      "ds_write_b32 v124, v125 offset:16\n"
      "s_waitcnt lgkmcnt(5)\n"
      "v_mfma_f32_16x16x128_f8f6f4 v[12:15], a[104:111], a[40:47], v[12:15]\n"
      "v_xor_b32 v123, 80, v121\n"
      "ds_read_b64_tr_b8 a[120:121], v123 offset:1024\n"
      "ds_read_b64_tr_b8 a[122:123], v123 offset:2048\n"
      "s_mov_b32 s63, 0\n"
      "v_mfma_f32_16x16x128_f8f6f4 v[44:47], a[104:111], a[48:55], v[44:47]\n"
      "ds_read_b64_tr_b8 a[124:125], v123 offset:9216\n"
      "ds_read_b64_tr_b8 a[126:127], v123 offset:10240\n"
      "v_mfma_f32_16x16x128_f8f6f4 v[76:79], a[104:111], a[56:63], v[76:79]\n"
      "v_mfma_f32_16x16x128_f8f6f4 v[108:111], a[104:111], a[64:71], v[108:111]\n"
      "s_waitcnt lgkmcnt(5)\n"
      "v_mfma_f32_16x16x128_f8f6f4 v[16:19], a[112:119], a[40:47], v[16:19]\n"
      "ds_read_b32 v126, v124 offset:0\n"
      "ds_read_b32 v127, v125 offset:76\n"
      "v_xor_b32 v122, 96, v121\n"
      "ds_read_b64_tr_b8 a[104:105], v122 offset:1024\n"
      "ds_read_b64_tr_b8 a[106:107], v122 offset:2048\n"
      "v_mfma_f32_16x16x128_f8f6f4 v[48:51], a[112:119], a[48:55], v[48:51]\n"
      "ds_read_b64_tr_b8 a[108:109], v122 offset:9216\n"
      "ds_read_b64_tr_b8 a[110:111], v122 offset:10240\n"
      "v_mfma_f32_16x16x128_f8f6f4 v[80:83], a[112:119], a[56:63], v[80:83]\n"
      "v_mfma_f32_16x16x128_f8f6f4 v[112:115], a[112:119], a[64:71], v[112:115]\n"
      "s_waitcnt lgkmcnt(6)\n"
      "v_mfma_f32_16x16x128_f8f6f4 v[20:23], a[120:127], a[40:47], v[20:23]\n"
      "v_xor_b32 v123, 112, v121\n"
      "ds_read_b64_tr_b8 a[112:113], v123 offset:1024\n"
      "ds_read_b64_tr_b8 a[114:115], v123 offset:2048\n"
      "v_mfma_f32_16x16x128_f8f6f4 v[52:55], a[120:127], a[48:55], v[52:55]\n"
      "ds_read_b64_tr_b8 a[116:117], v123 offset:9216\n"
      "ds_read_b64_tr_b8 a[118:119], v123 offset:10240\n"
      "v_mfma_f32_16x16x128_f8f6f4 v[84:87], a[120:127], a[56:63], v[84:87]\n"
      "v_add_u32 v125, 4, v125\n"
      "v_mfma_f32_16x16x128_f8f6f4 v[116:119], a[120:127], a[64:71], v[116:119]\n"
      "s_waitcnt lgkmcnt(4)\n"
      "v_readfirstlane_b32 s84, v126\n"
      "v_add_u32 v121, 0x4000, v121\n"
      "s_cmp_lt_u32 s84, s85\n"
      "s_cbranch_scc1 L_slowFS1_%=\n"
      "L_backFS1_%=:\n"
      "v_and_b32 v121, 0x1ffff, v121\n"
      "v_mfma_f32_16x16x128_f8f6f4 v[24:27], a[104:111], a[40:47], v[24:27]\n"
      "ds_read_b32 v126, v124 offset:48\n"
      "ds_read_b64_tr_b8 a[120:121], v121 offset:1024\n"
      "ds_read_b64_tr_b8 a[122:123], v121 offset:2048\n"
      "v_mfma_f32_16x16x128_f8f6f4 v[56:59], a[104:111], a[48:55], v[56:59]\n"
      "ds_read_b64_tr_b8 a[124:125], v121 offset:9216\n"
      "ds_read_b64_tr_b8 a[126:127], v121 offset:10240\n"
      "v_mfma_f32_16x16x128_f8f6f4 v[88:91], a[104:111], a[56:63], v[88:91]\n"
      "v_mfma_f32_16x16x128_f8f6f4 a[0:3], a[104:111], a[64:71], a[0:3]\n"
      "s_waitcnt lgkmcnt(5)\n"
      "v_mfma_f32_16x16x128_f8f6f4 v[28:31], a[112:119], a[40:47], v[28:31]\n"
      "v_xor_b32 v123, 16, v121\n"
      "ds_read_b64_tr_b8 a[104:105], v123 offset:1024\n"
      "ds_read_b64_tr_b8 a[106:107], v123 offset:2048\n"
      "v_mfma_f32_16x16x128_f8f6f4 v[60:63], a[112:119], a[48:55], v[60:63]\n"
      "ds_read_b64_tr_b8 a[108:109], v123 offset:9216\n"
      "ds_read_b64_tr_b8 a[110:111], v123 offset:10240\n"
      "s_waitcnt lgkmcnt(8)\n"
      "v_readfirstlane_b32 s50, v127\n"
      "v_readfirstlane_b32 s58, v126\n"
      "v_mfma_f32_16x16x128_f8f6f4 v[92:95], a[112:119], a[56:63], v[92:95]\n"
      "s_cmp_eq_u32 s50, 0\n"
      "s_cselect_b32 s53, 1, s58\n"
      "s_add_u32 s51, s50, -1\n"
      "s_add_u32 s52, s51, 4096\n"
      "v_mfma_f32_16x16x128_f8f6f4 a[4:7], a[112:119], a[64:71], a[4:7]\n"
      "s_add_u32 s85, s85, 1\n"
      "s_branch L_D2_%=\n"
      "L_SK2_%=:\n"
      "s_mov_b32 s63, s50\n"
      "s_mov_b32 s62, 1\n"
      "s_waitcnt lgkmcnt(4)\n"
      "v_mfma_f32_16x16x128_f8f6f4 v[0:3], a[120:127], a[72:79], v[0:3]\n"
      "v_xor_b32 v122, 32, v121\n"
      "ds_read_b64_tr_b8 a[112:113], v122 offset:1024\n"
      "ds_read_b64_tr_b8 a[114:115], v122 offset:2048\n"
      "v_mfma_f32_16x16x128_f8f6f4 v[32:35], a[120:127], a[80:87], v[32:35]\n"
      "ds_read_b64_tr_b8 a[116:117], v122 offset:9216\n"
      "ds_read_b64_tr_b8 a[118:119], v122 offset:10240\n"
      "v_mfma_f32_16x16x128_f8f6f4 v[64:67], a[120:127], a[88:95], v[64:67]\n"
      "v_mfma_f32_16x16x128_f8f6f4 v[96:99], a[120:127], a[96:103], v[96:99]\n"
      "s_waitcnt lgkmcnt(4)\n"
      "v_mfma_f32_16x16x128_f8f6f4 v[4:7], a[104:111], a[72:79], v[4:7]\n"
      "v_xor_b32 v123, 48, v121\n"
      "ds_read_b64_tr_b8 a[120:121], v123 offset:1024\n"
      "ds_read_b64_tr_b8 a[122:123], v123 offset:2048\n"
      "v_mfma_f32_16x16x128_f8f6f4 v[36:39], a[104:111], a[80:87], v[36:39]\n"
      "ds_read_b64_tr_b8 a[124:125], v123 offset:9216\n"
      "ds_read_b64_tr_b8 a[126:127], v123 offset:10240\n"
      "v_mfma_f32_16x16x128_f8f6f4 v[68:71], a[104:111], a[88:95], v[68:71]\n"
      "v_mfma_f32_16x16x128_f8f6f4 v[100:103], a[104:111], a[96:103], v[100:103]\n"
      "s_waitcnt lgkmcnt(4)\n"
      "v_mfma_f32_16x16x128_f8f6f4 v[8:11], a[112:119], a[72:79], v[8:11]\n"
      "v_xor_b32 v122, 64, v121\n"
      "ds_read_b64_tr_b8 a[104:105], v122 offset:1024\n"
      "ds_read_b64_tr_b8 a[106:107], v122 offset:2048\n"
      "v_mfma_f32_16x16x128_f8f6f4 v[40:43], a[112:119], a[80:87], v[40:43]\n"
      "ds_read_b64_tr_b8 a[108:109], v122 offset:9216\n"
      "ds_read_b64_tr_b8 a[110:111], v122 offset:10240\n"
      "v_mfma_f32_16x16x128_f8f6f4 v[72:75], a[112:119], a[88:95], v[72:75]\n"
      "v_mfma_f32_16x16x128_f8f6f4 v[104:107], a[112:119], a[96:103], v[104:107]\n"
      "ds_write_b32 v124, v125 offset:16\n"
      "s_waitcnt lgkmcnt(5)\n"
      "v_mfma_f32_16x16x128_f8f6f4 v[12:15], a[120:127], a[72:79], v[12:15]\n"
      "v_xor_b32 v123, 80, v121\n"
      "ds_read_b64_tr_b8 a[112:113], v123 offset:1024\n"
      "ds_read_b64_tr_b8 a[114:115], v123 offset:2048\n"
      "v_mfma_f32_16x16x128_f8f6f4 v[44:47], a[120:127], a[80:87], v[44:47]\n"
      "ds_read_b64_tr_b8 a[116:117], v123 offset:9216\n"
      "ds_read_b64_tr_b8 a[118:119], v123 offset:10240\n"
      "v_mfma_f32_16x16x128_f8f6f4 v[76:79], a[120:127], a[88:95], v[76:79]\n"
      "v_mfma_f32_16x16x128_f8f6f4 v[108:111], a[120:127], a[96:103], v[108:111]\n"
      "s_waitcnt lgkmcnt(5)\n"
      "v_mfma_f32_16x16x128_f8f6f4 v[16:19], a[104:111], a[72:79], v[16:19]\n"
      "ds_read_b32 v126, v124 offset:0\n"
      "ds_read_b32 v127, v125 offset:76\n"
      "v_xor_b32 v122, 96, v121\n"
      "ds_read_b64_tr_b8 a[120:121], v122 offset:1024\n"
      "ds_read_b64_tr_b8 a[122:123], v122 offset:2048\n"
      "v_mfma_f32_16x16x128_f8f6f4 v[48:51], a[104:111], a[80:87], v[48:51]\n"
      "ds_read_b64_tr_b8 a[124:125], v122 offset:9216\n"
      "ds_read_b64_tr_b8 a[126:127], v122 offset:10240\n"
      "v_mfma_f32_16x16x128_f8f6f4 v[80:83], a[104:111], a[88:95], v[80:83]\n"
      "v_mfma_f32_16x16x128_f8f6f4 v[112:115], a[104:111], a[96:103], v[112:115]\n"
      "s_waitcnt lgkmcnt(6)\n"
      "v_mfma_f32_16x16x128_f8f6f4 v[20:23], a[112:119], a[72:79], v[20:23]\n"
      "v_xor_b32 v123, 112, v121\n"
      "ds_read_b64_tr_b8 a[104:105], v123 offset:1024\n"
      "ds_read_b64_tr_b8 a[106:107], v123 offset:2048\n"
      "v_mfma_f32_16x16x128_f8f6f4 v[52:55], a[112:119], a[80:87], v[52:55]\n"
      "ds_read_b64_tr_b8 a[108:109], v123 offset:9216\n"
      "ds_read_b64_tr_b8 a[110:111], v123 offset:10240\n"
      "v_mfma_f32_16x16x128_f8f6f4 v[84:87], a[112:119], a[88:95], v[84:87]\n"
      "v_add_u32 v125, 4, v125\n"
      "v_mfma_f32_16x16x128_f8f6f4 v[116:119], a[112:119], a[96:103], v[116:119]\n"
      "s_waitcnt lgkmcnt(4)\n"
      "v_readfirstlane_b32 s84, v126\n"
      "v_add_u32 v121, 0x4000, v121\n"
      "s_cmp_lt_u32 s84, s85\n"
      "s_cbranch_scc1 L_slowFK2_%=\n"
      "L_backFK2_%=:\n"
      "v_and_b32 v121, 0x1ffff, v121\n"
      "v_mfma_f32_16x16x128_f8f6f4 v[24:27], a[120:127], a[72:79], v[24:27]\n"
      "ds_read_b32 v126, v124 offset:48\n"
      "ds_read_b64_tr_b8 a[112:113], v121 offset:1024\n"
      "ds_read_b64_tr_b8 a[114:115], v121 offset:2048\n"
      "v_mfma_f32_16x16x128_f8f6f4 v[56:59], a[120:127], a[80:87], v[56:59]\n"
      "ds_read_b64_tr_b8 a[116:117], v121 offset:9216\n"
      "ds_read_b64_tr_b8 a[118:119], v121 offset:10240\n"
      "v_mfma_f32_16x16x128_f8f6f4 v[88:91], a[120:127], a[88:95], v[88:91]\n"
      "v_mfma_f32_16x16x128_f8f6f4 a[0:3], a[120:127], a[96:103], a[0:3]\n"
      "s_waitcnt lgkmcnt(5)\n"
      "v_mfma_f32_16x16x128_f8f6f4 v[28:31], a[104:111], a[72:79], v[28:31]\n"
      "v_xor_b32 v123, 16, v121\n"
      "ds_read_b64_tr_b8 a[120:121], v123 offset:1024\n"
      "ds_read_b64_tr_b8 a[122:123], v123 offset:2048\n"
      "v_mfma_f32_16x16x128_f8f6f4 v[60:63], a[104:111], a[80:87], v[60:63]\n"
      "ds_read_b64_tr_b8 a[124:125], v123 offset:9216\n"
      "ds_read_b64_tr_b8 a[126:127], v123 offset:10240\n"
      "s_waitcnt lgkmcnt(8)\n"
      "v_readfirstlane_b32 s50, v127\n"
      "v_readfirstlane_b32 s58, v126\n"
      "v_mfma_f32_16x16x128_f8f6f4 v[92:95], a[104:111], a[88:95], v[92:95]\n"
      "s_cmp_eq_u32 s50, 0\n"
      "s_cselect_b32 s53, 1, s58\n"
      "s_add_u32 s51, s50, -1\n"
      "s_add_u32 s52, s51, 4096\n"
      "v_mfma_f32_16x16x128_f8f6f4 a[4:7], a[104:111], a[96:103], a[4:7]\n"
      "s_add_u32 s85, s85, 1\n"
      "s_cmp_lt_u32 s85, 31\n"
      "s_cbranch_scc1 L_D0_%=\n"
      "s_branch L_EPI_%=\n"
      "L_ON2_%=:\n"
      "s_waitcnt vmcnt(0)\n"
      "s_mov_b32 s79, s50\n"
      "s_cmp_eq_u32 s63, 0\n"
      "s_cbranch_scc1 L_slowO2_%=\n"
      "L_backO2_%=:\n"
      "s_add_u32 s63, s63, -1\n"
      "s_add_u32 s59, s63, 4096\n"
      "s_cmp_eq_u32 s79, 0\n"
      "s_cbranch_scc1 L_OS2_%=\n"
      "L_backA2_%=:\n"
      "s_add_u32 s79, s79, -1\n"
      "s_add_u32 s78, s79, 4096\n"
      "s_mov_b32 s62, 0\n"
      "s_cmp_lg_u32 s58, 0\n"
      "s_cbranch_scc1 L_OL2_%=\n"
      "s_waitcnt lgkmcnt(4)\n"
      "v_mfma_f32_16x16x128_f8f6f4 v[0:3], a[120:127], a[72:79], v[0:3]\n"
      "v_xor_b32 v122, 32, v121\n"
      "ds_read_b64_tr_b8 a[112:113], v122 offset:1024\n"
      "ds_read_b64_tr_b8 a[114:115], v122 offset:2048\n"
      "buffer_load_dwordx4 a[8:11], v120, s[68:71], s63 offen offset:0 sc0 sc1\n"
      "v_mfma_f32_16x16x128_f8f6f4 v[32:35], a[120:127], a[80:87], v[32:35]\n"
      "ds_read_b64_tr_b8 a[116:117], v122 offset:9216\n"
      "ds_read_b64_tr_b8 a[118:119], v122 offset:10240\n"
      "buffer_load_dwordx4 a[12:15], v120, s[68:71], s63 offen offset:1024 sc0 sc1\n"
      "v_mfma_f32_16x16x128_f8f6f4 v[64:67], a[120:127], a[88:95], v[64:67]\n"
      "buffer_load_dwordx4 a[16:19], v120, s[68:71], s63 offen offset:2048 sc0 sc1\n"
      "v_mfma_f32_16x16x128_f8f6f4 v[96:99], a[120:127], a[96:103], v[96:99]\n"
      "buffer_load_dwordx4 a[20:23], v120, s[68:71], s63 offen offset:3072 sc0 sc1\n"
      "s_waitcnt lgkmcnt(4)\n"
      "v_mfma_f32_16x16x128_f8f6f4 v[4:7], a[104:111], a[72:79], v[4:7]\n"
      "v_xor_b32 v123, 48, v121\n"
      "ds_read_b64_tr_b8 a[120:121], v123 offset:1024\n"
      "ds_read_b64_tr_b8 a[122:123], v123 offset:2048\n"
      "buffer_load_dwordx4 a[24:27], v120, s[68:71], s59 offen offset:0 sc0 sc1\n"
      "v_mfma_f32_16x16x128_f8f6f4 v[36:39], a[104:111], a[80:87], v[36:39]\n"
      "ds_read_b64_tr_b8 a[124:125], v123 offset:9216\n"
      "ds_read_b64_tr_b8 a[126:127], v123 offset:10240\n"
      "buffer_load_dwordx4 a[28:31], v120, s[68:71], s59 offen offset:1024 sc0 sc1\n"
      "v_mfma_f32_16x16x128_f8f6f4 v[68:71], a[104:111], a[88:95], v[68:71]\n"
      "buffer_load_dwordx4 a[32:35], v120, s[68:71], s59 offen offset:2048 sc0 sc1\n"
      "v_mfma_f32_16x16x128_f8f6f4 v[100:103], a[104:111], a[96:103], v[100:103]\n"
      "buffer_load_dwordx4 a[36:39], v120, s[68:71], s59 offen offset:3072 sc0 sc1\n"
      "s_waitcnt lgkmcnt(4)\n"
      "v_mfma_f32_16x16x128_f8f6f4 v[8:11], a[112:119], a[72:79], v[8:11]\n"
      "v_xor_b32 v122, 64, v121\n"
      "ds_read_b64_tr_b8 a[104:105], v122 offset:1024\n"
      "ds_read_b64_tr_b8 a[106:107], v122 offset:2048\n"
      "v_mfma_f32_16x16x128_f8f6f4 v[40:43], a[112:119], a[80:87], v[40:43]\n"
      "ds_read_b64_tr_b8 a[108:109], v122 offset:9216\n"
      "ds_read_b64_tr_b8 a[110:111], v122 offset:10240\n"
      "v_mfma_f32_16x16x128_f8f6f4 v[72:75], a[112:119], a[88:95], v[72:75]\n"
      "v_mfma_f32_16x16x128_f8f6f4 v[104:107], a[112:119], a[96:103], v[104:107]\n"
      "ds_write_b32 v124, v125 offset:16\n"
      "s_waitcnt lgkmcnt(5)\n"
      "v_mfma_f32_16x16x128_f8f6f4 v[12:15], a[120:127], a[72:79], v[12:15]\n"
      "v_xor_b32 v123, 80, v121\n"
      "ds_read_b64_tr_b8 a[112:113], v123 offset:1024\n"
      "ds_read_b64_tr_b8 a[114:115], v123 offset:2048\n"
      "v_mfma_f32_16x16x128_f8f6f4 v[44:47], a[120:127], a[80:87], v[44:47]\n"
      "ds_read_b64_tr_b8 a[116:117], v123 offset:9216\n"
      "ds_read_b64_tr_b8 a[118:119], v123 offset:10240\n"
      "v_mfma_f32_16x16x128_f8f6f4 v[76:79], a[120:127], a[88:95], v[76:79]\n"
      "v_mfma_f32_16x16x128_f8f6f4 v[108:111], a[120:127], a[96:103], v[108:111]\n"
      "s_waitcnt lgkmcnt(5)\n"
      "v_mfma_f32_16x16x128_f8f6f4 v[16:19], a[104:111], a[72:79], v[16:19]\n"
      "ds_read_b32 v126, v124 offset:0\n"
      "ds_read_b32 v127, v125 offset:76\n"
      "v_xor_b32 v122, 96, v121\n"
      "ds_read_b64_tr_b8 a[120:121], v122 offset:1024\n"
      "ds_read_b64_tr_b8 a[122:123], v122 offset:2048\n"
      "buffer_load_dwordx4 a[40:43], v120, s[68:71], s79 offen offset:0 sc0 sc1\n"
      "v_mfma_f32_16x16x128_f8f6f4 v[48:51], a[104:111], a[80:87], v[48:51]\n"
      "ds_read_b64_tr_b8 a[124:125], v122 offset:9216\n"
      "ds_read_b64_tr_b8 a[126:127], v122 offset:10240\n"
      "buffer_load_dwordx4 a[44:47], v120, s[68:71], s79 offen offset:1024 sc0 sc1\n"
      "v_mfma_f32_16x16x128_f8f6f4 v[80:83], a[104:111], a[88:95], v[80:83]\n"
      "buffer_load_dwordx4 a[48:51], v120, s[68:71], s79 offen offset:2048 sc0 sc1\n"
      "v_mfma_f32_16x16x128_f8f6f4 v[112:115], a[104:111], a[96:103], v[112:115]\n"
      "buffer_load_dwordx4 a[52:55], v120, s[68:71], s79 offen offset:3072 sc0 sc1\n"
      "s_waitcnt lgkmcnt(6)\n"
      "v_mfma_f32_16x16x128_f8f6f4 v[20:23], a[112:119], a[72:79], v[20:23]\n"
      "v_xor_b32 v123, 112, v121\n"
      "ds_read_b64_tr_b8 a[104:105], v123 offset:1024\n"
      "ds_read_b64_tr_b8 a[106:107], v123 offset:2048\n"
      "buffer_load_dwordx4 a[56:59], v120, s[68:71], s78 offen offset:0 sc0 sc1\n"
      "v_mfma_f32_16x16x128_f8f6f4 v[52:55], a[112:119], a[80:87], v[52:55]\n"
      "ds_read_b64_tr_b8 a[108:109], v123 offset:9216\n"
      "ds_read_b64_tr_b8 a[110:111], v123 offset:10240\n"
      "buffer_load_dwordx4 a[60:63], v120, s[68:71], s78 offen offset:1024 sc0 sc1\n"
      "v_mfma_f32_16x16x128_f8f6f4 v[84:87], a[112:119], a[88:95], v[84:87]\n"
      "v_add_u32 v125, 4, v125\n"
      "buffer_load_dwordx4 a[64:67], v120, s[68:71], s78 offen offset:2048 sc0 sc1\n"
      "v_mfma_f32_16x16x128_f8f6f4 v[116:119], a[112:119], a[96:103], v[116:119]\n"
      "buffer_load_dwordx4 a[68:71], v120, s[68:71], s78 offen offset:3072 sc0 sc1\n"
      "s_waitcnt lgkmcnt(4)\n"
      "v_readfirstlane_b32 s84, v126\n"
      "v_add_u32 v121, 0x4000, v121\n"
      "s_cmp_lt_u32 s84, s85\n"
      "s_cbranch_scc1 L_slowFO2_%=\n"
      "L_backFO2_%=:\n"
      "v_and_b32 v121, 0x1ffff, v121\n"
      "v_mfma_f32_16x16x128_f8f6f4 v[24:27], a[120:127], a[72:79], v[24:27]\n"
      "ds_read_b32 v126, v124 offset:48\n"
      "ds_read_b64_tr_b8 a[112:113], v121 offset:1024\n"
      "ds_read_b64_tr_b8 a[114:115], v121 offset:2048\n"
      "v_mfma_f32_16x16x128_f8f6f4 v[56:59], a[120:127], a[80:87], v[56:59]\n"
      "ds_read_b64_tr_b8 a[116:117], v121 offset:9216\n"
      "ds_read_b64_tr_b8 a[118:119], v121 offset:10240\n"
      "v_mfma_f32_16x16x128_f8f6f4 v[88:91], a[120:127], a[88:95], v[88:91]\n"
      "v_mfma_f32_16x16x128_f8f6f4 a[0:3], a[120:127], a[96:103], a[0:3]\n"
      "s_waitcnt lgkmcnt(5)\n"
      "v_mfma_f32_16x16x128_f8f6f4 v[28:31], a[104:111], a[72:79], v[28:31]\n"
      "v_xor_b32 v123, 16, v121\n"
      "ds_read_b64_tr_b8 a[120:121], v123 offset:1024\n"
      "ds_read_b64_tr_b8 a[122:123], v123 offset:2048\n"
      "v_mfma_f32_16x16x128_f8f6f4 v[60:63], a[104:111], a[80:87], v[60:63]\n"
      "ds_read_b64_tr_b8 a[124:125], v123 offset:9216\n"
      "ds_read_b64_tr_b8 a[126:127], v123 offset:10240\n"
      "s_waitcnt lgkmcnt(8)\n"
      "v_readfirstlane_b32 s50, v127\n"
      "v_readfirstlane_b32 s58, v126\n"
      "v_mfma_f32_16x16x128_f8f6f4 v[92:95], a[104:111], a[88:95], v[92:95]\n"
      "s_cmp_eq_u32 s50, 0\n"
      "s_cselect_b32 s53, 1, s58\n"
      "s_add_u32 s51, s50, -1\n"
      "s_add_u32 s52, s51, 4096\n"
      "v_mfma_f32_16x16x128_f8f6f4 a[4:7], a[104:111], a[96:103], a[4:7]\n"
      "s_add_u32 s85, s85, 1\n"
      "s_cmp_lt_u32 s85, 31\n"
      "s_cbranch_scc1 L_D0_%=\n"
      "s_branch L_EPI_%=\n"
      "L_OL2_%=:\n"
      "s_waitcnt lgkmcnt(4)\n"
      "v_mfma_f32_16x16x128_f8f6f4 v[0:3], a[120:127], a[72:79], v[0:3]\n"
      "v_xor_b32 v122, 32, v121\n"
      "ds_read_b64_tr_b8 a[112:113], v122 offset:1024\n"
      "ds_read_b64_tr_b8 a[114:115], v122 offset:2048\n"
      "v_mfma_f32_16x16x128_f8f6f4 v[32:35], a[120:127], a[80:87], v[32:35]\n"
      "ds_read_b64_tr_b8 a[116:117], v122 offset:9216\n"
      "ds_read_b64_tr_b8 a[118:119], v122 offset:10240\n"
      "v_mfma_f32_16x16x128_f8f6f4 v[64:67], a[120:127], a[88:95], v[64:67]\n"
      "v_mfma_f32_16x16x128_f8f6f4 v[96:99], a[120:127], a[96:103], v[96:99]\n"
      "s_waitcnt lgkmcnt(4)\n"
      "v_mfma_f32_16x16x128_f8f6f4 v[4:7], a[104:111], a[72:79], v[4:7]\n"
      "v_xor_b32 v123, 48, v121\n"
      "ds_read_b64_tr_b8 a[120:121], v123 offset:1024\n"
      "ds_read_b64_tr_b8 a[122:123], v123 offset:2048\n"
      "v_mfma_f32_16x16x128_f8f6f4 v[36:39], a[104:111], a[80:87], v[36:39]\n"
      "ds_read_b64_tr_b8 a[124:125], v123 offset:9216\n"
      "ds_read_b64_tr_b8 a[126:127], v123 offset:10240\n"
      "v_mfma_f32_16x16x128_f8f6f4 v[68:71], a[104:111], a[88:95], v[68:71]\n"
      "v_mfma_f32_16x16x128_f8f6f4 v[100:103], a[104:111], a[96:103], v[100:103]\n"
      "s_waitcnt lgkmcnt(4)\n"
      "v_mfma_f32_16x16x128_f8f6f4 v[8:11], a[112:119], a[72:79], v[8:11]\n"
      "v_xor_b32 v122, 64, v121\n"
      "ds_read_b64_tr_b8 a[104:105], v122 offset:1024\n"
      "ds_read_b64_tr_b8 a[106:107], v122 offset:2048\n"
      "v_mfma_f32_16x16x128_f8f6f4 v[40:43], a[112:119], a[80:87], v[40:43]\n"
      "ds_read_b64_tr_b8 a[108:109], v122 offset:9216\n"
      "ds_read_b64_tr_b8 a[110:111], v122 offset:10240\n"
      "v_mfma_f32_16x16x128_f8f6f4 v[72:75], a[112:119], a[88:95], v[72:75]\n"
      "v_mfma_f32_16x16x128_f8f6f4 v[104:107], a[112:119], a[96:103], v[104:107]\n"
      "ds_write_b32 v124, v125 offset:16\n"
      "s_waitcnt lgkmcnt(5)\n"
      "v_mfma_f32_16x16x128_f8f6f4 v[12:15], a[120:127], a[72:79], v[12:15]\n"
      "v_xor_b32 v123, 80, v121\n"
      "ds_read_b64_tr_b8 a[112:113], v123 offset:1024\n"
      "ds_read_b64_tr_b8 a[114:115], v123 offset:2048\n"
      "v_mfma_f32_16x16x128_f8f6f4 v[44:47], a[120:127], a[80:87], v[44:47]\n"
      "ds_read_b64_tr_b8 a[116:117], v123 offset:9216\n"
      "ds_read_b64_tr_b8 a[118:119], v123 offset:10240\n"
      "v_mfma_f32_16x16x128_f8f6f4 v[76:79], a[120:127], a[88:95], v[76:79]\n"
      "v_mfma_f32_16x16x128_f8f6f4 v[108:111], a[120:127], a[96:103], v[108:111]\n"
      "s_waitcnt lgkmcnt(5)\n"
      "v_mfma_f32_16x16x128_f8f6f4 v[16:19], a[104:111], a[72:79], v[16:19]\n"
      "ds_read_b32 v126, v124 offset:0\n"
      "ds_read_b32 v127, v125 offset:76\n"
      "v_xor_b32 v122, 96, v121\n"
      "ds_read_b64_tr_b8 a[120:121], v122 offset:1024\n"
      "ds_read_b64_tr_b8 a[122:123], v122 offset:2048\n"
      "buffer_load_dwordx4 a[8:11], v120, s[68:71], s63 offen offset:0 sc0 sc1\n"
      "v_mfma_f32_16x16x128_f8f6f4 v[48:51], a[104:111], a[80:87], v[48:51]\n"
      "ds_read_b64_tr_b8 a[124:125], v122 offset:9216\n"
      "ds_read_b64_tr_b8 a[126:127], v122 offset:10240\n"
      "buffer_load_dwordx4 a[12:15], v120, s[68:71], s63 offen offset:1024 sc0 sc1\n"
      "v_mfma_f32_16x16x128_f8f6f4 v[80:83], a[104:111], a[88:95], v[80:83]\n"
      "buffer_load_dwordx4 a[16:19], v120, s[68:71], s63 offen offset:2048 sc0 sc1\n"
      "v_mfma_f32_16x16x128_f8f6f4 v[112:115], a[104:111], a[96:103], v[112:115]\n"
      "buffer_load_dwordx4 a[20:23], v120, s[68:71], s63 offen offset:3072 sc0 sc1\n"
      "s_waitcnt lgkmcnt(6)\n"
      "v_mfma_f32_16x16x128_f8f6f4 v[20:23], a[112:119], a[72:79], v[20:23]\n"
      "v_xor_b32 v123, 112, v121\n"
      "ds_read_b64_tr_b8 a[104:105], v123 offset:1024\n"
      "ds_read_b64_tr_b8 a[106:107], v123 offset:2048\n"
      "buffer_load_dwordx4 a[24:27], v120, s[68:71], s59 offen offset:0 sc0 sc1\n"
      "v_mfma_f32_16x16x128_f8f6f4 v[52:55], a[112:119], a[80:87], v[52:55]\n"
      "ds_read_b64_tr_b8 a[108:109], v123 offset:9216\n"
      "ds_read_b64_tr_b8 a[110:111], v123 offset:10240\n"
      "buffer_load_dwordx4 a[28:31], v120, s[68:71], s59 offen offset:1024 sc0 sc1\n"
      "v_mfma_f32_16x16x128_f8f6f4 v[84:87], a[112:119], a[88:95], v[84:87]\n"
      "v_add_u32 v125, 4, v125\n"
      "buffer_load_dwordx4 a[32:35], v120, s[68:71], s59 offen offset:2048 sc0 sc1\n"
      "v_mfma_f32_16x16x128_f8f6f4 v[116:119], a[112:119], a[96:103], v[116:119]\n"
      "buffer_load_dwordx4 a[36:39], v120, s[68:71], s59 offen offset:3072 sc0 sc1\n"
      "s_waitcnt lgkmcnt(4)\n"
      "v_readfirstlane_b32 s84, v126\n"
      "v_add_u32 v121, 0x4000, v121\n"
      "s_cmp_lt_u32 s84, s85\n"
      "s_cbranch_scc1 L_slowFL2_%=\n"
      "L_backFL2_%=:\n"
      "v_and_b32 v121, 0x1ffff, v121\n"
      "v_mfma_f32_16x16x128_f8f6f4 v[24:27], a[120:127], a[72:79], v[24:27]\n"
      "ds_read_b32 v126, v124 offset:48\n"
      "ds_read_b64_tr_b8 a[112:113], v121 offset:1024\n"
      "ds_read_b64_tr_b8 a[114:115], v121 offset:2048\n"
      "buffer_load_dwordx4 a[40:43], v120, s[68:71], s79 offen offset:0 sc0 sc1\n"
      "v_mfma_f32_16x16x128_f8f6f4 v[56:59], a[120:127], a[80:87], v[56:59]\n"
      "ds_read_b64_tr_b8 a[116:117], v121 offset:9216\n"
      "ds_read_b64_tr_b8 a[118:119], v121 offset:10240\n"
      "buffer_load_dwordx4 a[44:47], v120, s[68:71], s79 offen offset:1024 sc0 sc1\n"
      "v_mfma_f32_16x16x128_f8f6f4 v[88:91], a[120:127], a[88:95], v[88:91]\n"
      "buffer_load_dwordx4 a[48:51], v120, s[68:71], s79 offen offset:2048 sc0 sc1\n"
      "v_mfma_f32_16x16x128_f8f6f4 a[0:3], a[120:127], a[96:103], a[0:3]\n"
      "buffer_load_dwordx4 a[52:55], v120, s[68:71], s79 offen offset:3072 sc0 sc1\n"
      "s_waitcnt lgkmcnt(5)\n"
      "v_mfma_f32_16x16x128_f8f6f4 v[28:31], a[104:111], a[72:79], v[28:31]\n"
      "v_xor_b32 v123, 16, v121\n"
      "ds_read_b64_tr_b8 a[120:121], v123 offset:1024\n"
      "ds_read_b64_tr_b8 a[122:123], v123 offset:2048\n"
      "buffer_load_dwordx4 a[56:59], v120, s[68:71], s78 offen offset:0 sc0 sc1\n"
      "v_mfma_f32_16x16x128_f8f6f4 v[60:63], a[104:111], a[80:87], v[60:63]\n"
      "ds_read_b64_tr_b8 a[124:125], v123 offset:9216\n"
      "ds_read_b64_tr_b8 a[126:127], v123 offset:10240\n"
      "s_waitcnt lgkmcnt(8)\n"
      "v_readfirstlane_b32 s50, v127\n"
      "v_readfirstlane_b32 s58, v126\n"
      "buffer_load_dwordx4 a[60:63], v120, s[68:71], s78 offen offset:1024 sc0 sc1\n"
      "v_mfma_f32_16x16x128_f8f6f4 v[92:95], a[104:111], a[88:95], v[92:95]\n"
      "s_cmp_eq_u32 s50, 0\n"
      "s_cselect_b32 s53, 1, s58\n"
      "s_add_u32 s51, s50, -1\n"
      "s_add_u32 s52, s51, 4096\n"
      "buffer_load_dwordx4 a[64:67], v120, s[68:71], s78 offen offset:2048 sc0 sc1\n"
      "v_mfma_f32_16x16x128_f8f6f4 a[4:7], a[104:111], a[96:103], a[4:7]\n"
      "s_add_u32 s85, s85, 1\n"
      "buffer_load_dwordx4 a[68:71], v120, s[68:71], s78 offen offset:3072 sc0 sc1\n"
      "s_cmp_lt_u32 s85, 31\n"
      "s_cbranch_scc1 L_D0_%=\n"
      "s_branch L_EPI_%=\n"
      "L_OS2_%=:\n"
      "s_waitcnt lgkmcnt(4)\n"
      "v_mfma_f32_16x16x128_f8f6f4 v[0:3], a[120:127], a[72:79], v[0:3]\n"
      "v_xor_b32 v122, 32, v121\n"
      "ds_read_b64_tr_b8 a[112:113], v122 offset:1024\n"
      "ds_read_b64_tr_b8 a[114:115], v122 offset:2048\n"
      "buffer_load_dwordx4 a[8:11], v120, s[68:71], s63 offen offset:0 sc0 sc1\n"
      "v_mfma_f32_16x16x128_f8f6f4 v[32:35], a[120:127], a[80:87], v[32:35]\n"
      "ds_read_b64_tr_b8 a[116:117], v122 offset:9216\n"
      "ds_read_b64_tr_b8 a[118:119], v122 offset:10240\n"
      "buffer_load_dwordx4 a[12:15], v120, s[68:71], s63 offen offset:1024 sc0 sc1\n"
      "v_mfma_f32_16x16x128_f8f6f4 v[64:67], a[120:127], a[88:95], v[64:67]\n"
      "buffer_load_dwordx4 a[16:19], v120, s[68:71], s63 offen offset:2048 sc0 sc1\n"
      "v_mfma_f32_16x16x128_f8f6f4 v[96:99], a[120:127], a[96:103], v[96:99]\n"
      "buffer_load_dwordx4 a[20:23], v120, s[68:71], s63 offen offset:3072 sc0 sc1\n"
      "s_waitcnt lgkmcnt(4)\n"
      "v_mfma_f32_16x16x128_f8f6f4 v[4:7], a[104:111], a[72:79], v[4:7]\n"
      "v_xor_b32 v123, 48, v121\n"
      "ds_read_b64_tr_b8 a[120:121], v123 offset:1024\n"
      "ds_read_b64_tr_b8 a[122:123], v123 offset:2048\n"
      "buffer_load_dwordx4 a[24:27], v120, s[68:71], s59 offen offset:0 sc0 sc1\n"
      "v_mfma_f32_16x16x128_f8f6f4 v[36:39], a[104:111], a[80:87], v[36:39]\n"
      "ds_read_b64_tr_b8 a[124:125], v123 offset:9216\n"
      "ds_read_b64_tr_b8 a[126:127], v123 offset:10240\n"
      "buffer_load_dwordx4 a[28:31], v120, s[68:71], s59 offen offset:1024 sc0 sc1\n"
      "v_mfma_f32_16x16x128_f8f6f4 v[68:71], a[104:111], a[88:95], v[68:71]\n"
      "buffer_load_dwordx4 a[32:35], v120, s[68:71], s59 offen offset:2048 sc0 sc1\n"
      "v_mfma_f32_16x16x128_f8f6f4 v[100:103], a[104:111], a[96:103], v[100:103]\n"
      "buffer_load_dwordx4 a[36:39], v120, s[68:71], s59 offen offset:3072 sc0 sc1\n"
      "s_waitcnt lgkmcnt(4)\n"
      "v_mfma_f32_16x16x128_f8f6f4 v[8:11], a[112:119], a[72:79], v[8:11]\n"
      "v_xor_b32 v122, 64, v121\n"
      "ds_read_b64_tr_b8 a[104:105], v122 offset:1024\n"
      "ds_read_b64_tr_b8 a[106:107], v122 offset:2048\n"
      "v_mfma_f32_16x16x128_f8f6f4 v[40:43], a[112:119], a[80:87], v[40:43]\n"
      "ds_read_b64_tr_b8 a[108:109], v122 offset:9216\n"
      "ds_read_b64_tr_b8 a[110:111], v122 offset:10240\n"
      "v_mfma_f32_16x16x128_f8f6f4 v[72:75], a[112:119], a[88:95], v[72:75]\n"
      "v_mfma_f32_16x16x128_f8f6f4 v[104:107], a[112:119], a[96:103], v[104:107]\n"
      "ds_write_b32 v124, v125 offset:16\n"
      "s_waitcnt lgkmcnt(5)\n"
      "v_mfma_f32_16x16x128_f8f6f4 v[12:15], a[120:127], a[72:79], v[12:15]\n"
      "v_xor_b32 v123, 80, v121\n"
      "ds_read_b64_tr_b8 a[112:113], v123 offset:1024\n"
      "ds_read_b64_tr_b8 a[114:115], v123 offset:2048\n"
      "s_mov_b32 s63, 0\n"
      "v_mfma_f32_16x16x128_f8f6f4 v[44:47], a[120:127], a[80:87], v[44:47]\n"
      "ds_read_b64_tr_b8 a[116:117], v123 offset:9216\n"
      "ds_read_b64_tr_b8 a[118:119], v123 offset:10240\n"
      "v_mfma_f32_16x16x128_f8f6f4 v[76:79], a[120:127], a[88:95], v[76:79]\n"
      "v_mfma_f32_16x16x128_f8f6f4 v[108:111], a[120:127], a[96:103], v[108:111]\n"
      "s_waitcnt lgkmcnt(5)\n"
      "v_mfma_f32_16x16x128_f8f6f4 v[16:19], a[104:111], a[72:79], v[16:19]\n"
      "ds_read_b32 v126, v124 offset:0\n"
      "ds_read_b32 v127, v125 offset:76\n"
      "v_xor_b32 v122, 96, v121\n"
      "ds_read_b64_tr_b8 a[120:121], v122 offset:1024\n"
      "ds_read_b64_tr_b8 a[122:123], v122 offset:2048\n"
      "v_mfma_f32_16x16x128_f8f6f4 v[48:51], a[104:111], a[80:87], v[48:51]\n"
      "ds_read_b64_tr_b8 a[124:125], v122 offset:9216\n"
      "ds_read_b64_tr_b8 a[126:127], v122 offset:10240\n"
      "v_mfma_f32_16x16x128_f8f6f4 v[80:83], a[104:111], a[88:95], v[80:83]\n"
      "v_mfma_f32_16x16x128_f8f6f4 v[112:115], a[104:111], a[96:103], v[112:115]\n"
      "s_waitcnt lgkmcnt(6)\n"
      "v_mfma_f32_16x16x128_f8f6f4 v[20:23], a[112:119], a[72:79], v[20:23]\n"
      "v_xor_b32 v123, 112, v121\n"
      "ds_read_b64_tr_b8 a[104:105], v123 offset:1024\n"
      "ds_read_b64_tr_b8 a[106:107], v123 offset:2048\n"
      "v_mfma_f32_16x16x128_f8f6f4 v[52:55], a[112:119], a[80:87], v[52:55]\n"
      "ds_read_b64_tr_b8 a[108:109], v123 offset:9216\n"
      "ds_read_b64_tr_b8 a[110:111], v123 offset:10240\n"
      "v_mfma_f32_16x16x128_f8f6f4 v[84:87], a[112:119], a[88:95], v[84:87]\n"
      "v_add_u32 v125, 4, v125\n"
      "v_mfma_f32_16x16x128_f8f6f4 v[116:119], a[112:119], a[96:103], v[116:119]\n"
      "s_waitcnt lgkmcnt(4)\n"
      "v_readfirstlane_b32 s84, v126\n"
      "v_add_u32 v121, 0x4000, v121\n"
      "s_cmp_lt_u32 s84, s85\n"
      "s_cbranch_scc1 L_slowFS2_%=\n"
      "L_backFS2_%=:\n"
      "v_and_b32 v121, 0x1ffff, v121\n"
      "v_mfma_f32_16x16x128_f8f6f4 v[24:27], a[120:127], a[72:79], v[24:27]\n"
      "ds_read_b32 v126, v124 offset:48\n"
      "ds_read_b64_tr_b8 a[112:113], v121 offset:1024\n"
      "ds_read_b64_tr_b8 a[114:115], v121 offset:2048\n"
      "v_mfma_f32_16x16x128_f8f6f4 v[56:59], a[120:127], a[80:87], v[56:59]\n"
      "ds_read_b64_tr_b8 a[116:117], v121 offset:9216\n"
      "ds_read_b64_tr_b8 a[118:119], v121 offset:10240\n"
      "v_mfma_f32_16x16x128_f8f6f4 v[88:91], a[120:127], a[88:95], v[88:91]\n"
      "v_mfma_f32_16x16x128_f8f6f4 a[0:3], a[120:127], a[96:103], a[0:3]\n"
      "s_waitcnt lgkmcnt(5)\n"
      "v_mfma_f32_16x16x128_f8f6f4 v[28:31], a[104:111], a[72:79], v[28:31]\n"
      "v_xor_b32 v123, 16, v121\n"
      "ds_read_b64_tr_b8 a[120:121], v123 offset:1024\n"
      "ds_read_b64_tr_b8 a[122:123], v123 offset:2048\n"
      "v_mfma_f32_16x16x128_f8f6f4 v[60:63], a[104:111], a[80:87], v[60:63]\n"
      "ds_read_b64_tr_b8 a[124:125], v123 offset:9216\n"
      "ds_read_b64_tr_b8 a[126:127], v123 offset:10240\n"
      "s_waitcnt lgkmcnt(8)\n"
      "v_readfirstlane_b32 s50, v127\n"
      "v_readfirstlane_b32 s58, v126\n"
      "v_mfma_f32_16x16x128_f8f6f4 v[92:95], a[104:111], a[88:95], v[92:95]\n"
      "s_cmp_eq_u32 s50, 0\n"
      "s_cselect_b32 s53, 1, s58\n"
      "s_add_u32 s51, s50, -1\n"
      "s_add_u32 s52, s51, 4096\n"
      "v_mfma_f32_16x16x128_f8f6f4 a[4:7], a[104:111], a[96:103], a[4:7]\n"
      "s_add_u32 s85, s85, 1\n"
      "s_cmp_lt_u32 s85, 31\n"
      "s_cbranch_scc1 L_D0_%=\n"
      "s_branch L_EPI_%=\n"
      "L_slowFp0_%=:\n"
      "s_mov_b32 s86, 0\n"
      "L_slowFp0l_%=:\n"
      "ds_read_b32 v126, v124 offset:0\n"
      "s_waitcnt lgkmcnt(0)\n"
      "v_readfirstlane_b32 s84, v126\n"
      "s_cmp_ge_u32 s84, s85\n"
      "s_cbranch_scc1 L_backFp0_%=\n"
      "s_sleep 1\n"
      "s_add_u32 s86, s86, 1\n"
      "s_cmp_lt_u32 s86, 0x800000\n"
      "s_cbranch_scc1 L_slowFp0l_%=\n"
      "s_mov_b32 s87, 1\n"
      "s_branch L_backFp0_%=\n"
      "L_slowFp1_%=:\n"
      "s_mov_b32 s86, 0\n"
      "L_slowFp1l_%=:\n"
      "ds_read_b32 v126, v124 offset:0\n"
      "s_waitcnt lgkmcnt(0)\n"
      "v_readfirstlane_b32 s84, v126\n"
      "s_cmp_ge_u32 s84, s85\n"
      "s_cbranch_scc1 L_backFp1_%=\n"
      "s_sleep 1\n"
      "s_add_u32 s86, s86, 1\n"
      "s_cmp_lt_u32 s86, 0x800000\n"
      "s_cbranch_scc1 L_slowFp1l_%=\n"
      "s_mov_b32 s87, 1\n"
      "s_branch L_backFp1_%=\n"
      "L_slowFN0_%=:\n"
      "s_mov_b32 s86, 0\n"
      "L_slowFN0l_%=:\n"
      "ds_read_b32 v126, v124 offset:0\n"
      "s_waitcnt lgkmcnt(0)\n"
      "v_readfirstlane_b32 s84, v126\n"
      "s_cmp_ge_u32 s84, s85\n"
      "s_cbranch_scc1 L_backFN0_%=\n"
      "s_sleep 1\n"
      "s_add_u32 s86, s86, 1\n"
      "s_cmp_lt_u32 s86, 0x800000\n"
      "s_cbranch_scc1 L_slowFN0l_%=\n"
      "s_mov_b32 s87, 1\n"
      "s_branch L_backFN0_%=\n"
      "L_slowFK0_%=:\n"
      "s_mov_b32 s86, 0\n"
      "L_slowFK0l_%=:\n"
      "ds_read_b32 v126, v124 offset:0\n"
      "s_waitcnt lgkmcnt(0)\n"
      "v_readfirstlane_b32 s84, v126\n"
      "s_cmp_ge_u32 s84, s85\n"
      "s_cbranch_scc1 L_backFK0_%=\n"
      "s_sleep 1\n"
      "s_add_u32 s86, s86, 1\n"
      "s_cmp_lt_u32 s86, 0x800000\n"
      "s_cbranch_scc1 L_slowFK0l_%=\n"
      "s_mov_b32 s87, 1\n"
      "s_branch L_backFK0_%=\n"
      "L_slowO0_%=:\n"
      "s_mov_b32 s86, 0\n"
      "L_slowO0l_%=:\n"
      "ds_read_b32 v126, v125 offset:68\n"
      "s_waitcnt lgkmcnt(0)\n"
      "v_readfirstlane_b32 s63, v126\n"
      "s_cmp_lg_u32 s63, 0\n"
      "s_cbranch_scc1 L_backO0_%=\n"
      "s_sleep 1\n"
      "s_add_u32 s86, s86, 1\n"
      "s_cmp_lt_u32 s86, 0x800000\n"
      "s_cbranch_scc1 L_slowO0l_%=\n"
      "s_mov_b32 s87, 1\n"
      "s_mov_b32 s63, 1\n"
      "s_branch L_backO0_%=\n"
      "L_slowFO0_%=:\n"
      "s_mov_b32 s86, 0\n"
      "L_slowFO0l_%=:\n"
      "ds_read_b32 v126, v124 offset:0\n"
      "s_waitcnt lgkmcnt(0)\n"
      "v_readfirstlane_b32 s84, v126\n"
      "s_cmp_ge_u32 s84, s85\n"
      "s_cbranch_scc1 L_backFO0_%=\n"
      "s_sleep 1\n"
      "s_add_u32 s86, s86, 1\n"
      "s_cmp_lt_u32 s86, 0x800000\n"
      "s_cbranch_scc1 L_slowFO0l_%=\n"
      "s_mov_b32 s87, 1\n"
      "s_branch L_backFO0_%=\n"
      "L_slowFL0_%=:\n"
      "s_mov_b32 s86, 0\n"
      "L_slowFL0l_%=:\n"
      "ds_read_b32 v126, v124 offset:0\n"
      "s_waitcnt lgkmcnt(0)\n"
      "v_readfirstlane_b32 s84, v126\n"
      "s_cmp_ge_u32 s84, s85\n"
      "s_cbranch_scc1 L_backFL0_%=\n"
      "s_sleep 1\n"
      "s_add_u32 s86, s86, 1\n"
      "s_cmp_lt_u32 s86, 0x800000\n"
      "s_cbranch_scc1 L_slowFL0l_%=\n"
      "s_mov_b32 s87, 1\n"
      "s_branch L_backFL0_%=\n"
      "L_slowFS0_%=:\n"
      "s_mov_b32 s86, 0\n"
      "L_slowFS0l_%=:\n"
      "ds_read_b32 v126, v124 offset:0\n"
      "s_waitcnt lgkmcnt(0)\n"
      "v_readfirstlane_b32 s84, v126\n"
      "s_cmp_ge_u32 s84, s85\n"
      "s_cbranch_scc1 L_backFS0_%=\n"
      "s_sleep 1\n"
      "s_add_u32 s86, s86, 1\n"
      "s_cmp_lt_u32 s86, 0x800000\n"
      "s_cbranch_scc1 L_slowFS0l_%=\n"
      "s_mov_b32 s87, 1\n"
      "s_branch L_backFS0_%=\n"
      "L_slowFN1_%=:\n"
      "s_mov_b32 s86, 0\n"
      "L_slowFN1l_%=:\n"
      "ds_read_b32 v126, v124 offset:0\n"
      "s_waitcnt lgkmcnt(0)\n"
      "v_readfirstlane_b32 s84, v126\n"
      "s_cmp_ge_u32 s84, s85\n"
      "s_cbranch_scc1 L_backFN1_%=\n"
      "s_sleep 1\n"
      "s_add_u32 s86, s86, 1\n"
      "s_cmp_lt_u32 s86, 0x800000\n"
      "s_cbranch_scc1 L_slowFN1l_%=\n"
      "s_mov_b32 s87, 1\n"
      "s_branch L_backFN1_%=\n"
      "L_slowFK1_%=:\n"
      "s_mov_b32 s86, 0\n"
      "L_slowFK1l_%=:\n"
      "ds_read_b32 v126, v124 offset:0\n"
      "s_waitcnt lgkmcnt(0)\n"
      "v_readfirstlane_b32 s84, v126\n"
      "s_cmp_ge_u32 s84, s85\n"
      "s_cbranch_scc1 L_backFK1_%=\n"
      "s_sleep 1\n"
      "s_add_u32 s86, s86, 1\n"
      "s_cmp_lt_u32 s86, 0x800000\n"
      "s_cbranch_scc1 L_slowFK1l_%=\n"
      "s_mov_b32 s87, 1\n"
      "s_branch L_backFK1_%=\n"
      "L_slowO1_%=:\n"
      "s_mov_b32 s86, 0\n"
      "L_slowO1l_%=:\n"
      "ds_read_b32 v126, v125 offset:68\n"
      "s_waitcnt lgkmcnt(0)\n"
      "v_readfirstlane_b32 s63, v126\n"
      "s_cmp_lg_u32 s63, 0\n"
      "s_cbranch_scc1 L_backO1_%=\n"
      "s_sleep 1\n"
      "s_add_u32 s86, s86, 1\n"
      "s_cmp_lt_u32 s86, 0x800000\n"
      "s_cbranch_scc1 L_slowO1l_%=\n"
      "s_mov_b32 s87, 1\n"
      "s_mov_b32 s63, 1\n"
      "s_branch L_backO1_%=\n"
      "L_slowFO1_%=:\n"
      "s_mov_b32 s86, 0\n"
      "L_slowFO1l_%=:\n"
      "ds_read_b32 v126, v124 offset:0\n"
      "s_waitcnt lgkmcnt(0)\n"
      "v_readfirstlane_b32 s84, v126\n"
      "s_cmp_ge_u32 s84, s85\n"
      "s_cbranch_scc1 L_backFO1_%=\n"
      "s_sleep 1\n"
      "s_add_u32 s86, s86, 1\n"
      "s_cmp_lt_u32 s86, 0x800000\n"
      "s_cbranch_scc1 L_slowFO1l_%=\n"
      "s_mov_b32 s87, 1\n"
      "s_branch L_backFO1_%=\n"
      "L_slowFL1_%=:\n"
      "s_mov_b32 s86, 0\n"
      "L_slowFL1l_%=:\n"
      "ds_read_b32 v126, v124 offset:0\n"
      "s_waitcnt lgkmcnt(0)\n"
      "v_readfirstlane_b32 s84, v126\n"
      "s_cmp_ge_u32 s84, s85\n"
      "s_cbranch_scc1 L_backFL1_%=\n"
      "s_sleep 1\n"
      "s_add_u32 s86, s86, 1\n"
      "s_cmp_lt_u32 s86, 0x800000\n"
      "s_cbranch_scc1 L_slowFL1l_%=\n"
      "s_mov_b32 s87, 1\n"
      "s_branch L_backFL1_%=\n"
      "L_slowFS1_%=:\n"
      "s_mov_b32 s86, 0\n"
      "L_slowFS1l_%=:\n"
      "ds_read_b32 v126, v124 offset:0\n"
      "s_waitcnt lgkmcnt(0)\n"
      "v_readfirstlane_b32 s84, v126\n"
      "s_cmp_ge_u32 s84, s85\n"
      "s_cbranch_scc1 L_backFS1_%=\n"
      "s_sleep 1\n"
      "s_add_u32 s86, s86, 1\n"
      "s_cmp_lt_u32 s86, 0x800000\n"
      "s_cbranch_scc1 L_slowFS1l_%=\n"
      "s_mov_b32 s87, 1\n"
      "s_branch L_backFS1_%=\n"
      "L_slowFN2_%=:\n"
      "s_mov_b32 s86, 0\n"
      "L_slowFN2l_%=:\n"
      "ds_read_b32 v126, v124 offset:0\n"
      "s_waitcnt lgkmcnt(0)\n"
      "v_readfirstlane_b32 s84, v126\n"
      "s_cmp_ge_u32 s84, s85\n"
      "s_cbranch_scc1 L_backFN2_%=\n"
      "s_sleep 1\n"
      "s_add_u32 s86, s86, 1\n"
      "s_cmp_lt_u32 s86, 0x800000\n"
      "s_cbranch_scc1 L_slowFN2l_%=\n"
      "s_mov_b32 s87, 1\n"
      "s_branch L_backFN2_%=\n"
      "L_slowFK2_%=:\n"
      "s_mov_b32 s86, 0\n"
      "L_slowFK2l_%=:\n"
      "ds_read_b32 v126, v124 offset:0\n"
      "s_waitcnt lgkmcnt(0)\n"
      "v_readfirstlane_b32 s84, v126\n"
      "s_cmp_ge_u32 s84, s85\n"
      "s_cbranch_scc1 L_backFK2_%=\n"
      "s_sleep 1\n"
      "s_add_u32 s86, s86, 1\n"
      "s_cmp_lt_u32 s86, 0x800000\n"
      "s_cbranch_scc1 L_slowFK2l_%=\n"
      "s_mov_b32 s87, 1\n"
      "s_branch L_backFK2_%=\n"
      "L_slowO2_%=:\n"
      "s_mov_b32 s86, 0\n"
      "L_slowO2l_%=:\n"
      "ds_read_b32 v126, v125 offset:68\n"
      "s_waitcnt lgkmcnt(0)\n"
      "v_readfirstlane_b32 s63, v126\n"
      "s_cmp_lg_u32 s63, 0\n"
      "s_cbranch_scc1 L_backO2_%=\n"
      "s_sleep 1\n"
      "s_add_u32 s86, s86, 1\n"
      "s_cmp_lt_u32 s86, 0x800000\n"
      "s_cbranch_scc1 L_slowO2l_%=\n"
      "s_mov_b32 s87, 1\n"
      "s_mov_b32 s63, 1\n"
      "s_branch L_backO2_%=\n"
      "L_slowFO2_%=:\n"
      "s_mov_b32 s86, 0\n"
      "L_slowFO2l_%=:\n"
      "ds_read_b32 v126, v124 offset:0\n"
      "s_waitcnt lgkmcnt(0)\n"
      "v_readfirstlane_b32 s84, v126\n"
      "s_cmp_ge_u32 s84, s85\n"
      "s_cbranch_scc1 L_backFO2_%=\n"
      "s_sleep 1\n"
      "s_add_u32 s86, s86, 1\n"
      "s_cmp_lt_u32 s86, 0x800000\n"
      "s_cbranch_scc1 L_slowFO2l_%=\n"
      "s_mov_b32 s87, 1\n"
      "s_branch L_backFO2_%=\n"
      "L_slowFL2_%=:\n"
      "s_mov_b32 s86, 0\n"
      "L_slowFL2l_%=:\n"
      "ds_read_b32 v126, v124 offset:0\n"
      "s_waitcnt lgkmcnt(0)\n"
      "v_readfirstlane_b32 s84, v126\n"
      "s_cmp_ge_u32 s84, s85\n"
      "s_cbranch_scc1 L_backFL2_%=\n"
      "s_sleep 1\n"
      "s_add_u32 s86, s86, 1\n"
      "s_cmp_lt_u32 s86, 0x800000\n"
      "s_cbranch_scc1 L_slowFL2l_%=\n"
      "s_mov_b32 s87, 1\n"
      "s_branch L_backFL2_%=\n"
      "L_slowFS2_%=:\n"
      "s_mov_b32 s86, 0\n"
      "L_slowFS2l_%=:\n"
      "ds_read_b32 v126, v124 offset:0\n"
      "s_waitcnt lgkmcnt(0)\n"
      "v_readfirstlane_b32 s84, v126\n"
      "s_cmp_ge_u32 s84, s85\n"
      "s_cbranch_scc1 L_backFS2_%=\n"
      "s_sleep 1\n"
      "s_add_u32 s86, s86, 1\n"
      "s_cmp_lt_u32 s86, 0x800000\n"
      "s_cbranch_scc1 L_slowFS2l_%=\n"
      "s_mov_b32 s87, 1\n"
      "s_branch L_backFS2_%=\n"
      "L_slowOt_%=:\n"
      "s_mov_b32 s86, 0\n"
      "L_slowOtl_%=:\n"
      "ds_read_b32 v126, v125 offset:68\n"
      "s_waitcnt lgkmcnt(0)\n"
      "v_readfirstlane_b32 s63, v126\n"
      "s_cmp_lg_u32 s63, 0\n"
      "s_cbranch_scc1 L_backOt_%=\n"
      "s_sleep 1\n"
      "s_add_u32 s86, s86, 1\n"
      "s_cmp_lt_u32 s86, 0x800000\n"
      "s_cbranch_scc1 L_slowOtl_%=\n"
      "s_mov_b32 s87, 1\n"
      "s_mov_b32 s63, 1\n"
      "s_branch L_backOt_%=\n"
      "L_slowAt_%=:\n"
      "s_mov_b32 s86, 0\n"
      "L_slowAtl_%=:\n"
      "ds_read_b32 v127, v125 offset:72\n"
      "s_waitcnt lgkmcnt(0)\n"
      "v_readfirstlane_b32 s79, v127\n"
      "s_cmp_lg_u32 s79, 0\n"
      "s_cbranch_scc1 L_backAt_%=\n"
      "s_sleep 1\n"
      "s_add_u32 s86, s86, 1\n"
      "s_cmp_lt_u32 s86, 0x800000\n"
      "s_cbranch_scc1 L_slowAtl_%=\n"
      "s_mov_b32 s87, 1\n"
      "s_mov_b32 s79, 1\n"
      "s_branch L_backAt_%=\n"
      "L_end_%=:\n"
      :
      : [px] "s"(px), [pa] "s"(pa), [po] "s"(p.out), [psx] "s"(p.sx), [psw] "s"(p.sw),
        [xoff] "s"(xoff), [awoff] "s"(awoff), [ooff] "s"(ooff), [wave4] "s"(wave4), [stg] "s"(stg), [ost0] "s"(ost0), [ost2] "s"(ost2), [phw] "s"(phw), [ntl] "s"(ntl_c)
      : "v0", "v1", "v2", "v3", "v4", "v5", "v6", "v7", "v8", "v9", "v10", "v11", "v12", "v13", "v14", "v15", "v16", "v17", "v18", "v19", "v20", "v21", "v22", "v23", "v24", "v25", "v26", "v27", "v28", "v29", "v30", "v31", "v32", "v33", "v34", "v35", "v36", "v37", "v38", "v39", "v40", "v41", "v42", "v43", "v44", "v45", "v46", "v47", "v48", "v49", "v50", "v51", "v52", "v53", "v54", "v55", "v56", "v57", "v58", "v59", "v60", "v61", "v62", "v63", "v64", "v65", "v66", "v67", "v68", "v69", "v70", "v71", "v72", "v73", "v74", "v75", "v76", "v77", "v78", "v79", "v80", "v81", "v82", "v83", "v84", "v85", "v86", "v87", "v88", "v89", "v90", "v91", "v92", "v93", "v94", "v95", "v96", "v97", "v98", "v99", "v100", "v101", "v102", "v103", "v104", "v105", "v106", "v107", "v108", "v109", "v110", "v111", "v112", "v113", "v114", "v115", "v116", "v117", "v118", "v119", "v120", "v121", "v122", "v123", "v124", "v125", "v126", "v127", "a0", "a1", "a2", "a3", "a4", "a5", "a6", "a7", "a8", "a9", "a10", "a11", "a12", "a13", "a14", "a15", "a16", "a17", "a18", "a19", "a20", "a21", "a22", "a23", "a24", "a25", "a26", "a27", "a28", "a29", "a30", "a31", "a32", "a33", "a34", "a35", "a36", "a37", "a38", "a39", "a40", "a41", "a42", "a43", "a44", "a45", "a46", "a47", "a48", "a49", "a50", "a51", "a52", "a53", "a54", "a55", "a56", "a57", "a58", "a59", "a60", "a61", "a62", "a63", "a64", "a65", "a66", "a67", "a68", "a69", "a70", "a71", "a72", "a73", "a74", "a75", "a76", "a77", "a78", "a79", "a80", "a81", "a82", "a83", "a84", "a85", "a86", "a87", "a88", "a89", "a90", "a91", "a92", "a93", "a94", "a95", "a96", "a97", "a98", "a99", "a100", "a101", "a102", "a103", "a104", "a105", "a106", "a107", "a108", "a109", "a110", "a111", "a112", "a113", "a114", "a115", "a116", "a117", "a118", "a119", "a120", "a121", "a122", "a123", "a124", "a125", "a126", "a127", "s50", "s51", "s52", "s53", "s54", "s55", "s56", "s57", "s58", "s59", "s60", "s61", "s62", "s63", "s64", "s65", "s66", "s67", "s68", "s69", "s70", "s71", "s72", "s73", "s74", "s75", "s76", "s77", "s78", "s79", "s80", "s81", "s82", "s83", "s84", "s85", "s86", "s87", "s88", "s89", "s90", "s91", "s92", "s93", "s94", "s95", "s96", "s97", "s98", "s99", "scc", "vcc", "memory");
  } else if (wave == 4) {
    __builtin_amdgcn_s_setprio(1);
    const unsigned long long wp = (unsigned long long)p.w;
    v4i rs;
    rs[0] = rfl((unsigned)wp); rs[1] = rfl((unsigned)(wp >> 32)) & 0xffff; rs[2] = KTOT * NTOT; rs[3] = 0x20000;
    const unsigned vo0 = (unsigned)(lane >> 3) * NTOT + (unsigned)(((lane & 7) ^ ((lane >> 4) & 3)) * 16);
    const unsigned vo1 = vo0 ^ 64u;
    unsigned vw[16];
#pragma unroll
    for (int i = 0; i < 16; ++i) vw[i] = ((i & 2) ? vo1 : vo0) + (unsigned)i * 131072u - (unsigned)(i & 3) * 1024u;
    auto issue_ks = [&](int t, unsigned ks) {
      const unsigned so = ks * 128u * NTOT + nt * 128u;
      const unsigned m0v = LDS_RING + (t & 7) * 16384;
#define WL4(A, B, C, D) "buffer_load_dwordx4 " A ", %[rs], %[so] offen lds\n buffer_load_dwordx4 " B ", %[rs], %[so] offen offset:1024 lds\n" \
                        "buffer_load_dwordx4 " C ", %[rs], %[so] offen offset:2048 lds\n buffer_load_dwordx4 " D ", %[rs], %[so] offen offset:3072 lds\n"
      asm volatile(
        "s_mov_b32 m0, %[m]\n s_nop 0\n" WL4("%[v0]", "%[v1]", "%[v2]", "%[v3]")
        "s_add_u32 m0, m0, 4096\n s_nop 0\n" WL4("%[v4]", "%[v5]", "%[v6]", "%[v7]")
        "s_add_u32 m0, m0, 4096\n s_nop 0\n" WL4("%[v8]", "%[v9]", "%[v10]", "%[v11]")
        "s_add_u32 m0, m0, 4096\n s_nop 0\n" WL4("%[v12]", "%[v13]", "%[v14]", "%[v15]")
        :: [m] "s"(m0v), [rs] "s"(rs), [so] "s"(so), [v0] "v"(vw[0]), [v1] "v"(vw[1]), [v2] "v"(vw[2]), [v3] "v"(vw[3]), [v4] "v"(vw[4]), [v5] "v"(vw[5]),
           [v6] "v"(vw[6]), [v7] "v"(vw[7]), [v8] "v"(vw[8]), [v9] "v"(vw[9]), [v10] "v"(vw[10]), [v11] "v"(vw[11]), [v12] "v"(vw[12]), [v13] "v"(vw[13]),
           [v14] "v"(vw[14]), [v15] "v"(vw[15]) : "scc", "memory");
    };
    unsigned ordv = 0;
    auto issue = [&](int t) { issue_ks(t, idx_ks((unsigned)__builtin_amdgcn_readlane((int)ordv, t) - 1u, rank)); };
    auto setf = [&](unsigned v) { if (lane < 4) ctl[C_FULL + lane] = v; };
    __builtin_amdgcn_s_barrier();
    issue_ks(0, rank * 4);
    {
      const v4i rxl = make_rsrc(p.x + (size_t)rank * MLOC * KSH, MLOC * KSH);
      const unsigned vx = (unsigned)(lane >> 3) * KSH + (unsigned)(lane & 7) * 16u;
#pragma unroll
      for (int u = 0; u < 6; ++u) {
        const unsigned so = (unsigned)(mt * 256 + (nt & 15) * 16 + (u & 1) * 8) * KSH + 128u * (unsigned)(1 + (u >> 1));
        const unsigned m0v = LDS_DUMP;
        asm volatile("s_mov_b32 m0, %[m]\n s_nop 0\n buffer_load_dwordx4 %[v], %[rs], %[so] offen lds\n" :: [m] "s"(m0v), [rs] "s"(rxl), [v] "v"(vx), [so] "s"(so) : "memory");
      }
    }
    __builtin_amdgcn_s_barrier();
    if (mt == 0) {
      issue_ks(1, rank * 4 + 1);
      asm volatile("s_waitcnt vmcnt(22)" ::: "memory"); setf(1);
      issue_ks(3, rank * 4 + 3);
      asm volatile("s_waitcnt vmcnt(16)" ::: "memory"); setf(2);
      issue_ks(2, rank * 4 + 2);
      asm volatile("s_waitcnt vmcnt(0)" ::: "memory"); setf(4);
    } else {
      issue_ks(2, rank * 4 + 2);
      asm volatile("s_waitcnt vmcnt(22)" ::: "memory"); setf(1);
      asm volatile("s_waitcnt vmcnt(0)" ::: "memory");
      issue_ks(1, rank * 4 + 1);
      asm volatile("s_waitcnt vmcnt(0)" ::: "memory"); setf(3);
      issue_ks(3, rank * 4 + 3);
      asm volatile("s_waitcnt vmcnt(0)" ::: "memory"); setf(4);
    }
    unsigned mask = 0xFu;
    unsigned sched = 0u;
    int decided = 4, spec = 0;
    bool burst = false, dead = false;
    auto blkadd = [&](unsigned d) { if (lane < 4) __hip_atomic_fetch_add((LAS unsigned*)(ctl + C_BLK + lane), d, __ATOMIC_RELAXED, __HIP_MEMORY_SCOPE_WORKGROUP); };
    auto setfull = [&]() { const unsigned f = __builtin_ctz(~mask | 0x80000000u) + ((mask == 0xFFFFFFFFu) ? 1u : 0u); return f >= 32u ? 64u : f; };
    auto drain = [&]() {
      asm volatile("s_waitcnt vmcnt(0)" ::: "memory");
      if (lane < 4) ctl[C_FULL + lane] = setfull();
      if (burst) { blkadd(0xFFFFFFFFu); burst = false; }
    };
    auto sync = [&](unsigned need, int upto) {
      bool first = true; unsigned spins = 0;
      for (;;) {
        const unsigned d0 = ctl[C_DONE], d1 = ctl[C_DONE + 1], d2 = ctl[C_DONE + 2], d3 = ctl[C_DONE + 3], el = ctl[C_ELIG];
        const unsigned dm = rfl(min(min(d0, d1), min(d2, d3)));
        unsigned cand = (dead ? 0x0FFFFFFFu : rfl(el)) & ~sched;
        if (dm >= 4u * need || dead) {
          const int d_first = decided;
          while (decided <= upto && cand != 0u) {
            const unsigned idx = __builtin_ctz(cand);
            cand &= cand - 1u; sched |= 1u << idx;
            if (lane == decided) ordv = idx + 1u;
            ++decided;
          }
          if (lane >= d_first && lane < decided) ctl[C_ORD + lane] = ordv;
          if (decided > upto) break;
        }
        if (first) { drain(); first = false; }
        else if (decided == 4 && spec < 7) {
          if ((spec & 1) == mt) {
            unsigned so = idx_ks((unsigned)spec, rank) * 128u * NTOT + nt * 128u;
            const unsigned m0v = LDS_DUMP;
            asm volatile(
              "s_mov_b32 m0, %[m]\n s_nop 0\n"
#define SP "buffer_load_dwordx4 %[v0], %[rs], %[so] offen lds\n s_add_u32 %[so], %[so], 0x20000\n"
              SP SP SP SP SP SP SP SP SP SP SP SP SP SP SP SP
              "s_waitcnt vmcnt(0)\n"
              : [so] "+s"(so) : [m] "s"(m0v), [rs] "s"(rs), [v0] "v"(vo0) : "scc", "memory");
          }
          ++spec;
        }
        else __builtin_amdgcn_s_sleep(1);
        if (++spins > 0x1000000u) dead = true;
      }
    };
    auto event = [&](int n, bool lead, bool two) {
      const unsigned pend = setfull();
      if (lead && !burst) blkadd(1u);
      issue(n);
      if (two) { issue(n + 1); }
      if (two) asm volatile("s_waitcnt vmcnt(32)" ::: "memory"); else asm volatile("s_waitcnt vmcnt(16)" ::: "memory");
      if (lane < 4) ctl[C_FULL + lane] = pend;
      if (burst && !lead) blkadd(0xFFFFFFFFu);
      burst = lead;
      mask |= (two ? 3u : 1u) << n;
    };
    const int phi = nt & 1;
    auto is_lead = [&](int n) { return (((n - phi) >> 1) & 1) == mt; };
    const int sa = (3 - (int)(((unsigned long long)p.w >> 7) & 7)) & 7;
    const bool slow = (nt & 7) == sa;
    sync(0, 4);
    {
      unsigned cand = rfl(ctl[C_ELIG]) & ~sched;
      const int d_first = decided;
      while (decided <= 9 && cand != 0u) {
        const unsigned ix = __builtin_ctz(cand);
        cand &= cand - 1u; sched |= 1u << ix;
        if (lane == decided) ordv = ix + 1u;
        ++decided;
      }
      if (lane >= d_first && lane < decided) ctl[C_ORD + lane] = ordv;
    }
    sync(1, 5);
    for (int n = 4; n <= 5; ++n) event(n, is_lead(n), false);
    sync(2, 9);
    for (int n = 6; n <= 9; ++n)
      if ((slow || is_lead(n)) && !((mask >> n) & 1u)) event(n, true, !slow && (((n - phi) & 1) == 0) && n < 9);
    const int pfk = idx - (idx > 2 * sa + 1 ? 2 : 0) - (idx > 2 * sa + 17 ? 2 : 0);
    const int pfn = (!slow && pfk < 22) ? 10 + pfk : 99;
    auto prefetch = [&](int n) {
      const unsigned ks = idx_ks((unsigned)__builtin_amdgcn_readlane((int)ordv, n) - 1u, rank);
      asm volatile("s_waitcnt vmcnt(0)" ::: "memory");
      if (lane < 4) ctl[C_FULL + lane] = setfull();
      if (!burst) blkadd(1u);
      burst = true;
#pragma unroll
      for (int hh = 0; hh < 2; ++hh) {
        unsigned so = ks * 128u * NTOT + (unsigned)(xcd * 16 + sa + 8 * hh) * 128u;
        const unsigned m0v = LDS_DUMP;
        asm volatile(
          "s_mov_b32 m0, %[m]\n"
          "s_nop 0\n"
#define PL "buffer_load_dwordx4 %[v0], %[rs], %[so] offen lds\n s_add_u32 %[so], %[so], 0x20000\n"
          PL PL PL PL PL PL PL PL PL PL PL PL PL PL PL PL
          : [so] "+s"(so) : [m] "s"(m0v), [rs] "s"(rs), [v0] "v"(vo0) : "scc", "memory");
      }
    };
    if (slow) {
      for (int tau = 6; tau + 4 < 32; ++tau) {
        sync(tau, min(31, tau + 7));
        event(tau + 4, false, false);
      }
    } else {
      for (int tau = 3; tau <= 28 && mask != 0xFFFFFFFFu; ++tau) {
        const int m = tau + 6;
        const bool due = is_lead(m) && (((m - phi) & 1) == 0);
        if (!due && tau + 7 != pfn && tau != 3) continue;
        sync(tau, min(31, tau + 7));
        if (due) {
          if (m < 32) {
            if (!((mask >> m) & 1u)) event(m, true, m < 31);
            else if (m < 31) event(m + 1, true, false);
          }
          for (int n = tau + 1; n < m && n < 32; ++n)
            if (!((mask >> n) & 1u)) event(n, false, false);
        }
        if (tau == 3)
          for (int n = 6; n <= 9; ++n)
            if (!((mask >> n) & 1u)) event(n, false, false);
        if (tau + 7 == pfn) prefetch(pfn);
      }
    }
    drain();
    storer(0);
  } else if (wave == 6) {
    __builtin_amdgcn_s_setprio(0);
    const int lev = idx >> 3, q = idx & 7;
    const bool pusher = xcd < NR - 1;
    const unsigned dest = (unsigned)(rank + 1 + xcd) & 7u;
    unsigned char* dws = p.ws[dest];
    unsigned* seq = (unsigned*)(p.ws[rank] + WS_SEQ) + b;
    const unsigned long long tc0 = rt();
    if (xcd == NR - 1 && idx < NR - 1 && lane == 0)
      __hip_atomic_fetch_add((unsigned*)(p.ws[((unsigned)rank + 1u + (unsigned)idx) & 7u] + WS_HELLO + rank * 128), 1u, __ATOMIC_RELAXED, __HIP_MEMORY_SCOPE_SYSTEM);
    unsigned callv = 0;
    const v4i rx = make_rsrc(p.x + (size_t)(dest * MLOC + q * 64) * KSH + lev * 128, 64 * KSH);
    auto xload = [&]() {
      const unsigned v0 = (unsigned)(lane >> 3) * KSH + (unsigned)(((lane & 7) ^ (lane >> 4)) << 4);
#pragma unroll
      for (int u = 0; u < 8; ++u) {
        const unsigned vu = (u & 1) ? (v0 ^ 64u) : v0;
        const unsigned sou = u * 8 * KSH, m0u = LDS_PUSH + u * 1024;
        asm volatile("s_mov_b32 m0, %[m]\n s_nop 0\n buffer_load_dwordx4 %[v], %[rs], %[so] offen lds\n" :: [m] "s"(m0u), [rs] "s"(rx), [so] "s"(sou), [v] "v"(vu) : "memory");
      }
      asm volatile("global_load_dword %0, %1, off sc1" : "=v"(callv) : "v"(seq) : "memory");
    };
    if (pusher && lev == 0) xload();
    __builtin_amdgcn_s_barrier();
    if (pusher && lev > 0) {
      __builtin_amdgcn_s_barrier();
      const unsigned long long wait = lev == 1 ? 350ull : lev == 2 ? 600ull : 900ull;
      while (rt() - tc0 < wait) __builtin_amdgcn_s_sleep(4);
      xload();
    }
    if (pusher) {
      asm volatile("s_waitcnt vmcnt(0)" : "+v"(callv) :: "memory");
      const unsigned call = rfl(callv) + 1u;
      const unsigned par = call & 1u;
      const unsigned r = lane & 15, g = lane >> 4;
      LAS unsigned char* lb = (LAS unsigned char*)smem + LDS_PUSH + r * 128;
      u32x4 v[8];
#pragma unroll
      for (int u = 0; u < 8; ++u) v[u] = *(LAS u32x4*)(lb + (u >> 1) * 2048 + ((((unsigned)(4 * (u & 1)) + g) ^ (r >> 1)) << 4));
      asm volatile("s_waitcnt lgkmcnt(0)" ::: "memory");
      if (lane == 0) ctl[C_PDONE] = 1u;
      unsigned char* dst = dws + WS_A + par * A_BYTES + ((size_t)(rank * 4 + lev) * 32 + q * 4) * 2048 + lane * 16;
      unsigned* cnt = (unsigned*)(dws + WS_CNT + (size_t)(rank * 4 + lev) * 128);
      if (lev > 0) { if (lane < 4) __hip_atomic_fetch_add((LAS unsigned*)(ctl + C_BLK + lane), 1u, __ATOMIC_RELAXED, __HIP_MEMORY_SCOPE_WORKGROUP); }
#pragma unroll
      for (int u = 0; u < 8; u += 2) {
        unsigned char* d0 = dst + u * 1024;
        asm volatile("global_store_dwordx4 %0, %1, off sc0 sc1\n\tglobal_store_dwordx4 %0, %2, off offset:1024 sc0 sc1\n\ts_nop 1" :: "v"(d0), "v"(v[u]), "v"(v[u + 1]) : "memory");
      }
      if (lev == 0) __builtin_amdgcn_s_barrier();
      asm volatile("s_waitcnt vmcnt(0)" ::: "memory");
      if (lane == 0) __hip_atomic_fetch_add(cnt, 1u, __ATOMIC_RELAXED, __HIP_MEMORY_SCOPE_SYSTEM);
      if (lev > 0) {
        asm volatile("s_waitcnt vmcnt(0)" ::: "memory");
        if (lane < 4) __hip_atomic_fetch_add((LAS unsigned*)(ctl + C_BLK + lane), 0xFFFFFFFFu, __ATOMIC_RELAXED, __HIP_MEMORY_SCOPE_WORKGROUP);
      }
    } else {
      if (lane == 0) ctl[C_PDONE] = 1u;
      __builtin_amdgcn_s_barrier();
    }
    storer(2);
  } else if (wave == 7) {
    {
      unsigned long long pc;
      asm volatile("s_getpc_b64 s[96:97]\n s_add_u32 s96, s96, fused@rel32@lo+4\n s_addc_u32 s97, s97, fused@rel32@hi+12\n s_mov_b64 %0, s[96:97]" : "=s"(pc) :: "scc", "s96", "s97");
      const unsigned line = (unsigned)idx + 32u * (unsigned)lane;
      if (line < 350u) {
        unsigned d;
        asm volatile("global_load_dword %0, %1, off\n s_waitcnt vmcnt(0)" : "=v"(d) : "v"(pc + (unsigned long long)line * 128ull) : "memory");
      }
    }
    __builtin_amdgcn_s_barrier();
    __builtin_amdgcn_s_barrier();
    storer(3);
  } else {
    __builtin_amdgcn_s_setprio(0);
    unsigned char* myws = p.ws[rank];
    unsigned* seq = (unsigned*)(myws + WS_SEQ) + b;
    unsigned* tmo = (unsigned*)(myws + WS_TMO);
    ctl[lane] = (lane == C_AOFF + 32 || lane == C_AOFF + 33) ? 1u : 0u;
    ctl[64 + lane] = 0u;
    unsigned callv;
    asm volatile("global_load_dword %0, %1, off sc1" : "=v"(callv) : "v"(seq) : "memory");
    asm volatile("s_waitcnt lgkmcnt(0)" ::: "memory");
    __builtin_amdgcn_s_barrier();
    __builtin_amdgcn_s_barrier();
    asm volatile("s_waitcnt vmcnt(0)" : "+v"(callv) :: "memory");
    const unsigned call = rfl(callv) + 1u;
    const unsigned par = call & 1u;
    const unsigned ej = (unsigned)lane % 7u, elev = (unsigned)lane / 7u;
    const unsigned delay = elev == 0 ? 50u : elev == 1 ? 270u : elev == 2 ? 520u : 820u;
    const bool hlane = lane >= 32 && lane < 39;
    const unsigned long long t0 = rt();
    unsigned long long th = 0;
    bool seen = !hlane;
    bool pub = !(lane >= 4 && lane < 32);
    bool dead = false;
    for (;;) {
      const unsigned o = pub ? 0u : ctl[C_ORD + lane];
      const unsigned dn = rfl(min(min(ctl[C_DONE], ctl[C_DONE + 1]), min(ctl[C_DONE + 2], ctl[C_DONE + 3]))) >> 2;
      const unsigned ks = o ? idx_ks(o - 1u, rank) : 0u;
      const bool want = (!pub && o != 0u && (unsigned)lane <= max(dn, 2u) + 6u) || (hlane && !seen);
      const bool anyw = __any(want);
      unsigned v = 0;
      if (!dead && (want || (lane == 39 && anyw))) {
        const unsigned* cp = hlane ? (const unsigned*)(myws + WS_HELLO + (size_t)(((unsigned)rank + 1u + (unsigned)(lane - 32)) & 7u) * 128)
                           : lane == 39 ? tmo : (const unsigned*)(myws + WS_CNT + (size_t)ks * 128);
        v = __hip_atomic_load(cp, __ATOMIC_RELAXED, __HIP_MEMORY_SCOPE_SYSTEM);
      }
      const unsigned long long now = rt();
      const bool ok = dead || v >= (hlane ? call : call * 8u);
      if (want && ok) {
        if (hlane) { seen = true; th = now; }
        else { __hip_atomic_store((LAS unsigned*)(ctl + C_AOFF + lane), par * (unsigned)A_BYTES + ks * 65536u + 1u, __ATOMIC_RELEASE, __HIP_MEMORY_SCOPE_WORKGROUP); pub = true; }
      }
      if (!dead && __any((lane == 39 && anyw && v == call) || (now - t0 > DEADLINE))) {
        dead = true;
        __hip_atomic_store(tmo, call, __ATOMIC_RELAXED, __HIP_MEMORY_SCOPE_AGENT);
      }
      const unsigned long long sm = __ballot(seen);
      const unsigned long long thp = __shfl(th, 32 + (int)ej);
      const bool el = lane < 28 && (dead || (((sm >> (32 + ej)) & 1ull) && now - thp >= delay));
      const unsigned em = (unsigned)__ballot(el);
      if (lane == 0) ctl[C_ELIG] = em;
      if (__all(pub)) break;
      if (now - t0 > 2 * DEADLINE) break;
      if (!anyw) __builtin_amdgcn_s_sleep(4); else if (!__any(want && !hlane)) __builtin_amdgcn_s_sleep(16);
    }
    {
      for (unsigned spins = 0; rfl(ctl[C_PDONE]) == 0u && spins < 0x1000000u; ++spins) __builtin_amdgcn_s_sleep(1);
      const bool bad = dead || __hip_atomic_load(tmo, __ATOMIC_RELAXED, __HIP_MEMORY_SCOPE_AGENT) == call;
      if (lane == 0) { ctl[C_FINAL] = bad ? 2u : 1u; __hip_atomic_store(seq, call, __ATOMIC_RELAXED, __HIP_MEMORY_SCOPE_AGENT); }
    }
    storer(1);
  }
}

extern "C" void kernel_launch(int rank, int nranks, ncclComm_t comm, hipStream_t stream,
                              void* const* d_in, const int* in_sizes, int n_in,
                              void* d_out, int out_size, const int* in_dtypes, int out_dtype,
                              void* const* d_out_all, void* const* d_ws_all, size_t ws_bytes) {
  (void)comm; (void)in_sizes; (void)n_in; (void)in_dtypes; (void)out_dtype; (void)d_out_all;
  if (nranks != NR || (unsigned)rank >= (unsigned)NR || ws_bytes < WS_NEED || out_size != MLOC * NTOT) {
    fprintf(stderr, "unexpected configuration\n");
    return;
  }
  static bool attr_set = false;
  if (!attr_set) {
    HIP_CHECK(hipFuncSetAttribute((const void*)fused, hipFuncAttributeMaxDynamicSharedMemorySize, LDS_BYTES));
    attr_set = true;
  }
  Params p;
  p.x = (const unsigned char*)d_in[0];
  p.w = (const unsigned char*)d_in[1];
  p.sx = (const float*)d_in[2];
  p.sw = (const float*)d_in[3];
  p.out = (float*)d_out;
  for (int i = 0; i < NR; ++i) p.ws[i] = (unsigned char*)d_ws_all[i];
  p.rank = rank;
  p.pad = 0;
  hipLaunchKernelGGL(fused, dim3(256), dim3(512), LDS_BYTES, stream, p);
  HIP_CHECK(hipGetLastError());
}
```
